```python
import numpy as np

kernel_name = "gpu_gemm_ar_m4096_k4096_n16384_w8a8_dqrelu"


ABSMAX_RTOL = 0.001

COMPUTE_DTYPE = "int8"
EPILOGUE = "dequant_relu"
IN_QUANT = "int8"
OUT_QUANT = None
IN_DTYPES = {"x": "int8", "w_mat": "int8",
             "scale_x": "float32", "scale_w": "float32"}
WORLD_SIZE = 8

def setup_inputs():

    x = np.random.randint(-127, 128, size=(4096, 4096)).astype(np.int8)
    w_mat = np.random.randint(-127, 128, size=(4096, 16384)).astype(np.int8)
    scale_x = (0.02 * (1.0 + np.random.rand(1))).astype(np.float32)
    scale_w = (0.02 * (1.0 + np.random.rand(1))).astype(np.float32)
    return {"x": x, "w_mat": w_mat, "scale_x": scale_x, "scale_w": scale_w}

def reference(x, w_mat, scale_x, scale_w):
    acc = x.astype(np.float64) @ w_mat.astype(np.float64)
    y = (acc.astype(np.float32) * (scale_x * scale_w)).astype(np.float32)
    return np.maximum(y, 0.0).astype(np.float32)
```

```cpp
#include <hip/hip_runtime.h>
#include <rccl/rccl.h>
#include <cstdio>

#define LAS __attribute__((address_space(3)))
#define GAS __attribute__((address_space(1)))
typedef int i32x4 __attribute__((ext_vector_type(4)));
typedef unsigned u32x4 __attribute__((ext_vector_type(4)));
typedef float f32x4 __attribute__((ext_vector_type(4)));

namespace {
constexpr int M_ = 4096, N_ = 16384, K_ = 4096, NR = 8;
constexpr int KS = K_ / NR;
constexpr int HTB = 16384, KTB = 32768;
constexpr int STAGE_BYTES = 8 * HTB;
constexpr int LDS_BYTES = STAGE_BYTES + 1024;
constexpr int NAG = 16;
#define NWA 4
constexpr int GRID = 256;
constexpr size_t WS_CTR = 0;
constexpr size_t WS_TMO = 128 * 1024;
constexpr size_t WS_FLAG = 256 * 1024;
constexpr size_t WS_DATA = 1024 * 1024;
constexpr size_t BLKB = 32ull * KTB;
constexpr size_t PARB = 80ull * BLKB;
constexpr size_t WS_NEED = WS_DATA + 2 * PARB;
constexpr unsigned long long DEADLINE = 50000000ull;

struct Params {
  char* ws[NR];
  const signed char* x; const signed char* w; const float* sx; const float* sw;
  float* out; int rank;
};

__device__ __forceinline__ unsigned perm(unsigned hi, unsigned lo, unsigned sel) { return __builtin_amdgcn_perm(hi, lo, sel); }

__device__ __forceinline__ unsigned slot_mask(int tau) {
  const int u = (tau >> 1) < 7 ? (tau >> 1) : 7;
  unsigned cm = (2u << u) - 1u;
  unsigned dn = tau < 128 ? 0xFFu : (0xFEu << ((tau - 128) >> 1)) & 0xFFu;
  if (tau < 160) dn |= 1u;
  if (tau < 162) dn |= 2u;
  cm &= dn;
  return cm | ((cm & 0x7Fu) << 8);
}
__device__ __forceinline__ int col_of(int slot, int round) { return round < 4 ? round * 15 + slot : 60 + 2 * (slot >> 3) + (slot & 7); }
constexpr int TEND = 162;
}

__global__ __launch_bounds__(512, 2) void fused_kernel(const Params p) {
  __shared__ __attribute__((aligned(16))) unsigned char shm[LDS_BYTES];
  LAS unsigned char* lds = (LAS unsigned char*)shm;
  const int tid = threadIdx.x, wid = __builtin_amdgcn_readfirstlane(tid >> 6), lane = tid & 63;
  char* const myws = p.ws[p.rank & 7];
  unsigned call;
  { unsigned t = 0;
    if (lane == 0) t = __hip_atomic_fetch_add((unsigned*)(myws + WS_CTR + (size_t)(blockIdx.x * 8 + wid) * 64), 1u, __ATOMIC_RELAXED, __HIP_MEMORY_SCOPE_AGENT);
    call = (unsigned)__builtin_amdgcn_readfirstlane((int)t) + 1u; }
  const size_t par = WS_DATA + (size_t)(call & 1u) * PARB;
  unsigned* const tmo = (unsigned*)(myws + WS_TMO);

  if (blockIdx.x < NAG) {
    if (wid >= NWA) return;
    const int me = blockIdx.x + NAG * wid;
    const int rk = p.rank & 7;
    const unsigned dlo = (unsigned)((lane >> 4) * 4096 + (lane & 15) * 16 + (rk >> 2) * 1024 + (rk & 3) * 256), dhi = dlo + HTB;
    const unsigned wsrc = (unsigned)lane * 4u;
    const unsigned xsrc = (unsigned)(((lane >> 4) * 32 + (lane & 15)) * KS);
    char* fl_base = p.ws[lane & 7] + WS_FLAG + rk * 4;
    int tau = 0, base = 0, idx = me;
    unsigned mask = slot_mask(0); int cnt = 16 + __builtin_popcount(mask);
    auto next_unit = [&](int& blk, int& kt) -> bool {
      while (idx >= base + cnt) { base += cnt; ++tau; if (tau >= TEND) return false; mask = slot_mask(tau); cnt = (tau < 32 ? 16 : 0) + __builtin_popcount(mask); }
      int r = idx - base; idx += NAG * NWA; kt = tau & 31;
      if (tau < 32) { if (r < 16) { blk = r; return true; } r -= 16; }
      unsigned m = mask; for (int i = 0; i < r; ++i) m &= m - 1u;
      const int slot = __builtin_ctz(m);
      blk = 16 + col_of(slot, (tau - 2 * (slot & 7)) >> 5);
      return true;
    };
    unsigned r[16];
    auto issue_loads = [&](int blk, int kt) {
      if (blk < 16) {
        const signed char* s = p.x + (size_t)blk * 256 * KS + kt * 16;
#pragma unroll
        for (int c = 0; c < 4; ++c) {
          const u32x4 v = *(const GAS u32x4*)(s + (xsrc + (unsigned)(((c >> 1) * 128 + (c & 1) * 16) * KS)));
          r[4 * c] = v[0]; r[4 * c + 1] = v[1]; r[4 * c + 2] = v[2]; r[4 * c + 3] = v[3];
        }
      } else {
        const signed char* s = p.w + (size_t)kt * 16 * N_ + (blk - 16) * 256;
#pragma unroll
        for (int j = 0; j < 16; ++j) r[j] = *(const GAS unsigned*)(s + (wsrc + (unsigned)(j * N_)));
      }
    };
    int blk, kt;
    bool have = next_unit(blk, kt);
    if (have) issue_loads(blk, kt);
    while (have) {
      u32x4 v[4];
      if (blk < 16) {
#pragma unroll
        for (int c = 0; c < 4; ++c) v[c] = (u32x4){r[4 * c], r[4 * c + 1], r[4 * c + 2], r[4 * c + 3]};
      } else {
#pragma unroll
        for (int q = 0; q < 4; ++q) {
          const unsigned t0 = perm(r[4 * q + 1], r[4 * q], 0x05010400u), t1 = perm(r[4 * q + 1], r[4 * q], 0x07030602u);
          const unsigned t2 = perm(r[4 * q + 3], r[4 * q + 2], 0x05010400u), t3 = perm(r[4 * q + 3], r[4 * q + 2], 0x07030602u);
          v[0][q] = perm(t2, t0, 0x05040100u); v[1][q] = perm(t2, t0, 0x07060302u);
          v[2][q] = perm(t3, t1, 0x05040100u); v[3][q] = perm(t3, t1, 0x07060302u);
        }
      }
      const size_t uoff = par + (size_t)(blk * 32 + kt) * KTB;
#pragma unroll
      for (int q = 1; q <= NR; ++q) {
        char* d = p.ws[(rk + q + me) & 7] + uoff;
        asm volatile("global_store_dwordx4 %0, %2, %6 sc0 sc1\n\t"
                     "global_store_dwordx4 %0, %3, %6 offset:2048 sc0 sc1\n\t"
                     "global_store_dwordx4 %1, %4, %6 sc0 sc1\n\t"
                     "global_store_dwordx4 %1, %5, %6 offset:2048 sc0 sc1\n\t"
                     "s_nop 1"
                     :: "v"(dlo), "v"(dhi), "v"(v[0]), "v"(v[1]), "v"(v[2]), "v"(v[3]), "s"(d) : "memory");
      }
      const unsigned foff = (unsigned)(blk * 32 + kt) * 32u;
      have = next_unit(blk, kt);
      if (!have) asm volatile("s_waitcnt vmcnt(0)" ::: "memory");
      else { issue_loads(blk, kt);
        if (blk < 16) asm volatile("s_waitcnt vmcnt(4)" ::: "memory"); else asm volatile("s_waitcnt vmcnt(16)" ::: "memory"); }
      if (lane < NR) __hip_atomic_store((unsigned*)(fl_base + foff), call, __ATOMIC_RELAXED, __HIP_MEMORY_SCOPE_SYSTEM);
    }
    asm volatile("s_waitcnt vmcnt(0)" ::: "memory");
    return;
  }

  __builtin_amdgcn_fence(__ATOMIC_ACQUIRE, "");
  const int wr = wid >> 2, wc = wid & 3, fr = lane & 15, fq = lane >> 4;
  int pm, slot;
  { const int b = blockIdx.x - NAG, x = b & 7, r = b >> 3, a = x & 3, hb = x >> 2;
    if (r < 28) { pm = 4 * a + (r & 3); slot = hb * 8 + (r >> 2); } else { pm = 4 * a + 2 * hb + (r - 28); slot = 7; } }
  const int d0 = 2 * (slot & 7), nrounds = (slot & 7) < 2 ? 5 : 4;
  const float scale = p.sx[0] * p.sw[0];
  const unsigned voff = (unsigned)(wid * 2048 + lane * 16);
  const unsigned ldsw = (unsigned)wid * 2048u;
  const int aoff = wr * 8192 + lane * 16, boff = wc * 4096 + lane * 16;
  const unsigned coff = (unsigned)((wr * 64 + 4 * fq) * (N_ * 4) + (wc * 64 + 4 * fr) * 4);
  const char* const cA = myws + par + (size_t)pm * BLKB;
  const char* const WT = myws + par + 16 * BLKB;
  const char* const flg = myws + WS_FLAG;
#define SA(b, h) (((b) * 2 + (h)) * HTB)
#define SB(b, h) ((4 + (b) * 2 + (h)) * HTB)
#define STAGE(bufoff, gbase) do { \
    __builtin_amdgcn_global_load_lds((const unsigned*)((const char*)(gbase) + voff), (LAS unsigned*)(lds + (bufoff) + ldsw), 16, 0, 17); \
    __builtin_amdgcn_global_load_lds((const unsigned*)((const char*)(gbase) + voff), (LAS unsigned*)(lds + (bufoff) + ldsw), 16, 1024, 17); } while (0)
#define LDA(dst, b, h) do { _Pragma("unroll") for (int m = 0; m < 4; ++m) _Pragma("unroll") for (int k = 0; k < 2; ++k) dst[m][k] = *(const LAS i32x4*)(lds + SA(b, h) + aoff + m * 2048 + k * 1024); } while (0)
#define LDB(dst, b, h) do { _Pragma("unroll") for (int n = 0; n < 2; ++n) _Pragma("unroll") for (int k = 0; k < 2; ++k) dst[n][k] = *(const LAS i32x4*)(lds + SB(b, h) + boff + n * 2048 + k * 1024); } while (0)
#define MMA(ai, bj, At, Bt, Z) do { _Pragma("unroll") for (int k = 0; k < 2; ++k) _Pragma("unroll") for (int m = 0; m < 4; ++m) _Pragma("unroll") for (int n = 0; n < 2; ++n) { \
    if ((Z) && k == 0) asm volatile("v_mfma_i32_16x16x64_i8 %0, %1, %2, 0" : "+v"(acc[ai][bj][m][n]) : "v"(At[m][k]), "v"(Bt[n][k])); \
    else acc[ai][bj][m][n] = __builtin_amdgcn_mfma_i32_16x16x64_i8(At[m][k], Bt[n][k], acc[ai][bj][m][n], 0, 0, 0); } } while (0)
#define MMA2(ai, Z) do { __builtin_amdgcn_s_setprio(1); MMA(ai, 0, At, B0, Z); MMA(ai, 1, At, B1, Z); __builtin_amdgcn_s_setprio(0); } while (0)
#define WAIT_V(n) asm volatile("s_waitcnt vmcnt(" #n ")" ::: "memory")
#define WAIT_L(n) asm volatile("s_waitcnt lgkmcnt(" #n ")" ::: "memory")
#define BAR __builtin_amdgcn_s_barrier()
#define SCHED __builtin_amdgcn_sched_barrier(0)
#define PIN do { asm volatile("" ::: "memory"); SCHED; } while (0)
#define EPI_ST(s0, s1) do { _Pragma("unroll") for (int s = (s0); s < (s1); ++s) { const int ai = s >> 4, m = (s >> 2) & 3, j = s & 3; f32x4 v; \
      v[0] = fmaxf((float)acc[ai][0][m][0][j] * scale, 0.f); v[1] = fmaxf((float)acc[ai][0][m][1][j] * scale, 0.f); \
      v[2] = fmaxf((float)acc[ai][1][m][0][j] * scale, 0.f); v[3] = fmaxf((float)acc[ai][1][m][1][j] * scale, 0.f); \
      __builtin_amdgcn_raw_buffer_store_b128(__builtin_bit_cast(u32x4, v), crs, coff, (ai * 128 + m * 16 + j) * (N_ * 4), 2); \
      asm volatile("s_nop 1" ::: "memory"); } } while (0)
  auto flag_ptr = [&](int col, int kt) -> const unsigned* {
    const unsigned o = (unsigned)(((lane & 16) ? (16 + col) : pm) * 1024 + kt * 32 + (lane & 15) * 4);
    return (const unsigned*)(flg + o); };
  auto wait_flags = [&](int col, int kt) {
    const unsigned* fp = flag_ptr(col, kt);
    const unsigned long long t0 = __builtin_amdgcn_s_memrealtime();
    for (;;) {
      const unsigned v = __hip_atomic_load(fp, __ATOMIC_RELAXED, __HIP_MEMORY_SCOPE_SYSTEM);
      if (__all((int)(v - call) >= 0)) break;
      if (__builtin_amdgcn_readfirstlane((int)__hip_atomic_load(tmo, __ATOMIC_RELAXED, __HIP_MEMORY_SCOPE_AGENT)) == (int)call
          || __builtin_amdgcn_s_memrealtime() - t0 > DEADLINE) {
        if (lane == 0) __hip_atomic_store(tmo, call, __ATOMIC_RELAXED, __HIP_MEMORY_SCOPE_AGENT);
        break;
      }
      __builtin_amdgcn_s_sleep(4);
    }
    asm volatile("s_waitcnt vmcnt(0)" ::: "memory");
  };
#define FLAG_FETCH(col, kt, sl) __builtin_amdgcn_global_load_lds(flag_ptr(col, kt), (LAS unsigned*)(lds + STAGE_BYTES + (sl) * 256), 4, 0, 17)

  i32x4 acc[2][2][4][2];
  i32x4 At[4][2], B0[2][2], B1[2][2];
  int pn = col_of(slot, 0);
  const char* cB = WT + (size_t)pn * BLKB;
  int kt = d0;
  if (wid == 0) { wait_flags(pn, kt); FLAG_FETCH(pn, (kt + 2) & 31, 0); }
  asm volatile("s_waitcnt vmcnt(0)" ::: "memory");
  BAR;
  { const char* a = cA + (size_t)kt * KTB; const char* b = cB + (size_t)kt * KTB;
    STAGE(SB(0, 0), b); STAGE(SB(0, 1), b + HTB); STAGE(SA(0, 0), a); STAGE(SA(0, 1), a + HTB);
    if (wr == 1) BAR;
    WAIT_V(2); BAR;
    STAGE(SB(1, 0), b + KTB); STAGE(SA(1, 0), a + KTB); STAGE(SB(1, 1), b + HTB + KTB);
    WAIT_V(0); BAR; }
#pragma unroll
  for (int a = 0; a < 2; ++a)
#pragma unroll
    for (int b = 0; b < 2; ++b)
#pragma unroll
      for (int m = 0; m < 4; ++m)
#pragma unroll
        for (int n = 0; n < 2; ++n) acc[a][b][m][n] = (i32x4){0, 0, 0, 0};
  __amdgpu_buffer_rsrc_t crs = __builtin_amdgcn_make_buffer_rsrc((void*)p.out, 0, 0u, 0x00020000);
  unsigned fsl = 0;
#define KT(b, an, a2, b2, Z, V1, V2, e0, e1, e2, FL) do { \
    unsigned fv = 0; \
    LDB(B0, b, 0); LDB(B1, b, 1); SCHED; LDA(At, b, 0); STAGE(SA((b) ^ 1, 1), (an) + HTB); \
    if ((FL) && wid == 0) { fv = *(const LAS unsigned*)(lds + STAGE_BYTES + fsl + lane * 4); \
      __builtin_amdgcn_global_load_lds(flag_ptr(c2, k2), (LAS unsigned*)(lds + STAGE_BYTES + (fsl ^ 256u)), 4, 0, 17); } \
    PIN; EPI_ST(e0, e1); PIN; \
    WAIT_V(V1); WAIT_L(0); \
    if ((FL) && wid == 0) { if (!__all((int)(fv - call) >= 0)) wait_flags(c1, k1); } \
    BAR; MMA2(0, Z); BAR; SCHED; \
    LDA(At, b, 1); STAGE(SB(b, 0), b2); STAGE(SB(b, 1), (b2) + HTB); STAGE(SA(b, 0), a2); PIN; EPI_ST(e1, e2); PIN; \
    WAIT_V(V2); WAIT_L(0); BAR; MMA2(1, Z); BAR; SCHED; } while (0)
#define PAIR(C1, C2, Z, Va, Vb, Vc, Vd, e0, e1, e2, e3) do { \
    const int c1 = (C1), c2 = (C2), k1 = (kt + 2) & 31, k2 = (kt + 4) & 31; \
    const char* a0 = cA + (size_t)kt * KTB; const char* a1 = cA + (size_t)k1 * KTB; const char* b1 = WT + (size_t)c1 * BLKB + (size_t)k1 * KTB; \
    KT(0, a0 + KTB, a1, b1, Z, Va, Vb, e0, e1, e2, 1); \
    KT(1, a1, a1 + KTB, b1 + KTB, 0, Vc, Vd, 0, 0, e3, 0); \
    kt = k1; fsl ^= 256u; } while (0)
  for (int ui = 0;; ++ui) {
    const bool has_next = ui + 1 < nrounds;
    const int qn = has_next ? col_of(slot, ui + 1) : pn;
    PAIR(pn, pn, 1, 30, 40, 29, 18, 11, 22, 32, 0);
    crs = __builtin_amdgcn_make_buffer_rsrc((void*)(p.out + (size_t)pm * 256 * N_ + pn * 256), 0, 256u * N_ * 4u, 0x00020000);
    for (int t = 1; t < 15; ++t) PAIR(pn, t == 14 ? qn : pn, 0, 8, 8, 8, 8, 0, 0, 0, 0);
    PAIR(qn, qn, 0, 8, 8, 8, 19, 0, 0, 0, 11);
    if (!has_next) break;
    pn = qn;
  }
  EPI_ST(11, 32); PIN;
  WAIT_V(0);
  if (wr == 0) BAR;
  BAR;
  if (__builtin_amdgcn_readfirstlane((int)__hip_atomic_load(tmo, __ATOMIC_RELAXED, __HIP_MEMORY_SCOPE_AGENT)) == (int)call) {
    const float q = __builtin_nanf("");
    for (int ui = 0; ui < nrounds; ++ui) {
      float* o = p.out + (size_t)pm * 256 * N_ + col_of(slot, ui) * 256;
      for (int i = tid; i < 256 * 256; i += 512) o[(size_t)(i >> 8) * N_ + (i & 255)] = q;
    }
  }
}

extern "C" void kernel_launch(int rank, int nranks, ncclComm_t comm, hipStream_t stream,
                              void* const* d_in, const int* in_sizes, int n_in,
                              void* d_out, int out_size, const int* in_dtypes, int out_dtype,
                              void* const* d_out_all, void* const* d_ws_all, size_t ws_bytes) {
  (void)comm; (void)in_dtypes; (void)out_dtype; (void)d_out_all;
  if (nranks != NR || (unsigned)rank >= (unsigned)NR || n_in != 4 || ws_bytes < WS_NEED || out_size != M_ * N_ ||
      in_sizes[0] != M_ * KS || in_sizes[1] != KS * N_) {
    fprintf(stderr, "kernel_launch: unexpected configuration\n");
    return;
  }
  Params p;
  for (int i = 0; i < NR; ++i) p.ws[i] = (char*)d_ws_all[i];
  p.x = (const signed char*)d_in[0]; p.w = (const signed char*)d_in[1];
  p.sx = (const float*)d_in[2]; p.sw = (const float*)d_in[3];
  p.out = (float*)d_out; p.rank = rank;
  hipLaunchKernelGGL(fused_kernel, dim3(GRID), dim3(512), 0, stream, p);
}
```

```cpp
#include <hip/hip_runtime.h>
#include <rccl/rccl.h>
#include <cstdio>

#define LAS __attribute__((address_space(3)))
#define GAS __attribute__((address_space(1)))
typedef int i32x4 __attribute__((ext_vector_type(4)));
typedef unsigned u32x4 __attribute__((ext_vector_type(4)));
typedef float f32x4 __attribute__((ext_vector_type(4)));

namespace {
constexpr int M_ = 4096, N_ = 16384, K_ = 4096, NR = 8;
constexpr int KS = K_ / NR;
constexpr int HTB = 16384;
constexpr int GH = 2048;
constexpr int STAGE_BYTES = 9 * HTB;
constexpr int LDS_BYTES = STAGE_BYTES + 1024;
constexpr int NAG = 16;
constexpr int GRID = 256;
#define SLP 4
#define TD 2
#define DGAP 8
#define LT0 96
constexpr size_t WS_CTR = 0;
constexpr size_t WS_TMO = 128 * 1024;
constexpr size_t WS_QUEUE = 128 * 1024 + 256;
constexpr size_t WS_LFLAG = 128 * 1024 + 1024;
constexpr size_t WS_FLAG = 256 * 1024;
constexpr size_t WS_DATA = 1024 * 1024;
constexpr int NUNITS = 80 * 32;
constexpr int NSLOT = 6 * 512;
constexpr unsigned SRCB = NSLOT * 4096u + 256u;
constexpr unsigned KTS = 16 * 4096;
constexpr size_t PARB = 97ull << 20;
static_assert(PARB >= 8ull * SRCB, "parity region");
constexpr size_t WS_NEED = WS_DATA + 2 * PARB;
constexpr unsigned long long DEADLINE = 50000000ull;

struct Params {
  char* ws[NR];
  const signed char* x; const signed char* w; const float* sx; const float* sw;
  float* out; int rank;
};

__device__ __forceinline__ unsigned perm(unsigned hi, unsigned lo, unsigned sel) { return __builtin_amdgcn_perm(hi, lo, sel); }

__device__ __forceinline__ int start_of(int c) { return 2 * c + (c >= 4 ? DGAP : 0); }
__device__ __forceinline__ unsigned slot_mask(int tau) {
  unsigned cm = 0;
#pragma unroll
  for (int c = 0; c < 8; ++c) cm |= ((unsigned)(tau - start_of(c)) < 128u ? 1u : 0u) << c;
  return cm | ((cm & 0x7Fu) << 8);
}
__device__ __forceinline__ int step_units(int tau, unsigned mask) { return (tau < 32 ? 16 : 0) + __builtin_popcount(mask) + ((unsigned)(tau - LT0) < 32u ? 4 : 0); }
constexpr int TEND = 14 + DGAP + 128;
static_assert(LT0 >= 32 && LT0 + 32 <= TEND && TD <= 4 && DGAP % 2 == 0 && 14 + DGAP < 32, "schedule");
}

__global__ __launch_bounds__(512, 2) void fused_kernel(const Params p) {
  __shared__ __attribute__((aligned(16))) unsigned char shm[LDS_BYTES];
  LAS unsigned char* lds = (LAS unsigned char*)shm;
  const int tid = threadIdx.x, wid = __builtin_amdgcn_readfirstlane(tid >> 6), lane = tid & 63;
  const int rk = p.rank & 7;
  char* const myws = p.ws[rk];
#define STAMP0() do {} while (0)
#define STAMP() do {} while (0)
#define STAMP_END() do {} while (0)
  unsigned* const tmo = (unsigned*)(myws + WS_TMO);
  char* const fl_base = p.ws[lane & 7] + WS_FLAG + rk * 4;
  unsigned call;
  size_t par;

  {
    const int b = ((int)blockIdx.x + GRID - NAG) & (GRID - 1);
    const bool hasx = wid >= 6;
    const int ex = ((wid & 1) << 8) + b, xkt = ex >> 4, xb = ex & 15;
    u32x4 vx[4] = {};
    if (hasx) {
      const signed char* s = p.x + (size_t)xb * 256 * KS + xkt * 16;
#pragma unroll
      for (int c = 0; c < 4; ++c) vx[c] = *(const GAS u32x4*)(s + (unsigned)((lane + c * 64) * KS));
    }
    const int ew = (wid << 8) + b, wkt = ew >> 6, col = ew & 63;
    unsigned q[16];
    { const signed char* s = p.w + (size_t)wkt * 16 * N_ + col * 256;
#pragma unroll
      for (int j = 0; j < 16; ++j) q[j] = *(const GAS unsigned*)(s + (unsigned)(lane * 4 + j * N_)); }
    asm volatile("" ::: "memory");
    unsigned tkt = 0;
    if (lane == 0) tkt = __hip_atomic_fetch_add((unsigned*)(myws + WS_CTR + (size_t)(blockIdx.x * 8 + wid) * 64), 1u, __ATOMIC_RELAXED, __HIP_MEMORY_SCOPE_AGENT);
    call = (unsigned)__builtin_amdgcn_readfirstlane((int)tkt) + 1u;
    STAMP0();
    par = WS_DATA + (size_t)(call & 1u) * PARB;
    u32x4 vw[4];
#pragma unroll
    for (int t = 0; t < 4; ++t) {
      const unsigned t0 = perm(q[4 * t + 1], q[4 * t], 0x05010400u), t1 = perm(q[4 * t + 1], q[4 * t], 0x07030602u);
      const unsigned t2 = perm(q[4 * t + 3], q[4 * t + 2], 0x05010400u), t3 = perm(q[4 * t + 3], q[4 * t + 2], 0x07030602u);
      vw[0][t] = perm(t2, t0, 0x05040100u); vw[1][t] = perm(t2, t0, 0x07060302u);
      vw[2][t] = perm(t3, t1, 0x05040100u); vw[3][t] = perm(t3, t1, 0x07060302u);
    }
    const int rnd = col < 60 ? col / 15 : 4, sl = col < 60 ? col % 15 : col - 60;
    const unsigned uw = (unsigned)(512 * (1 + rnd) + wkt * 16 + sl), ux = (unsigned)ex;
    const unsigned lo = (unsigned)rk * SRCB + (unsigned)lane * 16u;
#define PUT(d, o, v) asm volatile("global_store_dwordx4 %0, %1, %5 sc0 sc1\n\tglobal_store_dwordx4 %0, %2, %5 offset:1024 sc0 sc1\n\t" \
      "global_store_dwordx4 %0, %3, %5 offset:2048 sc0 sc1\n\tglobal_store_dwordx4 %0, %4, %5 offset:3072 sc0 sc1\n\ts_nop 1" \
      :: "v"(o), "v"(v[0]), "v"(v[1]), "v"(v[2]), "v"(v[3]), "s"(d) : "memory")
    PUT(myws + par, lo + uw * 4096u, vw);
    if (hasx) PUT(myws + par, lo + ux * 4096u, vx);
    const bool dw = wkt < TD && rnd == 0 && start_of(sl & 7) <= wkt, dx = hasx && xkt < TD;
    asm volatile("s_waitcnt vmcnt(0)" ::: "memory");
    STAMP0();
    __builtin_amdgcn_s_barrier();
    STAMP0();
    if (tid == 64) __hip_atomic_store((unsigned*)(myws + WS_LFLAG) + blockIdx.x, call, __ATOMIC_RELAXED, __HIP_MEMORY_SCOPE_AGENT);
    if (dw || dx) {
      if (dx) { vw[0] = vx[0]; vw[1] = vx[1]; vw[2] = vx[2]; vw[3] = vx[3]; }
      const unsigned o = lo + (dx ? ux : uw) * 4096u;
      __builtin_amdgcn_s_sleep(1);
#pragma unroll
      for (int i = 1; i < NR; ++i) PUT(p.ws[(rk + 1 + (i + b) % (NR - 1)) & 7] + par, o, vw);
      asm volatile("s_waitcnt vmcnt(0)" ::: "memory");
      if (lane < NR) __hip_atomic_store((unsigned*)(fl_base + (size_t)(dx ? ux : uw) * 32u), call, __ATOMIC_RELAXED, __HIP_MEMORY_SCOPE_SYSTEM);
    }
  }
  auto wait_words = [&](const unsigned* fp, int nw, int stride, bool own) -> bool {
    const unsigned long long t0 = __builtin_amdgcn_s_memrealtime();
    for (;;) {
      bool ok = true;
      for (int i = 0; i < nw; ++i) { const unsigned v = __hip_atomic_load(fp + i * stride, __ATOMIC_RELAXED, __HIP_MEMORY_SCOPE_SYSTEM); ok = ok && (int)(v - call) >= 0; }
      if (__all(ok || own)) { asm volatile("s_waitcnt vmcnt(0)" ::: "memory"); return true; }
      if (__builtin_amdgcn_readfirstlane((int)__hip_atomic_load(tmo, __ATOMIC_RELAXED, __HIP_MEMORY_SCOPE_AGENT)) == (int)call
          || __builtin_amdgcn_s_memrealtime() - t0 > DEADLINE) {
        if (lane == 0) __hip_atomic_store(tmo, call, __ATOMIC_RELAXED, __HIP_MEMORY_SCOPE_AGENT);
        asm volatile("s_waitcnt vmcnt(0)" ::: "memory");
        return false;
      }
      __builtin_amdgcn_s_sleep(SLP);
    }
  };
  const unsigned* const lflag = (const unsigned*)(myws + WS_LFLAG) + lane;

  if (blockIdx.x < NAG) {
    if (wid != 0) return;
    int ndirect = 0;
    for (int t = 0; t < TD; ++t) ndirect += step_units(t, slot_mask(t));
    const unsigned npush = (unsigned)(NUNITS - ndirect);
    const unsigned qbase = (call - 1u) * (npush + (unsigned)NAG);
    const char* const qctr = myws + WS_QUEUE;
    const char* const src = myws + par + (size_t)rk * SRCB;
    char* dst[NR - 1];
#pragma unroll
    for (int q = 0; q < NR - 1; ++q) dst[q] = p.ws[(rk + 1 + (q + (int)blockIdx.x) % (NR - 1)) & 7] + par + (size_t)rk * SRCB;
    const unsigned l16 = (unsigned)lane * 16u, vzero = 0u, vone = 1u;
    int tau = TD, base = 0;
    unsigned mask = slot_mask(TD); int cnt = step_units(TD, mask);
    auto decode = [&](int idx) -> unsigned {
      while (idx >= base + cnt && tau < TEND - 1) { base += cnt; ++tau; mask = slot_mask(tau); cnt = step_units(tau, mask); }
      int r = idx - base; const int kt = tau & 31;
      if (tau < 32) { if (r < 16) return (unsigned)(kt * 16 + r); r -= 16; }
      const int nreg = __builtin_popcount(mask);
      unsigned u;
      if (r >= nreg) u = (unsigned)(512 * 5 + (tau - LT0) * 16 + (r - nreg));
      else { unsigned m = mask; for (int i = 0; i < r; ++i) m &= m - 1u;
        const int slot = __builtin_ctz(m | 0x4000u);
        u = (unsigned)(512 * (1 + ((tau - start_of(slot & 7)) >> 5)) + kt * 16 + slot); }
      return u < (unsigned)NSLOT ? u : 0u;
    };
#define CLB_A "v224", "v225", "v226", "v227", "v228", "v229", "v230", "v231", "v232", "v233", "v234", "v235", "v236", "v237", "v238", "v239"
#define CLB_B "v240", "v241", "v242", "v243", "v244", "v245", "v246", "v247", "v248", "v249", "v250", "v251", "v252", "v253", "v254", "v255"
#define CLB "memory", "v223", CLB_A, CLB_B
#define DRAW() asm volatile("s_mov_b64 exec, 1\n\tglobal_atomic_add v223, %0, %1, %2 sc0\n\ts_mov_b64 exec, -1" :: "v"(vzero), "v"(vone), "s"(qctr) : CLB)
#define LOADU(R, u) do { const unsigned _o = (u) * 4096u + l16; \
      asm volatile("global_load_dwordx4 v[%c2:%c2+3], %0, %1 sc0 sc1\n\tglobal_load_dwordx4 v[%c2+4:%c2+7], %0, %1 offset:1024 sc0 sc1\n\t" \
                   "global_load_dwordx4 v[%c2+8:%c2+11], %0, %1 offset:2048 sc0 sc1\n\tglobal_load_dwordx4 v[%c2+12:%c2+15], %0, %1 offset:3072 sc0 sc1" \
                   :: "v"(_o), "s"(src), "i"(R) : CLB); } while (0)
#define STOREU(R, u) do { const unsigned _o = (u) * 4096u + l16; _Pragma("unroll") for (int q = 0; q < NR - 1; ++q) \
      asm volatile("global_store_dwordx4 %0, v[%c2:%c2+3], %1 sc0 sc1\n\tglobal_store_dwordx4 %0, v[%c2+4:%c2+7], %1 offset:1024 sc0 sc1\n\t" \
                   "global_store_dwordx4 %0, v[%c2+8:%c2+11], %1 offset:2048 sc0 sc1\n\tglobal_store_dwordx4 %0, v[%c2+12:%c2+15], %1 offset:3072 sc0 sc1\n\ts_nop 1" \
                   :: "v"(_o), "s"(dst[q]), "i"(R) : CLB); } while (0)
#define FLAGU(u) do { char* _f = fl_base + (size_t)(u) * 32u; \
      asm volatile("s_mov_b64 exec, 0xff\n\tglobal_store_dword %0, %1, off sc0 sc1\n\ts_mov_b64 exec, -1" :: "v"(_f), "v"(call) : CLB); } while (0)
#define WAIT_TK(n, tk) asm volatile("s_waitcnt vmcnt(" #n ")\n\tv_readfirstlane_b32 %0, v223" : "=s"(tk) :: CLB)
    unsigned tk;
    DRAW(); WAIT_TK(0, tk);
    unsigned idx = tk - qbase;
    if (!wait_words(lflag, 4, 64, false)) return;
    STAMP();
    if (idx < npush) {
      unsigned cur = decode((int)idx), prev = NSLOT - 1;
      LOADU(224, cur); DRAW();
      asm volatile("s_waitcnt vmcnt(0)" ::: CLB);
#define STEP(RC, RN) { \
        WAIT_TK(29, tk); \
        idx = tk - qbase; \
        if (idx >= npush) { STOREU(RC, cur); asm volatile("s_waitcnt vmcnt(28)" ::: CLB); FLAGU(prev); break; } \
        const unsigned nxt = decode((int)idx); \
        LOADU(RN, nxt); DRAW(); \
        STOREU(RC, cur); \
        asm volatile("s_waitcnt vmcnt(28)" ::: CLB);             \
        FLAGU(prev); STAMP(); \
        prev = cur; cur = nxt; }
      for (;;) { STEP(224, 240) STEP(240, 224) }
      asm volatile("s_waitcnt vmcnt(0)" ::: CLB);
      FLAGU(cur);
    }
    STAMP_END();
    asm volatile("s_waitcnt vmcnt(0)" ::: "memory");
    return;
  }

  const int wr = wid >> 2, wc = wid & 3, fr = lane & 15, fq = lane >> 4;
  int pm, slot;
  { const int b = blockIdx.x - NAG, x = b & 7, r = b >> 3, a = x & 3, hb = x >> 2;
    if (r < 28) { pm = 4 * a + (r & 3); slot = hb * 8 + (r >> 2); } else { pm = 4 * a + 2 * hb + (r - 28); slot = 7; } }
  const bool has_half = (slot & 7) < 4;
  const int hcol = (slot >> 3) * 2 + ((slot >> 1) & 1), hrow = slot & 1;
  const float scale = p.sx[0] * p.sw[0];
  const unsigned voff = (unsigned)((wid >> 2) * 4 + fq) * SRCB + (unsigned)((wid & 3) * 256 + fr * 16);
  const unsigned ldsw = (unsigned)wid * 2048u;
  const int aoff = wr * 1024 + lane * 16;
  int boff = 4 * HTB + wc * 2048 + lane * 16; asm volatile("" : "+v"(boff));
  const unsigned coff = (unsigned)((wr * 64 + 4 * fq) * (N_ * 4) + (wc * 64 + 4 * fr) * 4);
  const char* const cA = myws + par + (size_t)pm * 4096;
  const char* const WT = myws + par + (size_t)(512 + slot) * 4096;
  const char* const hA = cA + hrow * GH;
  const char* const hB = myws + par + (size_t)(512 * 5 + hcol) * 4096;
  const char* const flg = myws + WS_FLAG;
  const bool own = (lane & 7) == rk;
#define SA(b, h) (((b) * 2 + (h)) * HTB)
#define SB(b, h) ((4 + (b) * 2 + (h)) * HTB)
#define STAGE(bufoff, gbase) do { \
    __builtin_amdgcn_global_load_lds((const unsigned*)((const char*)(gbase) + voff), (LAS unsigned*)(lds + (bufoff) + ldsw), 16, 0, 17); \
    __builtin_amdgcn_global_load_lds((const unsigned*)((const char*)(gbase) + voff), (LAS unsigned*)(lds + (bufoff) + ldsw), 16, 1024, 17); } while (0)
#define LDA_(dst, base) do { _Pragma("unroll") for (int m = 0; m < 4; ++m) _Pragma("unroll") for (int k = 0; k < 2; ++k) dst[m][k] = *(const LAS i32x4*)(lds + (base) + m * 2048 + k * 8192); } while (0)
#define LDB_(dst, base) do { _Pragma("unroll") for (int n = 0; n < 2; ++n) _Pragma("unroll") for (int k = 0; k < 2; ++k) dst[n][k] = *(const LAS i32x4*)(lds + (base) + n * 1024 + k * 8192); } while (0)
#define LDA(dst, b, h) LDA_(dst, SA(b, h) + aoff)
#define LDB(dst, b, h) LDB_(dst, SA(b, h) + boff)
#define MFMA2(c, a0, b0, a1, b1, Z) do { \
    if (Z) asm volatile("v_mfma_i32_16x16x64_i8 %0, %1, %2, 0\n\tv_mfma_i32_16x16x64_i8 %0, %3, %4, %0" : "+v"(c) : "v"(a0), "v"(b0), "v"(a1), "v"(b1)); \
    else asm volatile("v_mfma_i32_16x16x64_i8 %0, %1, %2, %0\n\tv_mfma_i32_16x16x64_i8 %0, %3, %4, %0" : "+v"(c) : "v"(a0), "v"(b0), "v"(a1), "v"(b1)); } while (0)
#define MMA2(ai, Z) do { __builtin_amdgcn_s_setprio(1); \
    _Pragma("unroll") for (int i = 0; i < 16; ++i) { \
      const int k = i & 1, m = i >> 2, c = (m & 1) ? 3 - (i & 3) : (i & 3); \
      MFMA2(acc[ai][c >> 1][m][c & 1], At[m][k], (c < 2 ? B0[c & 1][k] : B1[c & 1][k]), At[m][k ^ 1], (c < 2 ? B0[c & 1][k ^ 1] : B1[c & 1][k ^ 1]), Z); } \
    __builtin_amdgcn_s_setprio(0); } while (0)
#define WAIT_V(n) asm volatile("s_waitcnt vmcnt(" #n ")" ::: "memory")
#define WAIT_L(n) asm volatile("s_waitcnt lgkmcnt(" #n ")" ::: "memory")
#define BAR __builtin_amdgcn_s_barrier()
#define SCHED __builtin_amdgcn_sched_barrier(0)
#define PIN do { asm volatile("" ::: "memory"); SCHED; } while (0)
#define EPI_ST(s0, s1) do { _Pragma("unroll") for (int s = (s0); s < (s1); ++s) { const int ai = s >> 4, m = (s >> 2) & 3, j = s & 3; f32x4 v; \
      v[0] = fmaxf((float)acc[ai][0][m][0][j] * scale, 0.f); v[1] = fmaxf((float)acc[ai][0][m][1][j] * scale, 0.f); \
      v[2] = fmaxf((float)acc[ai][1][m][0][j] * scale, 0.f); v[3] = fmaxf((float)acc[ai][1][m][1][j] * scale, 0.f); \
      __builtin_amdgcn_raw_buffer_store_b128(__builtin_bit_cast(u32x4, v), crs, coff, (ai * 128 + m * 16 + j) * (N_ * 4), 2); \
      asm volatile("s_nop 1" ::: "memory"); } } while (0)
  auto flag_ptr = [&](int rnd, int kt) -> const unsigned* {
    const unsigned o = (unsigned)(((lane & 16) ? 512 * (1 + rnd) + slot : pm) * 32 + kt * 512 + (lane & 8) * 64 + (lane & 7) * 4);
    return (const unsigned*)(flg + o); };

  i32x4 acc[2][2][4][2];
  i32x4 At[4][2], B0[2][2], B1[2][2];
  int kt = start_of(slot & 7);
  if (wid == 0) {
    wait_words(flag_ptr(0, kt), 1, 0, own);
    STAMP0();
    wait_words(lflag, 4, 64, false);
    STAMP0();
    __builtin_amdgcn_global_load_lds(flag_ptr(0, (kt + 2) & 31), (LAS unsigned*)(lds + STAGE_BYTES), 4, 0, 17);
  }
  asm volatile("s_waitcnt vmcnt(0)" ::: "memory");
  BAR;
  { const char* a = cA + (size_t)kt * KTS; const char* b = WT + (size_t)kt * KTS;
    STAGE(SB(0, 0), b); STAGE(SB(0, 1), b + GH); STAGE(SA(0, 0), a); STAGE(SA(0, 1), a + GH);
    if (wr == 1) BAR;
    WAIT_V(2); BAR;
    STAGE(SB(1, 0), b + KTS); STAGE(SA(1, 0), a + KTS); STAGE(SB(1, 1), b + GH + KTS);
    WAIT_V(0); BAR; }
#pragma unroll
  for (int a = 0; a < 2; ++a)
#pragma unroll
    for (int b = 0; b < 2; ++b)
#pragma unroll
      for (int m = 0; m < 4; ++m)
#pragma unroll
        for (int n = 0; n < 2; ++n) acc[a][b][m][n] = (i32x4){0, 0, 0, 0};
  __amdgpu_buffer_rsrc_t crs = __builtin_amdgcn_make_buffer_rsrc((void*)p.out, 0, 0u, 0x00020000);
  unsigned fsl = 0;
#define KT(b, ah, a2, b2, Z, V1, V2, e0, e1, e2, FL) do { \
    unsigned fv = 0; \
    LDB(B0, b, 0); LDB(B1, b, 1); SCHED; LDA(At, b, 0); STAGE(SA((b) ^ 1, 1), ah); \
    if ((FL) && wid == 0) { fv = *(const LAS unsigned*)(lds + STAGE_BYTES + fsl + lane * 4); \
      __builtin_amdgcn_global_load_lds(flag_ptr(c2, k2), (LAS unsigned*)(lds + STAGE_BYTES + (fsl ^ 256u)), 4, 0, 17); } \
    PIN; EPI_ST(e0, e1); PIN; \
    WAIT_V(V1); WAIT_L(0); \
    if ((FL) && wid == 0) { if (!__all((int)(fv - call) >= 0 || own)) wait_words(flag_ptr(c1, k1), 1, 0, own); \
      if ((FL) == 2 && tohalf) wait_words((const unsigned*)(flg + (unsigned)((512 * 5 + hcol) * 32 + (lane >> 3) * 512 + (lane & 7) * 4)), 4, 1024, own); \
      STAMP(); } \
    BAR; MMA2(0, Z); BAR; SCHED; \
    LDA(At, b, 1); STAGE(SB(b, 0), b2); STAGE(SB(b, 1), (b2) + GH); STAGE(SA(b, 0), a2); PIN; EPI_ST(e1, e2); PIN; \
    WAIT_V(V2); WAIT_L(0); BAR; MMA2(1, Z); BAR; SCHED; } while (0)
#define PAIR(C1, C2, Z, Va, Vb, Vc, Vd, e0, e1, e2, e3, FL) do { \
    const int c1 = (C1), c2 = (C2), k1 = (kt + 2) & 31, k2 = (kt + 4) & 31; \
    const char* a0 = cA + (size_t)kt * KTS; const char* a1 = cA + (size_t)k1 * KTS; const char* b1 = WT + (size_t)(c1 * 32 + k1) * KTS; \
    const char* ah1 = a1 + GH; \
    if ((FL) == 2 && tohalf) { a1 = hA; b1 = hB; ah1 = hB + 2 * KTS; }     \
    KT(0, a0 + KTS + GH, a1, b1, Z, Va, Vb, e0, e1, e2, FL); \
    KT(1, ah1, a1 + KTS, b1 + KTS, 0, Vc, Vd, 0, 0, e3, 0); \
    kt = k1; fsl ^= 256u; } while (0)
  for (int ui = 0;; ++ui) {
    const bool has_next = ui < 3, tohalf = !has_next && has_half;
    const int un = has_next ? ui + 1 : ui;
    PAIR(ui, ui, 1, 30, 40, 29, 18, 11, 22, 32, 0, 1);
    crs = __builtin_amdgcn_make_buffer_rsrc((void*)(p.out + (size_t)pm * 256 * N_ + (ui * 15 + slot) * 256), 0, 256u * N_ * 4u, 0x00020000);
    for (int t = 1; t < 15; ++t) PAIR(ui, t == 14 ? un : ui, 0, 8, 8, 8, 8, 0, 0, 0, 0, 1);
    PAIR(un, un, 0, 8, 8, 8, 19, 0, 0, 0, 11, 2);
    if (!has_next) break;
  }
  if (has_half) {
    int aoff2 = 8 * HTB + aoff; asm volatile("" : "+v"(aoff2));
    const int boff2 = boff - 4 * HTB;
#define HKT(j, tn, Z, V, e0, e1) do { \
      if ((j) < 2) { LDB(B0, j, 0); LDB(B1, j, 1); SCHED; LDA(At, j, 0); } \
      else { LDB_(B0, SA(0, 1) + boff2); LDB_(B1, SA(1, 1) + boff2); SCHED; LDA_(At, aoff2); } \
      { const char* a = hA + (size_t)(tn) * KTS; const char* b = hB + (size_t)(tn) * KTS; \
        if ((j) == 0) { STAGE(SA(0, 1), b); STAGE(SA(1, 1), b + GH); STAGE(8 * HTB, a); } \
        else { STAGE(SB((j) - 1, 0), b); STAGE(SB((j) - 1, 1), b + GH); STAGE(SA((j) - 1, 0), a); } } \
      PIN; EPI_ST(e0, e1); PIN; \
      WAIT_V(V); WAIT_L(0); BAR; MMA2(0, Z); BAR; SCHED; } while (0)
    HKT(0, 2, 1, 24, 11, 18); HKT(1, 3, 0, 20, 18, 25); HKT(2, 4, 0, 20, 25, 32);
    crs = __builtin_amdgcn_make_buffer_rsrc((void*)(p.out + (size_t)(pm * 256 + hrow * 128) * N_ + (60 + hcol) * 256), 0, 128u * N_ * 4u, 0x00020000);
    for (int t = 3; t < 30; t += 3) { HKT(0, t + 2, 0, 6, 0, 0); HKT(1, t + 3, 0, 6, 0, 0); HKT(2, t + 4, 0, 6, 0, 0); if (wid == 0) STAMP(); }
    HKT(0, 31, 0, 6, 0, 0); HKT(1, 31, 0, 6, 0, 0);
    asm volatile("s_nop 7" ::: "memory"); SCHED;
    EPI_ST(0, 16); PIN;
  } else {
    asm volatile("s_nop 7" ::: "memory"); SCHED;
    EPI_ST(11, 32); PIN;
  }
  if (wid == 0) STAMP();
  WAIT_V(0);
  if (wr == 0) BAR;
  BAR;
  if (wid == 0) { STAMP(); STAMP_END(); }
  if (__builtin_amdgcn_readfirstlane((int)__hip_atomic_load(tmo, __ATOMIC_RELAXED, __HIP_MEMORY_SCOPE_AGENT)) == (int)call) {
    const float q = __builtin_nanf("");
    for (int ui = 0; ui < 4; ++ui) {
      float* o = p.out + (size_t)pm * 256 * N_ + (ui * 15 + slot) * 256;
      for (int i = tid; i < 256 * 256; i += 512) o[(size_t)(i >> 8) * N_ + (i & 255)] = q;
    }
    if (has_half) {
      float* o = p.out + (size_t)(pm * 256 + hrow * 128) * N_ + (60 + hcol) * 256;
      for (int i = tid; i < 128 * 256; i += 512) o[(size_t)(i >> 8) * N_ + (i & 255)] = q;
    }
  }
}

extern "C" void kernel_launch(int rank, int nranks, ncclComm_t comm, hipStream_t stream,
                              void* const* d_in, const int* in_sizes, int n_in,
                              void* d_out, int out_size, const int* in_dtypes, int out_dtype,
                              void* const* d_out_all, void* const* d_ws_all, size_t ws_bytes) {
  (void)comm; (void)in_dtypes; (void)out_dtype; (void)d_out_all;
  if (nranks != NR || (unsigned)rank >= (unsigned)NR || n_in != 4 || ws_bytes < WS_NEED || out_size != M_ * N_ ||
      in_sizes[0] != M_ * KS || in_sizes[1] != KS * N_) {
    fprintf(stderr, "kernel_launch: unexpected configuration\n");
    return;
  }
  Params p;
  for (int i = 0; i < NR; ++i) p.ws[i] = (char*)d_ws_all[i];
  p.x = (const signed char*)d_in[0]; p.w = (const signed char*)d_in[1];
  p.sx = (const float*)d_in[2]; p.sw = (const float*)d_in[3];
  p.out = (float*)d_out; p.rank = rank;
  hipLaunchKernelGGL(fused_kernel, dim3(GRID), dim3(512), 0, stream, p);
}
```

```cpp
#include <hip/hip_runtime.h>
#include <rccl/rccl.h>
#include <cstdio>

#define LAS __attribute__((address_space(3)))
#define GAS __attribute__((address_space(1)))
typedef int i32x4 __attribute__((ext_vector_type(4)));
typedef unsigned u32x4 __attribute__((ext_vector_type(4)));
typedef float f32x4 __attribute__((ext_vector_type(4)));

namespace {
constexpr int M_ = 4096, N_ = 16384, K_ = 4096, NR = 8;
constexpr int KS = K_ / NR;
constexpr int HTB = 16384;
constexpr int GH = 2048;
constexpr int LDS_OP = 512;
constexpr int LDS_BYTES = LDS_OP + 9 * HTB;
constexpr int BIAS = 0x3e22f983;
constexpr int NAG = 16;
constexpr int GRID = 256;
#define SLP 4
#define TD 2
#define DGAP 8
#define DSIDE 4
#define LT0 96
constexpr size_t WS_CTR = 0;
constexpr size_t WS_TMO = 128 * 1024;
constexpr size_t WS_QUEUE = 128 * 1024 + 256;
constexpr size_t WS_LFLAG = 128 * 1024 + 1024;
constexpr size_t WS_FLAG = 256 * 1024;
constexpr size_t WS_DATA = 1024 * 1024;
constexpr int KSTR = 80;
constexpr int NUNITS = 32 * KSTR;
constexpr unsigned KTS = KSTR * 4096u;
constexpr unsigned SRCB = 32 * KTS + 256u;
constexpr size_t PARB = 81ull << 20;
static_assert(PARB >= 8ull * SRCB, "parity region");
constexpr size_t WS_NEED = WS_DATA + 2 * PARB;
constexpr unsigned DEADLINE = 50000000u;

struct Params {
  char* ws[NR];
  const signed char* x; const signed char* w; const float* sx; const float* sw;
  float* out; int rank;
};

__device__ __forceinline__ unsigned perm(unsigned hi, unsigned lo, unsigned sel) { return __builtin_amdgcn_perm(hi, lo, sel); }

__device__ __forceinline__ int start_of(int slot) { return 2 * (slot & 7) + ((slot & 7) >= 4 ? DGAP : 0) + (slot < 7 ? DSIDE : 0); }
__device__ __forceinline__ unsigned slot_mask(int tau) {
  unsigned m = 0;
#pragma unroll
  for (int j = 0; j < 15; ++j) m |= ((unsigned)(tau - start_of(j)) < 128u ? 1u : 0u) << j;
  return m;
}
__device__ __forceinline__ int step_units(int tau, unsigned mask) { return (tau < 32 ? 16 : 0) + __builtin_popcount(mask) + ((unsigned)(tau - LT0) < 32u ? 4 : 0); }
constexpr int TEND = (12 + DSIDE > 14 ? 12 + DSIDE : 14) + DGAP + 128;
static_assert(LT0 >= 32 && LT0 + 32 <= TEND && TD <= 4 && DGAP % 2 == 0 && DSIDE % 2 == 0 && TEND < 160, "schedule");
}

__global__ __launch_bounds__(512, 2) void fused_kernel(const Params p) {
  __shared__ __attribute__((aligned(16))) unsigned char shm[LDS_BYTES];
  LAS unsigned char* lds = (LAS unsigned char*)shm;
  const int tid = threadIdx.x, wid = __builtin_amdgcn_readfirstlane(tid >> 6), lane = tid & 63;
  const int rk = p.rank & 7;
  char* const myws = p.ws[rk];
#define STAMP0() do {} while (0)
#define STAMP() do {} while (0)
#define STAMP_END() do {} while (0)
  unsigned call;
  size_t par;

  {
    const int b = ((int)blockIdx.x + GRID - NAG) & (GRID - 1);
    const bool hasx = wid >= 6;
    const int ex = ((wid & 1) << 8) + b, xkt = ex >> 4, xb = ex & 15;
    u32x4 vx[4] = {};
    if (hasx) {
      const signed char* s = p.x + (size_t)xb * 256 * KS + xkt * 16;
#pragma unroll
      for (int c = 0; c < 4; ++c) vx[c] = *(const GAS u32x4*)(s + (unsigned)((lane + c * 64) * KS));
    }
    const int ew = (wid << 8) + b, wkt = ew >> 6, col = ew & 63;
    unsigned q[16];
    { const signed char* s = p.w + (size_t)wkt * 16 * N_ + col * 256;
#pragma unroll
      for (int j = 0; j < 16; ++j) q[j] = *(const GAS unsigned*)(s + (unsigned)(lane * 4 + j * N_)); }
    asm volatile("" ::: "memory");
    unsigned tkt = 0;
    if (lane == 0) tkt = __hip_atomic_fetch_add((unsigned*)(myws + WS_CTR + (size_t)(blockIdx.x * 8 + wid) * 64), 1u, __ATOMIC_RELAXED, __HIP_MEMORY_SCOPE_AGENT);
    call = (unsigned)__builtin_amdgcn_readfirstlane((int)tkt) + 1u;
    STAMP0();
    par = WS_DATA + (size_t)(call & 1u) * PARB;
    u32x4 vw[4];
#pragma unroll
    for (int t = 0; t < 4; ++t) {
      const unsigned t0 = perm(q[4 * t + 1], q[4 * t], 0x05010400u), t1 = perm(q[4 * t + 1], q[4 * t], 0x07030602u);
      const unsigned t2 = perm(q[4 * t + 3], q[4 * t + 2], 0x05010400u), t3 = perm(q[4 * t + 3], q[4 * t + 2], 0x07030602u);
      vw[0][t] = perm(t2, t0, 0x05040100u); vw[1][t] = perm(t2, t0, 0x07060302u);
      vw[2][t] = perm(t3, t1, 0x05040100u); vw[3][t] = perm(t3, t1, 0x07060302u);
    }
    const unsigned uw = (unsigned)(wkt * KSTR + 16 + col), ux = (unsigned)(xkt * KSTR + xb);
    const unsigned lo = (unsigned)rk * SRCB + (unsigned)lane * 16u;
#define PUT(d, o, v) asm volatile("global_store_dwordx4 %0, %1, %5 sc0 sc1\n\tglobal_store_dwordx4 %0, %2, %5 offset:1024 sc0 sc1\n\t" \
      "global_store_dwordx4 %0, %3, %5 offset:2048 sc0 sc1\n\tglobal_store_dwordx4 %0, %4, %5 offset:3072 sc0 sc1\n\ts_nop 1" \
      :: "v"(o), "v"(v[0]), "v"(v[1]), "v"(v[2]), "v"(v[3]), "s"(d) : "memory")
    PUT(myws + par, lo + uw * 4096u, vw);
    if (hasx) PUT(myws + par, lo + ux * 4096u, vx);
    const bool dw = wkt < TD && col < 15 && start_of(col) <= wkt, dx = hasx && xkt < TD;
    asm volatile("s_waitcnt vmcnt(0)" ::: "memory");
    STAMP0();
    __builtin_amdgcn_s_barrier();
    STAMP0();
    if (tid == 64) __hip_atomic_store((unsigned*)(myws + WS_LFLAG) + blockIdx.x, call, __ATOMIC_RELAXED, __HIP_MEMORY_SCOPE_AGENT);
    if (dw || dx) {
      if (dx) { vw[0] = vx[0]; vw[1] = vx[1]; vw[2] = vx[2]; vw[3] = vx[3]; }
      const unsigned o = lo + (dx ? ux : uw) * 4096u;
      __builtin_amdgcn_s_sleep(1);
#pragma unroll
      for (int i = 1; i < NR; ++i) PUT(p.ws[(rk + 1 + (i + b) % (NR - 1)) & 7] + par, o, vw);
      asm volatile("s_waitcnt vmcnt(0)" ::: "memory");
      if (lane < NR) __hip_atomic_store((unsigned*)(p.ws[lane & 7] + WS_FLAG + rk * 4 + (size_t)(dx ? ux : uw) * 32u), call, __ATOMIC_RELAXED, __HIP_MEMORY_SCOPE_SYSTEM);
    }
  }
  auto lane_id = [&]() -> int { int l; asm volatile("v_mbcnt_lo_u32_b32 %0, -1, 0\n\tv_mbcnt_hi_u32_b32 %0, -1, %0" : "=v"(l)); return l; };
  auto wait_words = [&](unsigned off, int nw, unsigned stride, unsigned long long own) -> bool {
    const unsigned t0 = (unsigned)__builtin_amdgcn_s_memrealtime();
    for (;;) {
      unsigned long long ok = ~0ull;
      for (int i = 0; i < nw; ++i) { const unsigned v = __hip_atomic_load((const unsigned*)(myws + (off + i * stride)), __ATOMIC_RELAXED, __HIP_MEMORY_SCOPE_SYSTEM);
        ok &= __ballot((int)(v - call) >= 0); }
      if ((ok | own) == ~0ull) return true;
      unsigned to; asm volatile("v_mov_b32 %0, %1" : "=v"(to) : "i"((unsigned)WS_TMO));
      unsigned* const tp = (unsigned*)(myws + to);
      const unsigned now = (unsigned)__builtin_amdgcn_s_memrealtime();
      if (__builtin_amdgcn_readfirstlane((int)__hip_atomic_load(tp, __ATOMIC_RELAXED, __HIP_MEMORY_SCOPE_AGENT)) == (int)call || now - t0 > DEADLINE) {
        unsigned cv; asm volatile("v_mov_b32 %0, %1" : "=v"(cv) : "s"(call));
        __hip_atomic_store(tp, cv, __ATOMIC_RELAXED, __HIP_MEMORY_SCOPE_AGENT);
        return false;
      }
      __builtin_amdgcn_s_sleep(SLP);
    }
  };

  if (blockIdx.x < NAG) {
    if (wid != 0) return;
    const int lane = lane_id();
    char* const fl_base = p.ws[lane & 7] + WS_FLAG + rk * 4;
    int ndirect = 0;
    for (int t = 0; t < TD; ++t) ndirect += step_units(t, slot_mask(t));
    const unsigned npush = (unsigned)(NUNITS - ndirect);
    const unsigned qbase = (call - 1u) * (npush + (unsigned)NAG);
    const char* const qctr = myws + WS_QUEUE;
    const char* const src = myws + par + (size_t)rk * SRCB;
    char* dst[NR - 1];
#pragma unroll
    for (int q = 0; q < NR - 1; ++q) dst[q] = p.ws[(rk + 1 + (q + (int)blockIdx.x) % (NR - 1)) & 7] + par + (size_t)rk * SRCB;
    const unsigned l16 = (unsigned)lane * 16u, vzero = 0u, vone = 1u;
    int tau = TD, base = 0;
    unsigned mask = slot_mask(TD); int cnt = step_units(TD, mask);
    auto decode = [&](int idx) -> unsigned {
      while (idx >= base + cnt && tau < TEND - 1) { base += cnt; ++tau; mask = slot_mask(tau); cnt = step_units(tau, mask); }
      int r = idx - base; const int kt = tau & 31;
      if (tau < 32) { if (r < 16) return (unsigned)(kt * KSTR + r); r -= 16; }
      const int nreg = __builtin_popcount(mask);
      unsigned u;
      if (r >= nreg) u = (unsigned)((tau - LT0) * KSTR + 76 + (r - nreg));
      else { unsigned m = mask; for (int i = 0; i < r; ++i) m &= m - 1u;
        const int slot = __builtin_ctz(m | 0x4000u);
        u = (unsigned)(kt * KSTR + 16 + 15 * ((tau - start_of(slot)) >> 5) + slot); }
      return u < (unsigned)NUNITS ? u : 0u;
    };
#define CLB_A "v224", "v225", "v226", "v227", "v228", "v229", "v230", "v231", "v232", "v233", "v234", "v235", "v236", "v237", "v238", "v239"
#define CLB_B "v240", "v241", "v242", "v243", "v244", "v245", "v246", "v247", "v248", "v249", "v250", "v251", "v252", "v253", "v254", "v255"
#define CLB "memory", "v223", CLB_A, CLB_B
#define DRAW() asm volatile("s_mov_b64 exec, 1\n\tglobal_atomic_add v223, %0, %1, %2 sc0\n\ts_mov_b64 exec, -1" :: "v"(vzero), "v"(vone), "s"(qctr) : CLB)
#define LOADU(R, u) do { const unsigned _o = (u) * 4096u + l16; \
      asm volatile("global_load_dwordx4 v[%c2:%c2+3], %0, %1 sc0 sc1\n\tglobal_load_dwordx4 v[%c2+4:%c2+7], %0, %1 offset:1024 sc0 sc1\n\t" \
                   "global_load_dwordx4 v[%c2+8:%c2+11], %0, %1 offset:2048 sc0 sc1\n\tglobal_load_dwordx4 v[%c2+12:%c2+15], %0, %1 offset:3072 sc0 sc1" \
                   :: "v"(_o), "s"(src), "i"(R) : CLB); } while (0)
#define STOREU(R, u) do { const unsigned _o = (u) * 4096u + l16; _Pragma("unroll") for (int q = 0; q < NR - 1; ++q) \
      asm volatile("global_store_dwordx4 %0, v[%c2:%c2+3], %1 sc0 sc1\n\tglobal_store_dwordx4 %0, v[%c2+4:%c2+7], %1 offset:1024 sc0 sc1\n\t" \
                   "global_store_dwordx4 %0, v[%c2+8:%c2+11], %1 offset:2048 sc0 sc1\n\tglobal_store_dwordx4 %0, v[%c2+12:%c2+15], %1 offset:3072 sc0 sc1\n\ts_nop 1" \
                   :: "v"(_o), "s"(dst[q]), "i"(R) : CLB); } while (0)
#define FLAGU(u) do { char* _f = fl_base + (size_t)(u) * 32u; \
      asm volatile("s_mov_b64 exec, 0xff\n\tglobal_store_dword %0, %1, off sc0 sc1\n\ts_mov_b64 exec, -1" :: "v"(_f), "v"(call) : CLB); } while (0)
#define WAIT_TK(n, tk) asm volatile("s_waitcnt vmcnt(" #n ")\n\tv_readfirstlane_b32 %0, v223" : "=s"(tk) :: CLB)
    unsigned tk;
    DRAW(); WAIT_TK(0, tk);
    unsigned idx = tk - qbase;
    if (!wait_words((unsigned)WS_LFLAG + lane * 4u, 4, 256u, 0ull)) return;
    STAMP();
    if (idx < npush) {
      unsigned cur = decode((int)idx), prev = NUNITS;
      LOADU(224, cur); DRAW();
      asm volatile("s_waitcnt vmcnt(0)" ::: CLB);
#define STEP(RC, RN) { \
        WAIT_TK(29, tk); \
        idx = tk - qbase; \
        if (idx >= npush) { STOREU(RC, cur); asm volatile("s_waitcnt vmcnt(28)" ::: CLB); FLAGU(prev); break; } \
        const unsigned nxt = decode((int)idx); \
        LOADU(RN, nxt); DRAW(); \
        STOREU(RC, cur); \
        asm volatile("s_waitcnt vmcnt(28)" ::: CLB);             \
        FLAGU(prev); STAMP(); \
        prev = cur; cur = nxt; }
      for (;;) { STEP(224, 240) STEP(240, 224) }
      asm volatile("s_waitcnt vmcnt(0)" ::: CLB);
      FLAGU(cur);
    }
    STAMP_END();
    asm volatile("s_waitcnt vmcnt(0)" ::: "memory");
    return;
  }

  const int wr = wid >> 2, wc = wid & 3, fr = lane & 15, fq = lane >> 4;
  int pm, slot;
  { const int b = blockIdx.x - NAG, x = b & 7, r = b >> 3, a = x >> 1, hb = x & 1;
    if (r < 28) { pm = 4 * a + (r & 3); slot = hb * 8 + (r >> 2); } else { pm = 4 * a + 2 * hb + (r - 28); slot = 7; } }
  const bool has_half = (slot & 7) < 4;
  const int hcol = (slot >> 3) * 2 + ((slot >> 1) & 1), hrow = slot & 1;
  const float scale = __builtin_bit_cast(float, __builtin_amdgcn_readfirstlane(__builtin_bit_cast(int, p.sx[0] * p.sw[0])));
  const unsigned voff = (unsigned)((wid >> 2) * 4 + fq) * SRCB + (unsigned)((wid & 3) * 256 + fr * 16);
  const unsigned lds0 = (unsigned)(size_t)lds, ldsw = lds0 + LDS_OP + (unsigned)wid * 2048u;
  int aoff = LDS_OP + wr * 1024 + lane * 16, boff = LDS_OP + 4 * HTB + wc * 2048 + lane * 16;
  asm volatile("" : "+v"(aoff), "+v"(boff));
  const unsigned coff = (unsigned)((wr * 64 + 4 * fq) * (N_ * 4) + (wc * 64 + 4 * fr) * 4);
  const u32x4 srd = {(unsigned)(size_t)myws, (unsigned)((size_t)myws >> 32) & 0xffffu, (unsigned)WS_NEED, 0x00020000u};
  const unsigned cA = (unsigned)par + (unsigned)pm * 4096u;
  const unsigned cW = (unsigned)par + (unsigned)(16 + slot) * 4096u;
  const unsigned hA = cA + hrow * GH, hB = (unsigned)par + (unsigned)(76 + hcol) * 4096u;
  const unsigned long long own = 0x0101010101010101ull << rk;
#define SA(b, h) (((b) * 2 + (h)) * HTB)
#define SB(b, h) ((4 + (b) * 2 + (h)) * HTB)
#define STAGE(bufoff, goff) asm volatile("s_add_u32 m0, %3, %4\n\ts_nop 0\n\tbuffer_load_dwordx4 %0, %1, %2 offen sc0 sc1 lds\n\t" \
    "buffer_load_dwordx4 %0, %1, %2 offen offset:1024 sc0 sc1 lds" :: "v"(voff), "s"(srd), "s"(goff), "s"(ldsw), "i"(bufoff) : "scc", "memory")
#define FLAG_FETCH(ko, sl) asm volatile("s_mov_b32 m0, %3\n\ts_nop 0\n\tbuffer_load_dword %0, %1, %2 offen sc0 sc1 lds" :: "v"(vf), "s"(srd), "s"((ko) >> 7), "s"(lds0 + (sl)) : "memory")
#define LDA_(dst, base) do { _Pragma("unroll") for (int m = 0; m < 4; ++m) _Pragma("unroll") for (int k = 0; k < 2; ++k) dst[m][k] = *(const LAS i32x4*)(lds + (base) + m * 2048 + k * 8192); } while (0)
#define LDB_(dst, base) do { _Pragma("unroll") for (int n = 0; n < 2; ++n) _Pragma("unroll") for (int k = 0; k < 2; ++k) dst[n][k] = *(const LAS i32x4*)(lds + (base) + n * 1024 + k * 8192); } while (0)
#define LDA(dst, b, h) LDA_(dst, SA(b, h) + aoff)
#define LDB(dst, b, h) LDB_(dst, SA(b, h) + boff)
#define MFMA2(c, a0, b0, a1, b1, Z) do { \
    if (Z) asm volatile("v_mfma_i32_16x16x64_i8 %0, %1, %2, 0.15915494\n\tv_mfma_i32_16x16x64_i8 %0, %3, %4, %0" : "+v"(c) : "v"(a0), "v"(b0), "v"(a1), "v"(b1)); \
    else asm volatile("v_mfma_i32_16x16x64_i8 %0, %1, %2, %0\n\tv_mfma_i32_16x16x64_i8 %0, %3, %4, %0" : "+v"(c) : "v"(a0), "v"(b0), "v"(a1), "v"(b1)); } while (0)
#define MMA2(ai, Z, PARK) do { __builtin_amdgcn_s_setprio(1); \
    _Pragma("unroll") for (int i = 0; i < 16; ++i) { \
      const int k = i & 1, m = i >> 2, c = (m & 1) ? 3 - (i & 3) : (i & 3); \
      if ((PARK) == 1 && m >= 2) asm volatile("v_mfma_i32_16x16x64_i8 %0, %1, %2, 0.15915494\n\tv_mfma_i32_16x16x64_i8 %0, %3, %4, %0" \
          : "=&v"(park[m & 1][c]) : "v"(At[m][k]), "v"(c < 2 ? B0[c & 1][k] : B1[c & 1][k]), "v"(At[m][k ^ 1]), "v"(c < 2 ? B0[c & 1][k ^ 1] : B1[c & 1][k ^ 1])); \
      else if ((PARK) == 2 && m >= 2) asm volatile("v_mfma_i32_16x16x64_i8 %0, %1, %2, %5\n\tv_mfma_i32_16x16x64_i8 %0, %3, %4, %0" \
          : "+v"(acc[ai][c >> 1][m][c & 1]) : "v"(At[m][k]), "v"(c < 2 ? B0[c & 1][k] : B1[c & 1][k]), "v"(At[m][k ^ 1]), "v"(c < 2 ? B0[c & 1][k ^ 1] : B1[c & 1][k ^ 1]), "v"(park[m & 1][c])); \
      else MFMA2(acc[ai][c >> 1][m][c & 1], At[m][k], (c < 2 ? B0[c & 1][k] : B1[c & 1][k]), At[m][k ^ 1], (c < 2 ? B0[c & 1][k ^ 1] : B1[c & 1][k ^ 1]), Z); } \
    __builtin_amdgcn_s_setprio(0); } while (0)
#define WAIT_V(n) asm volatile("s_waitcnt vmcnt(" #n ")" ::: "memory")
#define WAIT_L(n) do { asm volatile("" ::: "memory"); __builtin_amdgcn_s_waitcnt(0xC07F); asm volatile("" ::: "memory"); } while (0)
#define BAR __builtin_amdgcn_s_barrier()
#define SCHED __builtin_amdgcn_sched_barrier(0)
#define PIN do { asm volatile("" ::: "memory"); SCHED; } while (0)
#define EPI_ST(s0, s1) do { _Pragma("unroll") for (int s = (s0); s < (s1); ++s) { const int ai = s >> 4, m = (s >> 2) & 3, j = s & 3; f32x4 v; \
      v[0] = fmaxf((float)(acc[ai][0][m][0][j] - BIAS) * scale, 0.f); v[1] = fmaxf((float)(acc[ai][0][m][1][j] - BIAS) * scale, 0.f); \
      v[2] = fmaxf((float)(acc[ai][1][m][0][j] - BIAS) * scale, 0.f); v[3] = fmaxf((float)(acc[ai][1][m][1][j] - BIAS) * scale, 0.f); \
      __builtin_amdgcn_raw_buffer_store_b128(__builtin_bit_cast(u32x4, v), crs, coff, (ai * 128 + m * 16 + j) * (N_ * 4), 2); \
      asm volatile("s_nop 1" ::: "memory"); } } while (0)
  auto flag_off = [&](int rnd) -> unsigned { const int l = lane_id();
    return (unsigned)WS_FLAG + (unsigned)(((l & 16) ? 16 + 15 * rnd + slot : pm) * 32 + ((l & 8) ? (int)(KTS >> 7) : 0) + (l & 7) * 4); };
  auto nextk = [&](unsigned k) -> unsigned { k += 2 * KTS; return k < 32 * KTS ? k : 0u; };

  i32x4 acc[2][2][4][2];
  i32x4 At[4][2], B0[2][2], B1[2][2], park[2][4];
  unsigned k0 = (unsigned)start_of(slot) * KTS, k1 = nextk(k0);
  unsigned vf = flag_off(0);
  if (wid == 0) {
    wait_words(vf + (k0 >> 7), 1, 0u, own);
    STAMP0();
    wait_words((unsigned)WS_LFLAG + lane_id() * 4u, 4, 256u, 0ull);
    STAMP0();
    FLAG_FETCH(k1, 0u);
  }
  asm volatile("s_waitcnt vmcnt(0)" ::: "memory");
  BAR;
  { const unsigned a = cA + k0, b = cW + k0;
    STAGE(SB(0, 0), b); STAGE(SB(0, 1), b + GH); STAGE(SA(0, 0), a); STAGE(SA(0, 1), a + GH);
    if (wr == 1) BAR;
    WAIT_V(2); BAR;
    STAGE(SB(1, 0), b + KTS); STAGE(SA(1, 0), a + KTS); STAGE(SB(1, 1), b + GH + KTS);
    WAIT_V(0); BAR; }
#pragma unroll
  for (int a = 0; a < 2; ++a)
#pragma unroll
    for (int b = 0; b < 2; ++b)
#pragma unroll
      for (int m = 0; m < 4; ++m)
#pragma unroll
        for (int n = 0; n < 2; ++n) asm volatile("" : "=v"(acc[a][b][m][n]));
  __amdgpu_buffer_rsrc_t crs = __builtin_amdgcn_make_buffer_rsrc((void*)p.out, 0, 0u, 0x00020000);
  unsigned fsl = 0;
#define KT(b, ah, a2, b2, Z, PARK, V1, V2, e0, e1, e2, FL) do { \
    unsigned fv = 0; \
    LDB(B0, b, 0); LDB(B1, b, 1); SCHED; LDA(At, b, 0); STAGE(SA((b) ^ 1, 1), ah); \
    if ((FL) && wid == 0) { fv = *(const LAS unsigned*)(lds + fsl + lane_id() * 4); if (newf) vf = flag_off(un); FLAG_FETCH(k2, fsl ^ 256u); } \
    PIN; EPI_ST(e0, e1); PIN; \
    WAIT_V(V1); WAIT_L(0); \
    if ((FL) && wid == 0) { if ((__ballot((int)(fv - call) >= 0) | own) != ~0ull) wait_words(flag_off(c1) + (k1 >> 7), 1, 0u, own); \
      if ((FL) == 2 && tohalf) { const int l = lane_id(); wait_words((unsigned)WS_FLAG + (unsigned)((76 + hcol) * 32 + (l >> 3) * (int)(KTS >> 7) + (l & 7) * 4), 4, KTS >> 4, own); } \
      STAMP(); } \
    BAR; MMA2(0, Z, 0); BAR; SCHED; \
    LDA(At, b, 1); STAGE(SB(b, 0), b2); STAGE(SB(b, 1), (b2) + GH); STAGE(SA(b, 0), a2); PIN; EPI_ST(e1, e2); PIN; \
    WAIT_V(V2); WAIT_L(0); BAR; MMA2(1, Z, PARK); BAR; SCHED; } while (0)
#define PAIR(C1, NEWF, Z, Va, Vb, Vc, Vd, e0, e1, e2, e3, e4, FL) do { \
    const int c1 = (C1); const bool newf = (NEWF); const unsigned k2 = nextk(k1); \
    unsigned a1 = cA + k1, b1 = cW + (unsigned)c1 * (15u * 4096u) + k1, ah1 = a1 + GH; \
    if ((FL) == 2 && tohalf) { a1 = hA; b1 = hB; ah1 = hB + 2 * KTS; }     \
    KT(0, cA + k0 + KTS + GH, a1, b1, Z, Z, Va, Vb, e0, e1, e2, FL); \
    KT(1, ah1, a1 + KTS, b1 + KTS, 0, 2 * (Z), Vc, Vd, e2, e3, e4, 0); \
    k0 = k1; k1 = k2; fsl ^= 256u; } while (0)
  for (int ui = 0;; ++ui) {
    const bool has_next = ui < 3, tohalf = !has_next && has_half;
    const int un = has_next ? ui + 1 : ui;
    PAIR(ui, false, 1, 24, 32, 28, 24, 8, 16, 24, 28, 32, 1);
    crs = __builtin_amdgcn_make_buffer_rsrc((void*)(p.out + (size_t)pm * 256 * N_ + (ui * 15 + slot) * 256), 0, 256u * N_ * 4u, 0x00020000);
    for (int t = 1; t < 15; ++t) PAIR(ui, t == 14, 0, 8, 8, 8, 8, 0, 0, 0, 0, 0, 1);
    PAIR(un, false, 0, 8, 8, 8, 16, 0, 0, 0, 0, 8, 2);
    if (!has_next) break;
  }
  if (has_half) {
    int aoff2 = 8 * HTB + aoff; asm volatile("" : "+v"(aoff2));
    const int boff2 = boff - 4 * HTB;
#define HKT(j, tn, Z, V, e0, e1) do { \
      if ((j) < 2) { LDB(B0, j, 0); LDB(B1, j, 1); SCHED; LDA(At, j, 0); } \
      else { LDB_(B0, SA(0, 1) + boff2); LDB_(B1, SA(1, 1) + boff2); SCHED; LDA_(At, aoff2); } \
      { const unsigned a = hA + (unsigned)(tn) * KTS, b = hB + (unsigned)(tn) * KTS; \
        if ((j) == 0) { STAGE(SA(0, 1), b); STAGE(SA(1, 1), b + GH); STAGE(8 * HTB, a); } \
        else { STAGE(SB(((j) + 1) & 1, 0), b); STAGE(SB(((j) + 1) & 1, 1), b + GH); STAGE(SA(((j) + 1) & 1, 0), a); } } \
      PIN; EPI_ST(e0, e1); PIN; \
      WAIT_V(V); WAIT_L(0); BAR; MMA2(0, Z, 0); BAR; SCHED; } while (0)
    HKT(0, 2, 1, 22, 8, 16); HKT(1, 3, 0, 18, 16, 20); HKT(2, 4, 0, 14, 20, 24);
    HKT(0, 5, 0, 14, 24, 28); HKT(1, 6, 0, 14, 28, 32); HKT(2, 7, 0, 10, 0, 0);
    crs = __builtin_amdgcn_make_buffer_rsrc((void*)(p.out + (size_t)(pm * 256 + hrow * 128) * N_ + (60 + hcol) * 256), 0, 128u * N_ * 4u, 0x00020000);
    for (int t = 6; t < 30; t += 3) { HKT(0, t + 2, 0, 6, 0, 0); HKT(1, t + 3, 0, 6, 0, 0); HKT(2, t + 4, 0, 6, 0, 0); if (wid == 0) STAMP(); }
    HKT(0, 31, 0, 6, 0, 0); HKT(1, 31, 0, 6, 0, 0);
    asm volatile("s_nop 7" ::: "memory"); SCHED;
    EPI_ST(0, 16); PIN;
  } else {
    asm volatile("s_nop 7" ::: "memory"); SCHED;
    EPI_ST(8, 32); PIN;
  }
  if (wid == 0) STAMP();
  WAIT_V(0);
  if (wr == 0) BAR;
  BAR;
  if (wid == 0) { STAMP(); STAMP_END(); }
  if (__builtin_amdgcn_readfirstlane((int)__hip_atomic_load((unsigned*)(myws + WS_TMO), __ATOMIC_RELAXED, __HIP_MEMORY_SCOPE_AGENT)) == (int)call) {
    const int tid = wid * 64 + lane_id();
    const float q = __builtin_nanf("");
    for (int ui = 0; ui < 4; ++ui) {
      float* o = p.out + (size_t)pm * 256 * N_ + (ui * 15 + slot) * 256;
      for (int i = tid; i < 256 * 256; i += 512) o[(size_t)(i >> 8) * N_ + (i & 255)] = q;
    }
    if (has_half) {
      float* o = p.out + (size_t)(pm * 256 + hrow * 128) * N_ + (60 + hcol) * 256;
      for (int i = tid; i < 128 * 256; i += 512) o[(size_t)(i >> 8) * N_ + (i & 255)] = q;
    }
  }
}

extern "C" void kernel_launch(int rank, int nranks, ncclComm_t comm, hipStream_t stream,
                              void* const* d_in, const int* in_sizes, int n_in,
                              void* d_out, int out_size, const int* in_dtypes, int out_dtype,
                              void* const* d_out_all, void* const* d_ws_all, size_t ws_bytes) {
  (void)comm; (void)in_dtypes; (void)out_dtype; (void)d_out_all;
  if (nranks != NR || (unsigned)rank >= (unsigned)NR || n_in != 4 || ws_bytes < WS_NEED || out_size != M_ * N_ ||
      in_sizes[0] != M_ * KS || in_sizes[1] != KS * N_) {
    fprintf(stderr, "kernel_launch: unexpected configuration\n");
    return;
  }
  Params p;
  for (int i = 0; i < NR; ++i) p.ws[i] = (char*)d_ws_all[i];
  p.x = (const signed char*)d_in[0]; p.w = (const signed char*)d_in[1];
  p.sx = (const float*)d_in[2]; p.sw = (const float*)d_in[3];
  p.out = (float*)d_out; p.rank = rank;
  hipLaunchKernelGGL(fused_kernel, dim3(GRID), dim3(512), 0, stream, p);
}
```

```cpp
#include <hip/hip_runtime.h>
#include <rccl/rccl.h>
#include <cstdio>

#define LAS __attribute__((address_space(3)))
#define GAS __attribute__((address_space(1)))
typedef int i32x4 __attribute__((ext_vector_type(4)));
typedef unsigned u32x4 __attribute__((ext_vector_type(4)));
typedef float f32x4 __attribute__((ext_vector_type(4)));

namespace {
constexpr int M_ = 4096, N_ = 16384, K_ = 4096, NR = 8;
constexpr int KS = K_ / NR;
constexpr int HTB = 16384;
constexpr int GH = 2048;
constexpr int LDS_OP = 512;
constexpr int LDS_BYTES = LDS_OP + 9 * HTB;
constexpr int BIAS = 0x3e22f983;
constexpr int NAG = 16;
constexpr int GRID = 256;
#define OAUX 2
#define EPRIO 0
#define MPRIO 1
#define SLP 4
#define TD 2
#define DGAP 8
#define DSIDE 4
#define LT0 96
constexpr size_t WS_CTR = 0;
constexpr size_t WS_TMO = 128 * 1024;
constexpr size_t WS_QUEUE = 128 * 1024 + 256;
constexpr size_t WS_LFLAG = 128 * 1024 + 1024;
constexpr size_t WS_FLAG = 256 * 1024;
constexpr size_t WS_DATA = 1024 * 1024;
constexpr int KSTR = 80;
constexpr int NUNITS = 32 * KSTR;
constexpr unsigned KTS = KSTR * 4096u;
constexpr unsigned SRCB = 32 * KTS + 256u;
constexpr size_t PARB = 81ull << 20;
static_assert(PARB >= 8ull * SRCB, "parity region");
constexpr size_t WS_NEED = WS_DATA + 2 * PARB;
constexpr unsigned DEADLINE = 50000000u;

struct Params {
  char* ws[NR];
  const signed char* x; const signed char* w; const float* sx; const float* sw;
  float* out; int rank;
};

__device__ __forceinline__ unsigned perm(unsigned hi, unsigned lo, unsigned sel) { return __builtin_amdgcn_perm(hi, lo, sel); }

__device__ __forceinline__ int start_of(int slot) { return 2 * (slot & 7) + ((slot & 7) >= 4 ? DGAP : 0) + (slot < 7 ? DSIDE : 0); }
__device__ __forceinline__ unsigned slot_mask(int tau) {
  unsigned m = 0;
#pragma unroll
  for (int j = 0; j < 15; ++j) m |= ((unsigned)(tau - start_of(j)) < 128u ? 1u : 0u) << j;
  return m;
}
__device__ __forceinline__ int step_units(int tau, unsigned mask) { return (tau < 32 ? 16 : 0) + __builtin_popcount(mask) + ((unsigned)(tau - LT0) < 32u ? 4 : 0); }
constexpr int TEND = (12 + DSIDE > 14 ? 12 + DSIDE : 14) + DGAP + 128;
static_assert(LT0 >= 32 && LT0 + 32 <= TEND && TD <= 4 && DGAP % 2 == 0 && DSIDE % 2 == 0 && TEND < 160, "schedule");
}

__global__ __launch_bounds__(512, 2) void fused_kernel(const Params p) {
  __shared__ __attribute__((aligned(16))) unsigned char shm[LDS_BYTES];
  LAS unsigned char* lds = (LAS unsigned char*)shm;
  const int tid = threadIdx.x, wid = __builtin_amdgcn_readfirstlane(tid >> 6), lane = tid & 63;
  const int rk = p.rank & 7;
  char* const myws = p.ws[rk];
#define STAMP0() do {} while (0)
#define STAMP() do {} while (0)
#define STAMP_END() do {} while (0)
  unsigned call, touched = 0;
  size_t par;

  {
    const int b = ((int)blockIdx.x + GRID - NAG) & (GRID - 1);
    const bool hasx = wid >= 6;
    const int ex = ((wid & 1) << 8) + b, xkt = ex >> 4, xb = ex & 15;
    u32x4 vx[4] = {};
    if (hasx) {
      const signed char* s = p.x + (size_t)xb * 256 * KS + xkt * 16;
#pragma unroll
      for (int c = 0; c < 4; ++c) vx[c] = *(const GAS u32x4*)(s + (unsigned)((lane + c * 64) * KS));
    }
    const int ew = (wid << 8) + b, wkt = ew >> 6, col = ew & 63;
    unsigned q[16];
    { const signed char* s = p.w + (size_t)wkt * 16 * N_ + col * 256;
#pragma unroll
      for (int j = 0; j < 16; ++j) q[j] = *(const GAS unsigned*)(s + (unsigned)(lane * 4 + j * N_)); }
    asm volatile("" ::: "memory");
    unsigned tkt = 0;
    if (lane == 0) tkt = __hip_atomic_fetch_add((unsigned*)(myws + WS_CTR + (size_t)(blockIdx.x * 8 + wid) * 64), 1u, __ATOMIC_RELAXED, __HIP_MEMORY_SCOPE_AGENT);
    call = (unsigned)__builtin_amdgcn_readfirstlane((int)tkt) + 1u;
    STAMP0();
    par = WS_DATA + (size_t)(call & 1u) * PARB;
    u32x4 vw[4];
#pragma unroll
    for (int t = 0; t < 4; ++t) {
      const unsigned t0 = perm(q[4 * t + 1], q[4 * t], 0x05010400u), t1 = perm(q[4 * t + 1], q[4 * t], 0x07030602u);
      const unsigned t2 = perm(q[4 * t + 3], q[4 * t + 2], 0x05010400u), t3 = perm(q[4 * t + 3], q[4 * t + 2], 0x07030602u);
      vw[0][t] = perm(t2, t0, 0x05040100u); vw[1][t] = perm(t2, t0, 0x07060302u);
      vw[2][t] = perm(t3, t1, 0x05040100u); vw[3][t] = perm(t3, t1, 0x07060302u);
    }
    const unsigned uw = (unsigned)(wkt * KSTR + 16 + col), ux = (unsigned)(xkt * KSTR + xb);
    const unsigned lo = (unsigned)rk * SRCB + (unsigned)lane * 16u;
#define PUT(d, o, v) asm volatile("global_store_dwordx4 %0, %1, %5 sc0 sc1\n\tglobal_store_dwordx4 %0, %2, %5 offset:1024 sc0 sc1\n\t" \
      "global_store_dwordx4 %0, %3, %5 offset:2048 sc0 sc1\n\tglobal_store_dwordx4 %0, %4, %5 offset:3072 sc0 sc1\n\ts_nop 1" \
      :: "v"(o), "v"(v[0]), "v"(v[1]), "v"(v[2]), "v"(v[3]), "s"(d) : "memory")
    PUT(myws + par, lo + uw * 4096u, vw);
    if (hasx) PUT(myws + par, lo + ux * 4096u, vx);
    const bool dw = wkt < TD && col < 15 && start_of(col) <= wkt, dx = hasx && xkt < TD;
    {
      const char* t;
      if (blockIdx.x < NAG) t = p.ws[(rk + wid) & 7] + (lane < 11 ? par + (size_t)rk * SRCB + (size_t)lane * (1u << 20) - (lane == 10 ? 4 : 0) : WS_FLAG);
      else if (wid == 3) t = (const char*)p.out + ((size_t)(4 * ((b & 7) >> 1) + (lane >> 4)) << 24) + ((size_t)(lane & 15) << 20);
      else t = myws + WS_FLAG;
      unsigned dummy; asm volatile("global_load_dword %0, %1, off\n\ts_waitcnt vmcnt(1)" : "=&v"(dummy) : "v"(t) : "memory");
      touched = dummy; }
    STAMP0();
    __builtin_amdgcn_s_barrier();
    STAMP0();
    if (tid == 64) __hip_atomic_store((unsigned*)(myws + WS_LFLAG) + blockIdx.x, call, __ATOMIC_RELAXED, __HIP_MEMORY_SCOPE_AGENT);
    if (dw || dx) {
      if (dx) { vw[0] = vx[0]; vw[1] = vx[1]; vw[2] = vx[2]; vw[3] = vx[3]; }
      const unsigned o = lo + (dx ? ux : uw) * 4096u;
      __builtin_amdgcn_s_sleep(1);
#pragma unroll
      for (int i = 1; i < NR; ++i) PUT(p.ws[(rk + 1 + (i + b) % (NR - 1)) & 7] + par, o, vw);
      asm volatile("s_waitcnt vmcnt(0)" ::: "memory");
      if (lane < NR) __hip_atomic_store((unsigned*)(p.ws[lane & 7] + WS_FLAG + rk * 4 + (size_t)(dx ? ux : uw) * 32u), call, __ATOMIC_RELAXED, __HIP_MEMORY_SCOPE_SYSTEM);
    }
  }
  asm volatile("s_waitcnt vmcnt(0)" : "+v"(touched) :: "memory");
  auto lane_id = [&]() -> int { int l; asm volatile("v_mbcnt_lo_u32_b32 %0, -1, 0\n\tv_mbcnt_hi_u32_b32 %0, -1, %0" : "=v"(l)); return l; };
  auto wait_words = [&](unsigned off, int nw, unsigned stride, unsigned long long own) -> bool {
    const unsigned t0 = (unsigned)__builtin_amdgcn_s_memrealtime();
    for (;;) {
      unsigned long long ok = ~0ull;
      unsigned v[4];
#pragma unroll
      for (int i = 0; i < 4; ++i) if (i < nw) v[i] = __hip_atomic_load((const unsigned*)(myws + (off + i * stride)), __ATOMIC_RELAXED, __HIP_MEMORY_SCOPE_SYSTEM);
#pragma unroll
      for (int i = 0; i < 4; ++i) if (i < nw) ok &= __ballot((int)(v[i] - call) >= 0);
      if ((ok | own) == ~0ull) return true;
      unsigned to; asm volatile("v_mov_b32 %0, %1" : "=v"(to) : "i"((unsigned)WS_TMO));
      unsigned* const tp = (unsigned*)(myws + to);
      const unsigned now = (unsigned)__builtin_amdgcn_s_memrealtime();
      if (__builtin_amdgcn_readfirstlane((int)__hip_atomic_load(tp, __ATOMIC_RELAXED, __HIP_MEMORY_SCOPE_AGENT)) == (int)call || now - t0 > DEADLINE) {
        unsigned cv; asm volatile("v_mov_b32 %0, %1" : "=v"(cv) : "s"(call));
        __hip_atomic_store(tp, cv, __ATOMIC_RELAXED, __HIP_MEMORY_SCOPE_AGENT);
        return false;
      }
      __builtin_amdgcn_s_sleep(SLP);
    }
  };

  if (blockIdx.x < NAG) {
    if (wid != 0) return;
    const int lane = lane_id();
    char* const fl_base = p.ws[lane & 7] + WS_FLAG + rk * 4;
    int ndirect = 0;
    for (int t = 0; t < TD; ++t) ndirect += step_units(t, slot_mask(t));
    const unsigned npush = (unsigned)(NUNITS - ndirect);
    const unsigned qbase = (call - 1u) * (npush + (unsigned)NAG);
    const char* const qctr = myws + WS_QUEUE;
    const char* const src = myws + par + (size_t)rk * SRCB;
    char* dst[NR - 1];
#pragma unroll
    for (int q = 0; q < NR - 1; ++q) dst[q] = p.ws[(rk + 1 + (q + (int)blockIdx.x) % (NR - 1)) & 7] + par + (size_t)rk * SRCB;
    const unsigned l16 = (unsigned)lane * 16u, vzero = 0u, vone = 1u;
    int tau = TD, base = 0;
    unsigned mask = slot_mask(TD); int cnt = step_units(TD, mask);
    auto decode = [&](int idx) -> unsigned {
      while (idx >= base + cnt && tau < TEND - 1) { base += cnt; ++tau; mask = slot_mask(tau); cnt = step_units(tau, mask); }
      int r = idx - base; const int kt = tau & 31;
      if (tau < 32) { if (r < 16) return (unsigned)(kt * KSTR + r); r -= 16; }
      const int nreg = __builtin_popcount(mask);
      unsigned u;
      if (r >= nreg) u = (unsigned)((tau - LT0) * KSTR + 76 + (r - nreg));
      else { unsigned m = mask; for (int i = 0; i < r; ++i) m &= m - 1u;
        const int slot = __builtin_ctz(m | 0x4000u);
        u = (unsigned)(kt * KSTR + 16 + 15 * ((tau - start_of(slot)) >> 5) + slot); }
      return u < (unsigned)NUNITS ? u : 0u;
    };
#define CLB_A "v224", "v225", "v226", "v227", "v228", "v229", "v230", "v231", "v232", "v233", "v234", "v235", "v236", "v237", "v238", "v239"
#define CLB_B "v240", "v241", "v242", "v243", "v244", "v245", "v246", "v247", "v248", "v249", "v250", "v251", "v252", "v253", "v254", "v255"
#define CLB "memory", "v223", CLB_A, CLB_B
#define DRAW() asm volatile("s_mov_b64 exec, 1\n\tglobal_atomic_add v223, %0, %1, %2 sc0\n\ts_mov_b64 exec, -1" :: "v"(vzero), "v"(vone), "s"(qctr) : CLB)
#define LOADU(R, u) do { const unsigned _o = (u) * 4096u + l16; \
      asm volatile("global_load_dwordx4 v[%c2:%c2+3], %0, %1 sc0 sc1\n\tglobal_load_dwordx4 v[%c2+4:%c2+7], %0, %1 offset:1024 sc0 sc1\n\t" \
                   "global_load_dwordx4 v[%c2+8:%c2+11], %0, %1 offset:2048 sc0 sc1\n\tglobal_load_dwordx4 v[%c2+12:%c2+15], %0, %1 offset:3072 sc0 sc1" \
                   :: "v"(_o), "s"(src), "i"(R) : CLB); } while (0)
#define STOREU(R, u) do { const unsigned _o = (u) * 4096u + l16; _Pragma("unroll") for (int q = 0; q < NR - 1; ++q) \
      asm volatile("global_store_dwordx4 %0, v[%c2:%c2+3], %1 sc0 sc1\n\tglobal_store_dwordx4 %0, v[%c2+4:%c2+7], %1 offset:1024 sc0 sc1\n\t" \
                   "global_store_dwordx4 %0, v[%c2+8:%c2+11], %1 offset:2048 sc0 sc1\n\tglobal_store_dwordx4 %0, v[%c2+12:%c2+15], %1 offset:3072 sc0 sc1\n\ts_nop 1" \
                   :: "v"(_o), "s"(dst[q]), "i"(R) : CLB); } while (0)
#define FLAGU(u) do { char* _f = fl_base + (size_t)(u) * 32u; \
      asm volatile("s_mov_b64 exec, 0xff\n\tglobal_store_dword %0, %1, off sc0 sc1\n\ts_mov_b64 exec, -1" :: "v"(_f), "v"(call) : CLB); } while (0)
#define WAIT_TK(n, tk) asm volatile("s_waitcnt vmcnt(" #n ")\n\tv_readfirstlane_b32 %0, v223" : "=s"(tk) :: CLB)
    unsigned tk;
    DRAW(); WAIT_TK(0, tk);
    unsigned idx = tk - qbase;
    if (!wait_words((unsigned)WS_LFLAG + lane * 4u, 4, 256u, 0ull)) return;
    STAMP();
    if (idx < npush) {
      unsigned cur = decode((int)idx), prev = NUNITS;
      LOADU(224, cur); DRAW();
      asm volatile("s_waitcnt vmcnt(0)" ::: CLB);
#define STEP(RC, RN) { \
        WAIT_TK(29, tk); \
        idx = tk - qbase; \
        if (idx >= npush) { STOREU(RC, cur); asm volatile("s_waitcnt vmcnt(28)" ::: CLB); FLAGU(prev); break; } \
        const unsigned nxt = decode((int)idx); \
        LOADU(RN, nxt); DRAW(); \
        STOREU(RC, cur); \
        asm volatile("s_waitcnt vmcnt(28)" ::: CLB);             \
        FLAGU(prev); STAMP(); \
        prev = cur; cur = nxt; }
      for (;;) { STEP(224, 240) STEP(240, 224) }
      asm volatile("s_waitcnt vmcnt(0)" ::: CLB);
      FLAGU(cur);
    }
    STAMP_END();
    asm volatile("s_waitcnt vmcnt(0)" ::: "memory");
    return;
  }

  const int wr = wid >> 2, wc = wid & 3, fr = lane & 15, fq = lane >> 4;
  int pm, slot;
  { const int b = blockIdx.x - NAG, x = b & 7, r = b >> 3, a = x >> 1, hb = x & 1;
    if (r < 28) { pm = 4 * a + (r & 3); slot = hb * 8 + (r >> 2); } else { pm = 4 * a + 2 * hb + (r - 28); slot = 7; } }
  const bool has_half = (slot & 7) < 4;
  const int hcol = (slot >> 3) * 2 + ((slot >> 1) & 1), hrow = slot & 1;
  const float scale = __builtin_bit_cast(float, __builtin_amdgcn_readfirstlane(__builtin_bit_cast(int, p.sx[0] * p.sw[0])));
  const unsigned voff = (unsigned)((wid >> 2) * 4 + fq) * SRCB + (unsigned)((wid & 3) * 256 + fr * 16);
  const unsigned lds0 = (unsigned)(size_t)lds, ldsw = lds0 + LDS_OP + (unsigned)wid * 2048u;
  int aoff = LDS_OP + wr * 1024 + lane * 16, boff = LDS_OP + 4 * HTB + wc * 2048 + lane * 16;
  asm volatile("" : "+v"(aoff), "+v"(boff));
  const unsigned coff = (unsigned)((wr * 64 + 4 * fq) * (N_ * 4) + (wc * 64 + 4 * fr) * 4);
  const u32x4 srd = {(unsigned)(size_t)myws, (unsigned)((size_t)myws >> 32) & 0xffffu, (unsigned)WS_NEED, 0x00020000u};
  const unsigned cA = (unsigned)par + (unsigned)pm * 4096u;
  const unsigned cW = (unsigned)par + (unsigned)(16 + slot) * 4096u;
  const unsigned hA = cA + hrow * GH, hB = (unsigned)par + (unsigned)(76 + hcol) * 4096u;
  const unsigned long long own = 0x0101010101010101ull << rk;
#define SA(b, h) (((b) * 2 + (h)) * HTB)
#define SB(b, h) ((4 + (b) * 2 + (h)) * HTB)
#define STAGE(bufoff, goff) asm volatile("s_add_u32 m0, %3, %4\n\ts_nop 0\n\tbuffer_load_dwordx4 %0, %1, %2 offen sc0 sc1 lds\n\t" \
    "buffer_load_dwordx4 %0, %1, %2 offen offset:1024 sc0 sc1 lds" :: "v"(voff), "s"(srd), "s"(goff), "s"(ldsw), "i"(bufoff) : "scc", "memory")
#define FLAG_FETCH(ko, sl) asm volatile("s_mov_b32 m0, %3\n\ts_nop 0\n\tbuffer_load_dword %0, %1, %2 offen sc0 sc1 lds" :: "v"(vf), "s"(srd), "s"((ko) >> 7), "s"(lds0 + (sl)) : "memory")
#define LDA_(dst, base) do { _Pragma("unroll") for (int m = 0; m < 4; ++m) _Pragma("unroll") for (int k = 0; k < 2; ++k) dst[m][k] = *(const LAS i32x4*)(lds + (base) + m * 2048 + k * 8192); } while (0)
#define LDB_(dst, base) do { _Pragma("unroll") for (int n = 0; n < 2; ++n) _Pragma("unroll") for (int k = 0; k < 2; ++k) dst[n][k] = *(const LAS i32x4*)(lds + (base) + n * 1024 + k * 8192); } while (0)
#define LDA(dst, b, h) LDA_(dst, SA(b, h) + aoff)
#define LDB(dst, b, h) LDB_(dst, SA(b, h) + boff)
#define MFMA2(c, a0, b0, a1, b1, Z) do { \
    if (Z) asm volatile("v_mfma_i32_16x16x64_i8 %0, %1, %2, 0.15915494\n\tv_mfma_i32_16x16x64_i8 %0, %3, %4, %0" : "+v"(c) : "v"(a0), "v"(b0), "v"(a1), "v"(b1)); \
    else asm volatile("v_mfma_i32_16x16x64_i8 %0, %1, %2, %0\n\tv_mfma_i32_16x16x64_i8 %0, %3, %4, %0" : "+v"(c) : "v"(a0), "v"(b0), "v"(a1), "v"(b1)); } while (0)
#define MMA2(ai, Z, PARK) do { __builtin_amdgcn_s_setprio(MPRIO); \
    _Pragma("unroll") for (int i = 0; i < 16; ++i) { \
      const int k = i & 1, m = i >> 2, c = (m & 1) ? 3 - (i & 3) : (i & 3); \
      if ((PARK) == 1 && m >= 2) asm volatile("v_mfma_i32_16x16x64_i8 %0, %1, %2, 0.15915494\n\tv_mfma_i32_16x16x64_i8 %0, %3, %4, %0" \
          : "=&v"(park[m & 1][c]) : "v"(At[m][k]), "v"(c < 2 ? B0[c & 1][k] : B1[c & 1][k]), "v"(At[m][k ^ 1]), "v"(c < 2 ? B0[c & 1][k ^ 1] : B1[c & 1][k ^ 1])); \
      else if ((PARK) == 2 && m >= 2) asm volatile("v_mfma_i32_16x16x64_i8 %0, %1, %2, %5\n\tv_mfma_i32_16x16x64_i8 %0, %3, %4, %0" \
          : "+v"(acc[ai][c >> 1][m][c & 1]) : "v"(At[m][k]), "v"(c < 2 ? B0[c & 1][k] : B1[c & 1][k]), "v"(At[m][k ^ 1]), "v"(c < 2 ? B0[c & 1][k ^ 1] : B1[c & 1][k ^ 1]), "v"(park[m & 1][c])); \
      else MFMA2(acc[ai][c >> 1][m][c & 1], At[m][k], (c < 2 ? B0[c & 1][k] : B1[c & 1][k]), At[m][k ^ 1], (c < 2 ? B0[c & 1][k ^ 1] : B1[c & 1][k ^ 1]), Z); } \
    __builtin_amdgcn_s_setprio(0); } while (0)
#define WAIT_V(n) asm volatile("s_waitcnt vmcnt(" #n ")" ::: "memory")
#define WAIT_L(n) do { asm volatile("" ::: "memory"); __builtin_amdgcn_s_waitcnt(0xC07F); asm volatile("" ::: "memory"); } while (0)
#define BAR __builtin_amdgcn_s_barrier()
#define SCHED __builtin_amdgcn_sched_barrier(0)
#define PIN do { asm volatile("" ::: "memory"); SCHED; } while (0)
#define DEQ(a) fmaxf((float)((a) - BIAS) * scale, 0.f)
#define EPI_ST(s0, s1) do { if ((s1) > (s0) && EPRIO) __builtin_amdgcn_s_setprio(EPRIO); \
    _Pragma("unroll") for (int s = (s0); s < (s1); ++s) { const int ai = s >> 4, m = (s >> 2) & 3, j = s & 3; f32x4 v; \
      v[0] = DEQ(acc[ai][0][m][0][j]); v[1] = DEQ(acc[ai][0][m][1][j]); v[2] = DEQ(acc[ai][1][m][0][j]); v[3] = DEQ(acc[ai][1][m][1][j]); \
      asm volatile("buffer_store_dwordx4 %0, %1, %2, %3 offen nt\n\ts_nop 1" :: "v"(v), "v"(coff), "s"(crs), "s"((ai * 128 + m * 16 + j) * (N_ * 4)) : "memory"); } \
    if ((s1) > (s0) && EPRIO) __builtin_amdgcn_s_setprio(0); } while (0)
  auto flag_off = [&](int rnd) -> unsigned { const int l = lane_id();
    return (unsigned)WS_FLAG + (unsigned)(((l & 16) ? 16 + 15 * rnd + slot : pm) * 32 + ((l & 8) ? (int)(KTS >> 7) : 0) + (l & 7) * 4); };
  auto nextk = [&](unsigned k) -> unsigned { k += 2 * KTS; return k < 32 * KTS ? k : 0u; };

  i32x4 acc[2][2][4][2];
  i32x4 At[4][2], B0[2][2], B1[2][2], park[2][4];
  unsigned k0 = (unsigned)start_of(slot) * KTS, k1 = nextk(k0);
  unsigned vf = flag_off(0);
  if (wid == 0) {
    wait_words(vf + (k0 >> 7), 1, 0u, own);
    STAMP0();
    wait_words((unsigned)WS_LFLAG + lane_id() * 4u, 4, 256u, 0ull);
    STAMP0();
    FLAG_FETCH(k1, 0u);
  }
  asm volatile("s_waitcnt vmcnt(0)" ::: "memory");
  BAR;
  { const unsigned a = cA + k0, b = cW + k0;
    STAGE(SB(0, 0), b); STAGE(SB(0, 1), b + GH); STAGE(SA(0, 0), a); STAGE(SA(0, 1), a + GH);
    if (wr == 1) BAR;
    WAIT_V(2); BAR;
    STAGE(SB(1, 0), b + KTS); STAGE(SA(1, 0), a + KTS); STAGE(SB(1, 1), b + GH + KTS);
    WAIT_V(0); BAR; }
#pragma unroll
  for (int a = 0; a < 2; ++a)
#pragma unroll
    for (int b = 0; b < 2; ++b)
#pragma unroll
      for (int m = 0; m < 4; ++m)
#pragma unroll
        for (int n = 0; n < 2; ++n) asm volatile("" : "=v"(acc[a][b][m][n]));
  auto mk_rs = [](const float* b, unsigned n) { return (u32x4){(unsigned)(size_t)b, (unsigned)((size_t)b >> 32) & 0xffffu, n, 0x00020000u}; };
  u32x4 crs = mk_rs(p.out, 0u);
  unsigned fsl = 0;
#define KT(b, ah, a2, b2, Z, PARK, V1, V2, e0, e1, e2, FL) do { \
    unsigned fv = 0; \
    LDB(B0, b, 0); LDB(B1, b, 1); SCHED; LDA(At, b, 0); STAGE(SA((b) ^ 1, 1), ah); \
    if ((FL) && wid == 0) { fv = *(const LAS unsigned*)(lds + fsl + lane_id() * 4); if (newf) vf = flag_off(un); FLAG_FETCH(k2, fsl ^ 256u); } \
    PIN; EPI_ST(e0, e1); PIN; \
    WAIT_V(V1); WAIT_L(0); \
    if ((FL) && wid == 0) { if ((__ballot((int)(fv - call) >= 0) | own) != ~0ull) wait_words(flag_off(c1) + (k1 >> 7), 1, 0u, own); \
      if ((FL) == 2 && tohalf) { const int l = lane_id(); wait_words((unsigned)WS_FLAG + (unsigned)((76 + hcol) * 32 + (l >> 3) * (int)(KTS >> 7) + (l & 7) * 4), 4, KTS >> 4, own); } \
      STAMP(); } \
    BAR; MMA2(0, Z, 0); BAR; SCHED; \
    LDA(At, b, 1); STAGE(SB(b, 0), b2); STAGE(SB(b, 1), (b2) + GH); STAGE(SA(b, 0), a2); PIN; EPI_ST(e1, e2); PIN; \
    WAIT_V(V2); WAIT_L(0); BAR; MMA2(1, Z, PARK); BAR; SCHED; } while (0)
#define PAIR(C1, NEWF, Z, Va, Vb, Vc, Vd, e0, e1, e2, e3, e4, FL) do { \
    const int c1 = (C1); const bool newf = (NEWF); const unsigned k2 = nextk(k1); \
    unsigned a1 = cA + k1, b1 = cW + (unsigned)c1 * (15u * 4096u) + k1, ah1 = a1 + GH; \
    if ((FL) == 2 && tohalf) { a1 = hA; b1 = hB; ah1 = hB + 2 * KTS; }     \
    KT(0, cA + k0 + KTS + GH, a1, b1, Z, Z, Va, Vb, e0, e1, e2, FL); \
    KT(1, ah1, a1 + KTS, b1 + KTS, 0, 2 * (Z), Vc, Vd, e2, e3, e4, 0); \
    k0 = k1; k1 = k2; fsl ^= 256u; } while (0)
  for (int ui = 0;; ++ui) {
    const bool has_next = ui < 3, tohalf = !has_next && has_half;
    const int un = has_next ? ui + 1 : ui;
    PAIR(ui, false, 1, 24, 32, 28, 24, 8, 16, 24, 28, 32, 1);
    crs = mk_rs(p.out + (size_t)pm * 256 * N_ + (ui * 15 + slot) * 256, 256u * N_ * 4u);
    for (int t = 1; t < 15; ++t) PAIR(ui, t == 14, 0, 8, 8, 8, 8, 0, 0, 0, 0, 0, 1);
    PAIR(un, false, 0, 8, 8, 8, 16, 0, 0, 0, 0, 8, 2);
    if (!has_next) break;
  }
  if (has_half) {
    int aoff2 = 8 * HTB + aoff; asm volatile("" : "+v"(aoff2));
    const int boff2 = boff - 4 * HTB;
#define HV 3
#define HST(j, w, g) do { if ((j) == 0) { if ((w) == 0) STAGE(SA(0, 1), g); else if ((w) == 1) STAGE(SA(1, 1), g); else STAGE(8 * HTB, g); } \
      else { if ((w) == 0) STAGE(SB(((j) + 1) & 1, 0), g); else if ((w) == 1) STAGE(SB(((j) + 1) & 1, 1), g); else STAGE(SA(((j) + 1) & 1, 0), g); } } while (0)
#define HKT(j, tn, Z, V, e0, e1) do { \
      const unsigned a = hA + (unsigned)(tn) * KTS, b = hB + (unsigned)(tn) * KTS; \
      if (HV == 1) { HST(j, 0, b); HST(j, 1, b + GH); HST(j, 2, a); } \
      if (HV == 2) HST(j, 0, b); \
      if ((j) < 2) LDB(B0, j, 0); else LDB_(B0, SA(0, 1) + boff2); \
      if (HV == 3) HST(j, 0, b); \
      if ((j) < 2) LDB(B1, j, 1); else LDB_(B1, SA(1, 1) + boff2); \
      if (HV >= 2) HST(j, 1, b + GH); \
      SCHED; if ((j) < 2) LDA(At, j, 0); else LDA_(At, aoff2); \
      if (HV == 0) { HST(j, 0, b); HST(j, 1, b + GH); } \
      if (HV != 1) HST(j, 2, a); \
      PIN; EPI_ST(e0, e1); PIN; \
      WAIT_V(V); WAIT_L(0); BAR; MMA2(0, Z, 0); BAR; SCHED; } while (0)
    HKT(0, 2, 1, 22, 8, 16); HKT(1, 3, 0, 18, 16, 20); HKT(2, 4, 0, 14, 20, 24);
    HKT(0, 5, 0, 14, 24, 28); HKT(1, 6, 0, 14, 28, 32); HKT(2, 7, 0, 10, 0, 0);
    crs = mk_rs(p.out + (size_t)(pm * 256 + hrow * 128) * N_ + (60 + hcol) * 256, 128u * N_ * 4u);
    for (int t = 6; t < 30; t += 3) { HKT(0, t + 2, 0, 6, 0, 0); HKT(1, t + 3, 0, 6, 0, 0); HKT(2, t + 4, 0, 6, 0, 0); if (wid == 0) STAMP(); }
    HKT(0, 31, 0, 6, 0, 0); HKT(1, 31, 0, 6, 0, 0);
    asm volatile("s_nop 7" ::: "memory"); SCHED;
    EPI_ST(0, 16); PIN;
  } else {
    asm volatile("s_nop 7" ::: "memory"); SCHED;
    EPI_ST(8, 32); PIN;
  }
  if (wid == 0) STAMP();
  WAIT_V(0);
  if (wr == 0) BAR;
  BAR;
  if (wid == 0) { STAMP(); STAMP_END(); }
  if (__builtin_amdgcn_readfirstlane((int)__hip_atomic_load((unsigned*)(myws + WS_TMO), __ATOMIC_RELAXED, __HIP_MEMORY_SCOPE_AGENT)) == (int)call) {
    const int tid = wid * 64 + lane_id();
    const float q = __builtin_nanf("");
    for (int ui = 0; ui < 4; ++ui) {
      float* o = p.out + (size_t)pm * 256 * N_ + (ui * 15 + slot) * 256;
      for (int i = tid; i < 256 * 256; i += 512) o[(size_t)(i >> 8) * N_ + (i & 255)] = q;
    }
    if (has_half) {
      float* o = p.out + (size_t)(pm * 256 + hrow * 128) * N_ + (60 + hcol) * 256;
      for (int i = tid; i < 128 * 256; i += 512) o[(size_t)(i >> 8) * N_ + (i & 255)] = q;
    }
  }
}

extern "C" void kernel_launch(int rank, int nranks, ncclComm_t comm, hipStream_t stream,
                              void* const* d_in, const int* in_sizes, int n_in,
                              void* d_out, int out_size, const int* in_dtypes, int out_dtype,
                              void* const* d_out_all, void* const* d_ws_all, size_t ws_bytes) {
  (void)comm; (void)in_dtypes; (void)out_dtype; (void)d_out_all;
  if (nranks != NR || (unsigned)rank >= (unsigned)NR || n_in != 4 || ws_bytes < WS_NEED || out_size != M_ * N_ ||
      in_sizes[0] != M_ * KS || in_sizes[1] != KS * N_) {
    fprintf(stderr, "kernel_launch: unexpected configuration\n");
    return;
  }
  Params p;
  for (int i = 0; i < NR; ++i) p.ws[i] = (char*)d_ws_all[i];
  p.x = (const signed char*)d_in[0]; p.w = (const signed char*)d_in[1];
  p.sx = (const float*)d_in[2]; p.sw = (const float*)d_in[3];
  p.out = (float*)d_out; p.rank = rank;
  hipLaunchKernelGGL(fused_kernel, dim3(GRID), dim3(512), 0, stream, p);
}
```

```cpp
#include <hip/hip_runtime.h>
#include <rccl/rccl.h>
#include <cstdio>

#define LAS __attribute__((address_space(3)))
#define GAS __attribute__((address_space(1)))
typedef int i32x4 __attribute__((ext_vector_type(4)));
typedef unsigned u32x4 __attribute__((ext_vector_type(4)));
typedef float f32x4 __attribute__((ext_vector_type(4)));

namespace {
constexpr int M_ = 4096, N_ = 16384, K_ = 4096, NR = 8;
constexpr int KS = K_ / NR;
constexpr int HTB = 16384;
constexpr int GH = 2048;
constexpr int LDS_OP = 512;
constexpr int LDS_BYTES = LDS_OP + 9 * HTB;
constexpr int BIAS = 0x3e22f983;
constexpr int NAG = 16;
constexpr int GRID = 256;
constexpr int TD = 2;
constexpr int DGAP = 14;
constexpr int DSIDE = 4;
constexpr int LT0 = 96;
constexpr size_t WS_CTR = 0;
constexpr size_t WS_TMO = 128 * 1024;
constexpr size_t WS_QUEUE = 128 * 1024 + 256;
constexpr size_t WS_LFLAG = 128 * 1024 + 1024;
constexpr size_t WS_FLAG = 256 * 1024;
constexpr size_t WS_DATA = 1024 * 1024;
constexpr int KSTR = 80;
constexpr int NUNITS = 32 * KSTR;
constexpr unsigned KTS = KSTR * 4096u;
constexpr unsigned SRCB = 32 * KTS + 256u;
constexpr size_t PARB = 81ull << 20;
static_assert(PARB >= 8ull * SRCB, "parity region");
constexpr size_t WS_NEED = WS_DATA + 2 * PARB;
constexpr unsigned DEADLINE = 50000000u;

struct Params {
  char* ws[NR];
  const signed char* x; const signed char* w; const float* sx; const float* sw;
  float* out; int rank;
};

__device__ __forceinline__ unsigned perm(unsigned hi, unsigned lo, unsigned sel) { return __builtin_amdgcn_perm(hi, lo, sel); }

__device__ __forceinline__ int start_of(int slot) { return (slot & 2) + ((slot & 4) ? DGAP : 0) + (slot < 7 ? DSIDE : 0); }
__device__ __forceinline__ unsigned slot_mask(int tau) {
  unsigned m = 0;
#pragma unroll
  for (int j = 0; j < 15; ++j) m |= ((unsigned)(tau - start_of(j)) < 128u ? 1u : 0u) << j;
  return m;
}
__device__ __forceinline__ int step_units(int tau, unsigned mask) { return (tau < 32 ? 16 : 0) + __builtin_popcount(mask) + ((unsigned)(tau - LT0) < 32u ? 4 : 0); }
constexpr int TEND = 2 + DSIDE + DGAP + 128;
static_assert(LT0 >= 32 && LT0 + 32 <= TEND && TD <= 4 && DGAP % 2 == 0 && DSIDE % 2 == 0 && TEND < 160, "schedule");
}

__global__ __launch_bounds__(512, 2) void fused_kernel(const Params p) {
  __shared__ __attribute__((aligned(16))) unsigned char shm[LDS_BYTES];
  LAS unsigned char* lds = (LAS unsigned char*)shm;
  const int tid = threadIdx.x, wid = __builtin_amdgcn_readfirstlane(tid >> 6), lane = tid & 63;
  const int rk = p.rank & 7;
  char* const myws = p.ws[rk];
  unsigned call;
  size_t par;

  {
    const int b = ((int)blockIdx.x + GRID - NAG) & (GRID - 1);
    const bool hasx = wid >= 6;
    const int ex = ((wid & 1) << 8) + b, xkt = ex >> 4, xb = ex & 15;
    u32x4 vx[4] = {};
    if (hasx) {
      const signed char* s = p.x + (size_t)xb * 256 * KS + xkt * 16;
#pragma unroll
      for (int c = 0; c < 4; ++c) vx[c] = *(const GAS u32x4*)(s + (unsigned)((lane + c * 64) * KS));
    }
    const int ew = (wid << 8) + b, wkt = ew >> 6, col = ew & 63;
    unsigned q[16];
    { const signed char* s = p.w + (size_t)wkt * 16 * N_ + col * 256;
#pragma unroll
      for (int j = 0; j < 16; ++j) q[j] = *(const GAS unsigned*)(s + (unsigned)(lane * 4 + j * N_)); }
    asm volatile("" ::: "memory");
    unsigned tkt = 0;
    if (lane == 0) tkt = __hip_atomic_fetch_add((unsigned*)(myws + WS_CTR + (size_t)(blockIdx.x * 8 + wid) * 64), 1u, __ATOMIC_RELAXED, __HIP_MEMORY_SCOPE_AGENT);
    call = (unsigned)__builtin_amdgcn_readfirstlane((int)tkt) + 1u;
    par = WS_DATA + (size_t)(call & 1u) * PARB;
    u32x4 vw[4];
#pragma unroll
    for (int t = 0; t < 4; ++t) {
      const unsigned t0 = perm(q[4 * t + 1], q[4 * t], 0x05010400u), t1 = perm(q[4 * t + 1], q[4 * t], 0x07030602u);
      const unsigned t2 = perm(q[4 * t + 3], q[4 * t + 2], 0x05010400u), t3 = perm(q[4 * t + 3], q[4 * t + 2], 0x07030602u);
      vw[0][t] = perm(t2, t0, 0x05040100u); vw[1][t] = perm(t2, t0, 0x07060302u);
      vw[2][t] = perm(t3, t1, 0x05040100u); vw[3][t] = perm(t3, t1, 0x07060302u);
    }
    const unsigned uw = (unsigned)(wkt * KSTR + 16 + col), ux = (unsigned)(xkt * KSTR + xb);
    const unsigned lo = (unsigned)rk * SRCB + (unsigned)lane * 16u;
#define PUT(d, o, v) asm volatile("global_store_dwordx4 %0, %1, %5 sc0 sc1\n\tglobal_store_dwordx4 %0, %2, %5 offset:1024 sc0 sc1\n\t" \
      "global_store_dwordx4 %0, %3, %5 offset:2048 sc0 sc1\n\tglobal_store_dwordx4 %0, %4, %5 offset:3072 sc0 sc1\n\ts_nop 1" \
      :: "v"(o), "v"(v[0]), "v"(v[1]), "v"(v[2]), "v"(v[3]), "s"(d) : "memory")
    PUT(myws + par, lo + uw * 4096u, vw);
    if (hasx) PUT(myws + par, lo + ux * 4096u, vx);
    const bool dw = wkt < TD && col < 15 && start_of(col) <= wkt, dx = hasx && xkt < TD;
    asm volatile("s_waitcnt vmcnt(0)" ::: "memory");
    __builtin_amdgcn_s_barrier();
    if (tid == 64) __hip_atomic_store((unsigned*)(myws + WS_LFLAG) + blockIdx.x, call, __ATOMIC_RELAXED, __HIP_MEMORY_SCOPE_AGENT);
    if (dw || dx) {
      if (dx) { vw[0] = vx[0]; vw[1] = vx[1]; vw[2] = vx[2]; vw[3] = vx[3]; }
      const unsigned o = lo + (dx ? ux : uw) * 4096u;
      __builtin_amdgcn_s_sleep(1);
#pragma unroll
      for (int i = 1; i < NR; ++i) PUT(p.ws[(rk + 1 + (i + b) % (NR - 1)) & 7] + par, o, vw);
      asm volatile("s_waitcnt vmcnt(0)" ::: "memory");
      if (lane < NR) __hip_atomic_store((unsigned*)(p.ws[lane & 7] + WS_FLAG + rk * 4 + (size_t)(dx ? ux : uw) * 32u), call, __ATOMIC_RELAXED, __HIP_MEMORY_SCOPE_SYSTEM);
    }
  }
  auto lane_id = [&]() -> int { int l; asm volatile("v_mbcnt_lo_u32_b32 %0, -1, 0\n\tv_mbcnt_hi_u32_b32 %0, -1, %0" : "=v"(l)); return l; };
  auto wait_words = [&](unsigned off, int nw, unsigned stride, unsigned long long own) -> bool {
    const unsigned t0 = (unsigned)__builtin_amdgcn_s_memrealtime();
    for (;;) {
      unsigned long long ok = ~0ull;
      unsigned v[4];
#pragma unroll
      for (int i = 0; i < 4; ++i) if (i < nw) v[i] = __hip_atomic_load((const unsigned*)(myws + (off + i * stride)), __ATOMIC_RELAXED, __HIP_MEMORY_SCOPE_SYSTEM);
#pragma unroll
      for (int i = 0; i < 4; ++i) if (i < nw) ok &= __ballot((int)(v[i] - call) >= 0);
      if ((ok | own) == ~0ull) return true;
      unsigned to; asm volatile("v_mov_b32 %0, %1" : "=v"(to) : "i"((unsigned)WS_TMO));
      unsigned* const tp = (unsigned*)(myws + to);
      const unsigned now = (unsigned)__builtin_amdgcn_s_memrealtime();
      if (__builtin_amdgcn_readfirstlane((int)__hip_atomic_load(tp, __ATOMIC_RELAXED, __HIP_MEMORY_SCOPE_AGENT)) == (int)call || now - t0 > DEADLINE) {
        unsigned cv; asm volatile("v_mov_b32 %0, %1" : "=v"(cv) : "s"(call));
        __hip_atomic_store(tp, cv, __ATOMIC_RELAXED, __HIP_MEMORY_SCOPE_AGENT);
        return false;
      }
      __builtin_amdgcn_s_sleep(4);
    }
  };

  if (blockIdx.x < NAG) {
    if (wid != 0) return;
    const int lane = lane_id();
    char* const fl_base = p.ws[lane & 7] + WS_FLAG + rk * 4;
    int ndirect = 0;
    for (int t = 0; t < TD; ++t) ndirect += step_units(t, slot_mask(t));
    const unsigned npush = (unsigned)(NUNITS - ndirect);
    const unsigned qbase = (call - 1u) * (npush + (unsigned)NAG);
    const char* const qctr = myws + WS_QUEUE;
    const char* const src = myws + par + (size_t)rk * SRCB;
    char* dst[NR - 1];
#pragma unroll
    for (int q = 0; q < NR - 1; ++q) dst[q] = p.ws[(rk + 1 + (q + (int)blockIdx.x) % (NR - 1)) & 7] + par + (size_t)rk * SRCB;
    const unsigned l16 = (unsigned)lane * 16u, vzero = 0u, vone = 1u;
    int tau = TD, base = 0;
    unsigned mask = slot_mask(TD); int cnt = step_units(TD, mask);
    auto decode = [&](int idx) -> unsigned {
      while (idx >= base + cnt && tau < TEND - 1) { base += cnt; ++tau; mask = slot_mask(tau); cnt = step_units(tau, mask); }
      int r = idx - base; const int kt = tau & 31;
      if (tau < 32) { if (r < 16) return (unsigned)(kt * KSTR + r); r -= 16; }
      const int nreg = __builtin_popcount(mask);
      unsigned u;
      if (r >= nreg) u = (unsigned)((tau - LT0) * KSTR + 76 + (r - nreg));
      else { unsigned m = mask; for (int i = 0; i < r; ++i) m &= m - 1u;
        const int slot = __builtin_ctz(m | 0x4000u);
        u = (unsigned)(kt * KSTR + 16 + 15 * ((tau - start_of(slot)) >> 5) + slot); }
      return u < (unsigned)NUNITS ? u : 0u;
    };
#define CLB_A "v224", "v225", "v226", "v227", "v228", "v229", "v230", "v231", "v232", "v233", "v234", "v235", "v236", "v237", "v238", "v239"
#define CLB_B "v240", "v241", "v242", "v243", "v244", "v245", "v246", "v247", "v248", "v249", "v250", "v251", "v252", "v253", "v254", "v255"
#define CLB "memory", "v223", CLB_A, CLB_B
#define DRAW() asm volatile("s_mov_b64 exec, 1\n\tglobal_atomic_add v223, %0, %1, %2 sc0\n\ts_mov_b64 exec, -1" :: "v"(vzero), "v"(vone), "s"(qctr) : CLB)
#define LOADU(R, u) do { const unsigned _o = (u) * 4096u + l16; \
      asm volatile("global_load_dwordx4 v[%c2:%c2+3], %0, %1 sc0 sc1\n\tglobal_load_dwordx4 v[%c2+4:%c2+7], %0, %1 offset:1024 sc0 sc1\n\t" \
                   "global_load_dwordx4 v[%c2+8:%c2+11], %0, %1 offset:2048 sc0 sc1\n\tglobal_load_dwordx4 v[%c2+12:%c2+15], %0, %1 offset:3072 sc0 sc1" \
                   :: "v"(_o), "s"(src), "i"(R) : CLB); } while (0)
#define STOREU(R, u) do { const unsigned _o = (u) * 4096u + l16; _Pragma("unroll") for (int q = 0; q < NR - 1; ++q) \
      asm volatile("global_store_dwordx4 %0, v[%c2:%c2+3], %1 sc0 sc1\n\tglobal_store_dwordx4 %0, v[%c2+4:%c2+7], %1 offset:1024 sc0 sc1\n\t" \
                   "global_store_dwordx4 %0, v[%c2+8:%c2+11], %1 offset:2048 sc0 sc1\n\tglobal_store_dwordx4 %0, v[%c2+12:%c2+15], %1 offset:3072 sc0 sc1\n\ts_nop 1" \
                   :: "v"(_o), "s"(dst[q]), "i"(R) : CLB); } while (0)
#define FLAGU(u) do { char* _f = fl_base + (size_t)(u) * 32u; \
      asm volatile("s_mov_b64 exec, 0xff\n\tglobal_store_dword %0, %1, off sc0 sc1\n\ts_mov_b64 exec, -1" :: "v"(_f), "v"(call) : CLB); } while (0)
#define WAIT_TK(n, tk) asm volatile("s_waitcnt vmcnt(" #n ")\n\tv_readfirstlane_b32 %0, v223" : "=s"(tk) :: CLB)
    unsigned tk;
    DRAW(); WAIT_TK(0, tk);
    unsigned idx = tk - qbase;
    if (!wait_words((unsigned)WS_LFLAG + lane * 4u, 4, 256u, 0ull)) return;
    if (idx < npush) {
      unsigned cur = decode((int)idx), prev = NUNITS;
      LOADU(224, cur); DRAW();
      asm volatile("s_waitcnt vmcnt(0)" ::: CLB);
#define STEP(RC, RN) { \
        WAIT_TK(29, tk); \
        idx = tk - qbase; \
        if (idx >= npush) { STOREU(RC, cur); asm volatile("s_waitcnt vmcnt(28)" ::: CLB); FLAGU(prev); break; } \
        const unsigned nxt = decode((int)idx); \
        LOADU(RN, nxt); DRAW(); \
        STOREU(RC, cur); \
        asm volatile("s_waitcnt vmcnt(28)" ::: CLB);             \
        FLAGU(prev); \
        prev = cur; cur = nxt; }
      for (;;) { STEP(224, 240) STEP(240, 224) }
      asm volatile("s_waitcnt vmcnt(0)" ::: CLB);
      FLAGU(cur);
    }
    asm volatile("s_waitcnt vmcnt(0)" ::: "memory");
    return;
  }

  const int wr = wid >> 2, wc = wid & 3, fr = lane & 15, fq = lane >> 4;
  int pm, slot;
  { const int b = blockIdx.x - NAG, x = b & 7, r = b >> 3, a = x >> 1, hb = x & 1;
    if (r < 28) { pm = 4 * a + (r & 3); slot = hb * 8 + (r >> 2); } else { pm = 4 * a + 2 * hb + (r - 28); slot = 7; } }
  const bool has_half = (slot & 7) < 4;
  const int hcol = (slot >> 3) * 2 + ((slot >> 1) & 1), hrow = slot & 1;
  const float scale = __builtin_bit_cast(float, __builtin_amdgcn_readfirstlane(__builtin_bit_cast(int, p.sx[0] * p.sw[0])));
  const unsigned voff = (unsigned)((wid >> 2) * 4 + fq) * SRCB + (unsigned)((wid & 3) * 256 + fr * 16);
  const unsigned lds0 = (unsigned)(size_t)lds, ldsw = lds0 + LDS_OP + (unsigned)wid * 2048u;
  int aoff = LDS_OP + wr * 1024 + lane * 16, boff = LDS_OP + 4 * HTB + wc * 2048 + lane * 16;
  asm volatile("" : "+v"(aoff), "+v"(boff));
  const unsigned coff = (unsigned)((wr * 64 + 4 * fq) * (N_ * 4) + (wc * 64 + 4 * fr) * 4);
  const u32x4 srd = {(unsigned)(size_t)myws, (unsigned)((size_t)myws >> 32) & 0xffffu, (unsigned)WS_NEED, 0x00020000u};
  const unsigned cA = (unsigned)par + (unsigned)pm * 4096u;
  const unsigned cW = (unsigned)par + (unsigned)(16 + slot) * 4096u;
  const unsigned hA = cA + hrow * GH, hB = (unsigned)par + (unsigned)(76 + hcol) * 4096u;
  const unsigned long long own = 0x0101010101010101ull << rk;
#define SA(b, h) (((b) * 2 + (h)) * HTB)
#define SB(b, h) ((4 + (b) * 2 + (h)) * HTB)
#define STAGE(bufoff, goff) asm volatile("s_add_u32 m0, %3, %4\n\ts_nop 0\n\tbuffer_load_dwordx4 %0, %1, %2 offen sc0 sc1 lds\n\t" \
    "buffer_load_dwordx4 %0, %1, %2 offen offset:1024 sc0 sc1 lds" :: "v"(voff), "s"(srd), "s"(goff), "s"(ldsw), "i"(bufoff) : "scc", "memory")
#define FLAG_FETCH(ko, sl) asm volatile("s_mov_b32 m0, %3\n\ts_nop 0\n\tbuffer_load_dword %0, %1, %2 offen sc0 sc1 lds" :: "v"(vf), "s"(srd), "s"((ko) >> 7), "s"(lds0 + (sl)) : "memory")
#define LDA_(dst, base) do { _Pragma("unroll") for (int m = 0; m < 4; ++m) _Pragma("unroll") for (int k = 0; k < 2; ++k) dst[m][k] = *(const LAS i32x4*)(lds + (base) + m * 2048 + k * 8192); } while (0)
#define LDB_(dst, base) do { _Pragma("unroll") for (int n = 0; n < 2; ++n) _Pragma("unroll") for (int k = 0; k < 2; ++k) dst[n][k] = *(const LAS i32x4*)(lds + (base) + n * 1024 + k * 8192); } while (0)
#define LDA(dst, b, h) LDA_(dst, SA(b, h) + aoff)
#define LDB(dst, b, h) LDB_(dst, SA(b, h) + boff)
#define MFMA2(c, a0, b0, a1, b1, Z) do { \
    if (Z) asm volatile("v_mfma_i32_16x16x64_i8 %0, %1, %2, 0.15915494\n\tv_mfma_i32_16x16x64_i8 %0, %3, %4, %0" : "+v"(c) : "v"(a0), "v"(b0), "v"(a1), "v"(b1)); \
    else asm volatile("v_mfma_i32_16x16x64_i8 %0, %1, %2, %0\n\tv_mfma_i32_16x16x64_i8 %0, %3, %4, %0" : "+v"(c) : "v"(a0), "v"(b0), "v"(a1), "v"(b1)); } while (0)
#define MMA2(ai, Z, PARK) do { __builtin_amdgcn_s_setprio(1); \
    _Pragma("unroll") for (int i = 0; i < 16; ++i) { \
      const int k = i & 1, m = i >> 2, c = (m & 1) ? 3 - (i & 3) : (i & 3); \
      if ((PARK) == 1 && m >= 2) asm volatile("v_mfma_i32_16x16x64_i8 %0, %1, %2, 0.15915494\n\tv_mfma_i32_16x16x64_i8 %0, %3, %4, %0" \
          : "=&v"(park[m & 1][c]) : "v"(At[m][k]), "v"(c < 2 ? B0[c & 1][k] : B1[c & 1][k]), "v"(At[m][k ^ 1]), "v"(c < 2 ? B0[c & 1][k ^ 1] : B1[c & 1][k ^ 1])); \
      else if ((PARK) == 2 && m >= 2) asm volatile("v_mfma_i32_16x16x64_i8 %0, %1, %2, %5\n\tv_mfma_i32_16x16x64_i8 %0, %3, %4, %0" \
          : "+v"(acc[ai][c >> 1][m][c & 1]) : "v"(At[m][k]), "v"(c < 2 ? B0[c & 1][k] : B1[c & 1][k]), "v"(At[m][k ^ 1]), "v"(c < 2 ? B0[c & 1][k ^ 1] : B1[c & 1][k ^ 1]), "v"(park[m & 1][c])); \
      else MFMA2(acc[ai][c >> 1][m][c & 1], At[m][k], (c < 2 ? B0[c & 1][k] : B1[c & 1][k]), At[m][k ^ 1], (c < 2 ? B0[c & 1][k ^ 1] : B1[c & 1][k ^ 1]), Z); } \
    __builtin_amdgcn_s_setprio(0); } while (0)
#define WAIT_V(n) asm volatile("s_waitcnt vmcnt(" #n ")" ::: "memory")
#define WAIT_L(n) do { asm volatile("" ::: "memory"); __builtin_amdgcn_s_waitcnt(0xC07F); asm volatile("" ::: "memory"); } while (0)
#define BAR __builtin_amdgcn_s_barrier()
#define SCHED __builtin_amdgcn_sched_barrier(0)
#define PIN do { asm volatile("" ::: "memory"); SCHED; } while (0)
#define DEQ(a) fmaxf((float)((a) - BIAS) * scale, 0.f)
#define EPI_ST(s0, s1) do { \
    _Pragma("unroll") for (int s = (s0); s < (s1); ++s) { const int ai = s >> 4, m = (s >> 2) & 3, j = s & 3; f32x4 v; \
      v[0] = DEQ(acc[ai][0][m][0][j]); v[1] = DEQ(acc[ai][0][m][1][j]); v[2] = DEQ(acc[ai][1][m][0][j]); v[3] = DEQ(acc[ai][1][m][1][j]); \
      asm volatile("buffer_store_dwordx4 %0, %1, %2, %3 offen nt\n\ts_nop 1" :: "v"(v), "v"(coff), "s"(crs), "s"((ai * 128 + m * 16 + j) * (N_ * 4)) : "memory"); } } while (0)
  auto flag_off = [&](int rnd) -> unsigned { const int l = lane_id();
    return (unsigned)WS_FLAG + (unsigned)(((l & 16) ? 16 + 15 * rnd + slot : pm) * 32 + ((l & 8) ? (int)(KTS >> 7) : 0) + (l & 7) * 4); };
  auto nextk = [&](unsigned k) -> unsigned { k += 2 * KTS; return k < 32 * KTS ? k : 0u; };

  i32x4 acc[2][2][4][2];
  i32x4 At[4][2], B0[2][2], B1[2][2], park[2][4];
  unsigned k0 = (unsigned)start_of(slot) * KTS, k1 = nextk(k0);
  unsigned vf = flag_off(0);
  if (wid == 0) {
    wait_words(vf + (k0 >> 7), 1, 0u, own);
    wait_words((unsigned)WS_LFLAG + lane_id() * 4u, 4, 256u, 0ull);
    FLAG_FETCH(k1, 0u);
  }
  asm volatile("s_waitcnt vmcnt(0)" ::: "memory");
  BAR;
  { const unsigned a = cA + k0, b = cW + k0;
    STAGE(SB(0, 0), b); STAGE(SB(0, 1), b + GH); STAGE(SA(0, 0), a); STAGE(SA(0, 1), a + GH);
    if (wr == 1) BAR;
    WAIT_V(2); BAR;
    STAGE(SB(1, 0), b + KTS); STAGE(SA(1, 0), a + KTS); STAGE(SB(1, 1), b + GH + KTS);
    WAIT_V(0); BAR; PIN; }
#pragma unroll
  for (int a = 0; a < 2; ++a)
#pragma unroll
    for (int b = 0; b < 2; ++b)
#pragma unroll
      for (int m = 0; m < 4; ++m)
#pragma unroll
        for (int n = 0; n < 2; ++n) asm volatile("" : "=v"(acc[a][b][m][n]));
  auto mk_rs = [](const float* b, unsigned n) { return (u32x4){(unsigned)(size_t)b, (unsigned)((size_t)b >> 32) & 0xffffu, n, 0x00020000u}; };
  u32x4 crs = mk_rs(p.out, 0u);
  unsigned fsl = 0;
#define KT(b, ah, a2, b2, Z, PARK, V1, V2, e0, e1, e2, FL) do { \
    unsigned fv = 0; \
    LDB(B0, b, 0); LDB(B1, b, 1); SCHED; LDA(At, b, 0); STAGE(SA((b) ^ 1, 1), ah); \
    if ((FL) && wid == 0) { fv = *(const LAS unsigned*)(lds + fsl + lane_id() * 4); if (newf) vf = flag_off(un); FLAG_FETCH(k2, fsl ^ 256u); } \
    PIN; EPI_ST(e0, e1); PIN; \
    WAIT_V(V1); WAIT_L(0); \
    if ((FL) && wid == 0) { if ((__ballot((int)(fv - call) >= 0) | own) != ~0ull) wait_words(flag_off(c1) + (k1 >> 7), 1, 0u, own); \
      if ((FL) == 2 && tohalf) { const int l = lane_id(); wait_words((unsigned)WS_FLAG + (unsigned)((76 + hcol) * 32 + (l >> 3) * (int)(KTS >> 7) + (l & 7) * 4), 4, KTS >> 4, own); } \
    } \
    BAR; MMA2(0, Z, 0); BAR; PIN; \
    LDA(At, b, 1); STAGE(SB(b, 0), b2); STAGE(SB(b, 1), (b2) + GH); STAGE(SA(b, 0), a2); PIN; EPI_ST(e1, e2); PIN; \
    WAIT_V(V2); WAIT_L(0); BAR; MMA2(1, Z, PARK); BAR; PIN; } while (0)
#define PAIR(C1, NEWF, Z, Va, Vb, Vc, Vd, e0, e1, e2, e3, e4, FL) do { \
    const int c1 = (C1); const bool newf = (NEWF); const unsigned k2 = nextk(k1); \
    unsigned a1 = cA + k1, b1 = cW + (unsigned)c1 * (15u * 4096u) + k1, ah1 = a1 + GH; \
    if ((FL) == 2 && tohalf) { a1 = hA; b1 = hB; ah1 = hB + 2 * KTS; }     \
    KT(0, cA + k0 + KTS + GH, a1, b1, Z, Z, Va, Vb, e0, e1, e2, FL); \
    KT(1, ah1, a1 + KTS, b1 + KTS, 0, 2 * (Z), Vc, Vd, e2, e3, e4, 0); \
    k0 = k1; k1 = k2; fsl ^= 256u; } while (0)
  for (int ui = 0;; ++ui) {
    const bool has_next = ui < 3, tohalf = !has_next && has_half;
    const int un = has_next ? ui + 1 : ui;
    PAIR(ui, false, 1, 24, 32, 28, 24, 8, 16, 24, 28, 32, 1);
    crs = mk_rs(p.out + (size_t)pm * 256 * N_ + (ui * 15 + slot) * 256, 256u * N_ * 4u);
    for (int t = 1; t < 15; ++t) PAIR(ui, t == 14, 0, 8, 8, 8, 8, 0, 0, 0, 0, 0, 1);
    PAIR(un, false, 0, 8, 8, 8, 16, 0, 0, 0, 0, 8, 2);
    if (!has_next) break;
  }
  if (has_half) {
    int aoff2 = 8 * HTB + aoff; asm volatile("" : "+v"(aoff2));
    const int boff2 = boff - 4 * HTB;
#define HST(j, w, g) do { if ((j) == 0) { if ((w) == 0) STAGE(SA(0, 1), g); else if ((w) == 1) STAGE(SA(1, 1), g); else STAGE(8 * HTB, g); } \
      else { if ((w) == 0) STAGE(SB(((j) + 1) & 1, 0), g); else if ((w) == 1) STAGE(SB(((j) + 1) & 1, 1), g); else STAGE(SA(((j) + 1) & 1, 0), g); } } while (0)
#define HKT(j, tn, Z, V, e0, e1) do { \
      const unsigned a = hA + (unsigned)(tn) * KTS, b = hB + (unsigned)(tn) * KTS; \
      if ((j) < 2) LDB(B0, j, 0); else LDB_(B0, SA(0, 1) + boff2); \
      HST(j, 0, b); \
      if ((j) < 2) LDB(B1, j, 1); else LDB_(B1, SA(1, 1) + boff2); \
      HST(j, 1, b + GH); \
      SCHED; if ((j) < 2) LDA(At, j, 0); else LDA_(At, aoff2); \
      HST(j, 2, a); \
      PIN; EPI_ST(e0, e1); PIN; \
      WAIT_V(V); WAIT_L(0); BAR; MMA2(0, Z, 0); BAR; PIN; } while (0)
    HKT(0, 2, 1, 22, 8, 16); HKT(1, 3, 0, 18, 16, 20); HKT(2, 4, 0, 14, 20, 24);
    HKT(0, 5, 0, 14, 24, 28); HKT(1, 6, 0, 14, 28, 32); HKT(2, 7, 0, 10, 0, 0);
    crs = mk_rs(p.out + (size_t)(pm * 256 + hrow * 128) * N_ + (60 + hcol) * 256, 128u * N_ * 4u);
    for (int t = 6; t < 30; t += 3) { HKT(0, t + 2, 0, 6, 0, 0); HKT(1, t + 3, 0, 6, 0, 0); HKT(2, t + 4, 0, 6, 0, 0); }
    HKT(0, 31, 0, 6, 0, 0); HKT(1, 31, 0, 6, 0, 0);
  }
  const unsigned gave_up = __hip_atomic_load((unsigned*)(myws + WS_TMO), __ATOMIC_RELAXED, __HIP_MEMORY_SCOPE_AGENT);
  asm volatile("s_nop 7" ::: "memory"); SCHED;
  if (has_half) EPI_ST(0, 4); else EPI_ST(8, 12);
  if (wr == 0) BAR;
  if (has_half) EPI_ST(4, 16); else EPI_ST(12, 32);
  PIN;
  if (__builtin_amdgcn_readfirstlane((int)gave_up) == (int)call) {
    WAIT_V(0);
    const float q = __builtin_nanf(""); const f32x4 v = {q, q, q, q};
    for (int ui = 0; ui < (has_half ? 5 : 4); ++ui) {
      crs = ui < 4 ? mk_rs(p.out + (size_t)pm * 256 * N_ + (ui * 15 + slot) * 256, 256u * N_ * 4u)
                   : mk_rs(p.out + (size_t)(pm * 256 + hrow * 128) * N_ + (60 + hcol) * 256, 128u * N_ * 4u);
      for (int s = 0; s < 32; ++s)
        asm volatile("buffer_store_dwordx4 %0, %1, %2, %3 offen\n\ts_nop 1" :: "v"(v), "v"(coff), "s"(crs), "s"(((s >> 4) * 128 + ((s >> 2) & 3) * 16 + (s & 3)) * (N_ * 4)) : "memory");
    }
  }
  WAIT_V(0);
}

extern "C" void kernel_launch(int rank, int nranks, ncclComm_t comm, hipStream_t stream,
                              void* const* d_in, const int* in_sizes, int n_in,
                              void* d_out, int out_size, const int* in_dtypes, int out_dtype,
                              void* const* d_out_all, void* const* d_ws_all, size_t ws_bytes) {
  (void)comm; (void)in_dtypes; (void)out_dtype; (void)d_out_all;
  if (nranks != NR || (unsigned)rank >= (unsigned)NR || n_in != 4 || ws_bytes < WS_NEED || out_size != M_ * N_ ||
      in_sizes[0] != M_ * KS || in_sizes[1] != KS * N_) {
    fprintf(stderr, "kernel_launch: unexpected configuration\n");
    return;
  }
  Params p;
  for (int i = 0; i < NR; ++i) p.ws[i] = (char*)d_ws_all[i];
  p.x = (const signed char*)d_in[0]; p.w = (const signed char*)d_in[1];
  p.sx = (const float*)d_in[2]; p.sw = (const float*)d_in[3];
  p.out = (float*)d_out; p.rank = rank;
  hipLaunchKernelGGL(fused_kernel, dim3(GRID), dim3(512), 0, stream, p);
}
```

```cpp
#include <hip/hip_runtime.h>
#include <rccl/rccl.h>
#include <cstdio>

#define LAS __attribute__((address_space(3)))
#define GAS __attribute__((address_space(1)))
typedef int i32x4 __attribute__((ext_vector_type(4)));
typedef unsigned u32x4 __attribute__((ext_vector_type(4)));
typedef float f32x4 __attribute__((ext_vector_type(4)));

namespace {
constexpr int M_ = 4096, N_ = 16384, K_ = 4096, NR = 8;
constexpr int KS = K_ / NR;
constexpr int HTB = 16384;
constexpr int GH = 2048;
constexpr int LDS_OP = 512;
constexpr int LDS_BYTES = LDS_OP + 9 * HTB;
constexpr int BIAS = 0x3e22f983;
constexpr int NAG = 16;
constexpr int GRID = 256;
constexpr int TD = 2;
constexpr int DGAP = 16;
constexpr int DSIDE = 4;
constexpr int LT0 = 96;
constexpr size_t WS_CTR = 0;
constexpr size_t WS_TMO = 128 * 1024;
constexpr size_t WS_QUEUE = 128 * 1024 + 256;
constexpr size_t WS_LFLAG = 128 * 1024 + 1024;
constexpr size_t WS_FLAG = 256 * 1024;
constexpr size_t WS_DATA = 1024 * 1024;
constexpr int KSTR = 80;
constexpr int NUNITS = 32 * KSTR;
constexpr unsigned KTS = KSTR * 4096u;
constexpr unsigned SRCB = 32 * KTS + 256u;
constexpr size_t PARB = 81ull << 20;
static_assert(PARB >= 8ull * SRCB, "parity region");
constexpr size_t WS_NEED = WS_DATA + 2 * PARB;
constexpr unsigned DEADLINE = 50000000u;

struct Params {
  char* ws[NR];
  const signed char* x; const signed char* w; const float* sx; const float* sw;
  float* out; int rank;
};

__device__ __forceinline__ unsigned perm(unsigned hi, unsigned lo, unsigned sel) { return __builtin_amdgcn_perm(hi, lo, sel); }

__device__ __forceinline__ int start_of(int slot) { return (slot & 2) + ((slot & 4) ? DGAP : 0) + (slot < 7 ? DSIDE : 0); }
__device__ __forceinline__ unsigned slot_mask(int tau) {
  unsigned m = 0;
#pragma unroll
  for (int j = 0; j < 15; ++j) m |= ((unsigned)(tau - start_of(j)) < 128u ? 1u : 0u) << j;
  return m;
}
__device__ __forceinline__ int step_units(int tau, unsigned mask) { return (tau < 32 ? 16 : 0) + __builtin_popcount(mask) + ((unsigned)(tau - LT0) < 32u ? 4 : 0); }
constexpr int TEND = 2 + DSIDE + DGAP + 128;
static_assert(LT0 >= 32 && LT0 + 32 <= TEND && TD <= 4 && TD % 2 == 0 && LT0 % 2 == 0 && DGAP % 2 == 0 && DSIDE % 2 == 0 && TEND < 160, "schedule");
}

__global__ __launch_bounds__(512, 2) void fused_kernel(const Params p) {
  __shared__ __attribute__((aligned(16))) unsigned char shm[LDS_BYTES];
  LAS unsigned char* lds = (LAS unsigned char*)shm;
  const int tid = threadIdx.x, wid = __builtin_amdgcn_readfirstlane(tid >> 6), lane = tid & 63;
  const int rk = p.rank & 7;
  char* const myws = p.ws[rk];
  unsigned call;
  size_t par;

  {
    const int b = ((int)blockIdx.x + GRID - NAG) & (GRID - 1);
    const bool hasx = wid >= 6;
    const int ex = ((wid & 1) << 8) + b, xkt = ex >> 4, xb = ex & 15;
    u32x4 vx[4] = {};
    if (hasx) {
      const signed char* s = p.x + (size_t)xb * 256 * KS + xkt * 16;
#pragma unroll
      for (int c = 0; c < 4; ++c) vx[c] = *(const GAS u32x4*)(s + (unsigned)((lane + c * 64) * KS));
    }
    const int ew = (wid << 8) + b, wkt = ew >> 6, col = ew & 63;
    unsigned q[16];
    { const signed char* s = p.w + (size_t)wkt * 16 * N_ + col * 256;
#pragma unroll
      for (int j = 0; j < 16; ++j) q[j] = *(const GAS unsigned*)(s + (unsigned)(lane * 4 + j * N_)); }
    asm volatile("" ::: "memory");
    unsigned tkt = 0;
    if (lane == 0) tkt = __hip_atomic_fetch_add((unsigned*)(myws + WS_CTR + (size_t)(blockIdx.x * 8 + wid) * 64), 1u, __ATOMIC_RELAXED, __HIP_MEMORY_SCOPE_AGENT);
    call = (unsigned)__builtin_amdgcn_readfirstlane((int)tkt) + 1u;
    par = WS_DATA + (size_t)(call & 1u) * PARB;
    u32x4 vw[4];
#pragma unroll
    for (int t = 0; t < 4; ++t) {
      const unsigned t0 = perm(q[4 * t + 1], q[4 * t], 0x05010400u), t1 = perm(q[4 * t + 1], q[4 * t], 0x07030602u);
      const unsigned t2 = perm(q[4 * t + 3], q[4 * t + 2], 0x05010400u), t3 = perm(q[4 * t + 3], q[4 * t + 2], 0x07030602u);
      vw[0][t] = perm(t2, t0, 0x05040100u); vw[1][t] = perm(t2, t0, 0x07060302u);
      vw[2][t] = perm(t3, t1, 0x05040100u); vw[3][t] = perm(t3, t1, 0x07060302u);
    }
    const unsigned uw = (unsigned)(wkt * KSTR + 16 + col), ux = (unsigned)(xkt * KSTR + xb);
    const unsigned lo = (unsigned)rk * SRCB + (unsigned)lane * 16u;
#define PUT(d, o, v) asm volatile("global_store_dwordx4 %0, %1, %5 sc0 sc1\n\tglobal_store_dwordx4 %0, %2, %5 offset:1024 sc0 sc1\n\t" \
      "global_store_dwordx4 %0, %3, %5 offset:2048 sc0 sc1\n\tglobal_store_dwordx4 %0, %4, %5 offset:3072 sc0 sc1\n\ts_nop 1" \
      :: "v"(o), "v"(v[0]), "v"(v[1]), "v"(v[2]), "v"(v[3]), "s"(d) : "memory")
    PUT(myws + par, lo + uw * 4096u, vw);
    if (hasx) PUT(myws + par, lo + ux * 4096u, vx);
    const bool dw = wkt < TD && col < 15 && start_of(col) <= wkt, dx = hasx && xkt < TD;
    asm volatile("s_waitcnt vmcnt(0)" ::: "memory");
    __builtin_amdgcn_s_barrier();
    if (tid == 64) __hip_atomic_store((unsigned*)(myws + WS_LFLAG) + blockIdx.x, call, __ATOMIC_RELAXED, __HIP_MEMORY_SCOPE_AGENT);
    if (dw || dx) {
      if (dx) { vw[0] = vx[0]; vw[1] = vx[1]; vw[2] = vx[2]; vw[3] = vx[3]; }
      const unsigned o = lo + (dx ? ux : uw) * 4096u; const int dkt = dx ? xkt : wkt;
      __builtin_amdgcn_s_sleep(1);
#pragma unroll
      for (int i = 1; i < NR; ++i) PUT(p.ws[(rk + 1 + (i + b) % (NR - 1)) & 7] + par, o, vw);
      asm volatile("s_waitcnt vmcnt(0)" ::: "memory");
      if (lane < NR) __hip_atomic_store((unsigned*)(p.ws[lane & 7] + WS_FLAG + rk * 8 + (size_t)(((dkt >> 1) * KSTR + (dx ? xb : 16 + col)) * 64 + (dkt & 1) * 4)), call, __ATOMIC_RELAXED, __HIP_MEMORY_SCOPE_SYSTEM);
    }
  }
  auto lane_id = [&]() -> int { int l; asm volatile("v_mbcnt_lo_u32_b32 %0, -1, 0\n\tv_mbcnt_hi_u32_b32 %0, -1, %0" : "=v"(l)); return l; };
  auto wait_words = [&](unsigned off, int nw, unsigned stride, unsigned long long own) -> bool {
    const unsigned t0 = (unsigned)__builtin_amdgcn_s_memrealtime();
    for (;;) {
      unsigned long long ok = ~0ull;
      unsigned v[4];
#pragma unroll
      for (int i = 0; i < 4; ++i) if (i < nw) v[i] = __hip_atomic_load((const unsigned*)(myws + (off + i * stride)), __ATOMIC_RELAXED, __HIP_MEMORY_SCOPE_SYSTEM);
#pragma unroll
      for (int i = 0; i < 4; ++i) if (i < nw) ok &= __ballot((int)(v[i] - call) >= 0);
      if ((ok | own) == ~0ull) return true;
      unsigned to; asm volatile("v_mov_b32 %0, %1" : "=v"(to) : "i"((unsigned)WS_TMO));
      unsigned* const tp = (unsigned*)(myws + to);
      const unsigned now = (unsigned)__builtin_amdgcn_s_memrealtime();
      if (__builtin_amdgcn_readfirstlane((int)__hip_atomic_load(tp, __ATOMIC_RELAXED, __HIP_MEMORY_SCOPE_AGENT)) == (int)call || now - t0 > DEADLINE) {
        unsigned cv; asm volatile("v_mov_b32 %0, %1" : "=v"(cv) : "s"(call));
        __hip_atomic_store(tp, cv, __ATOMIC_RELAXED, __HIP_MEMORY_SCOPE_AGENT);
        return false;
      }
      __builtin_amdgcn_s_sleep(4);
    }
  };

  if (blockIdx.x < NAG) {
    if (wid != 0) return;
    const int lane = lane_id();
    char* const fl_base = p.ws[lane & 7] + WS_FLAG + rk * 8;
    int ndirect = 0;
    for (int t = 0; t < TD; t += 2) ndirect += step_units(t, slot_mask(t));
    const unsigned npush = (unsigned)(NUNITS / 2 - ndirect);
    const unsigned qbase = (call - 1u) * (npush + (unsigned)NAG);
    const char* const qctr = myws + WS_QUEUE;
    const char* const src = myws + par + (size_t)rk * SRCB;
    char* dst[NR - 1];
#pragma unroll
    for (int q = 0; q < NR - 1; ++q) dst[q] = p.ws[(rk + 1 + (q + (int)blockIdx.x) % (NR - 1)) & 7] + par + (size_t)rk * SRCB;
    const unsigned l16 = (unsigned)lane * 16u, vzero = 0u, vone = 1u;
    const unsigned long long call2 = ((unsigned long long)call << 32) | call;
    int tau = TD, base = 0;
    unsigned mask = slot_mask(TD); int cnt = step_units(TD, mask);
    auto decode = [&](int idx) -> unsigned {
      while (idx >= base + cnt && tau < TEND - 2) { base += cnt; tau += 2; mask = slot_mask(tau); cnt = step_units(tau, mask); }
      int r = idx - base; const int kp = (tau & 31) >> 1;
      if (tau < 32) { if (r < 16) return (unsigned)(kp * KSTR + r); r -= 16; }
      const int nreg = __builtin_popcount(mask);
      unsigned u;
      if (r >= nreg) u = (unsigned)(((tau - LT0) >> 1) * KSTR + 76 + (r - nreg));
      else { unsigned m = mask; for (int i = 0; i < r; ++i) m &= m - 1u;
        const int slot = __builtin_ctz(m | 0x4000u);
        u = (unsigned)(kp * KSTR + 16 + 15 * ((tau - start_of(slot)) >> 5) + slot); }
      return u < (unsigned)(NUNITS / 2) ? u : 0u;
    };
    auto data_off = [&](unsigned u) -> unsigned { return (u + (u / KSTR) * KSTR) * 4096u; };
#define CLB_A "v224", "v225", "v226", "v227", "v228", "v229", "v230", "v231", "v232", "v233", "v234", "v235", "v236", "v237", "v238", "v239"
#define CLB_B "v240", "v241", "v242", "v243", "v244", "v245", "v246", "v247", "v248", "v249", "v250", "v251", "v252", "v253", "v254", "v255"
#define CLB "memory", "v223", CLB_A, CLB_B
#define DRAW() asm volatile("s_mov_b64 exec, 1\n\tglobal_atomic_add v223, %0, %1, %2 sc0\n\ts_mov_b64 exec, -1" :: "v"(vzero), "v"(vone), "s"(qctr) : CLB)
#define LOADU(R, o) asm volatile("global_load_dwordx4 v[%c2:%c2+3], %0, %1 sc0 sc1\n\tglobal_load_dwordx4 v[%c2+4:%c2+7], %0, %1 offset:1024 sc0 sc1\n\t" \
                   "global_load_dwordx4 v[%c2+8:%c2+11], %0, %1 offset:2048 sc0 sc1\n\tglobal_load_dwordx4 v[%c2+12:%c2+15], %0, %1 offset:3072 sc0 sc1" \
                   :: "v"((o) + l16), "s"(src), "i"(R) : CLB)
#define ST2(R, c, q) asm volatile("global_store_dwordx4 %0, v[%c2:%c2+3], %1 offset:%c3 sc0 sc1\n\tglobal_store_dwordx4 %0, v[%c2+4:%c2+7], %1 offset:%c3+1024 sc0 sc1" \
                   :: "v"(_o), "s"(dst[q]), "i"((R) + 4 * (c)), "i"((c) * 1024) : CLB)
#define ST14(R, c) ST2(R, c, 0); ST2(R, c, 1); ST2(R, c, 2); ST2(R, c, 3); ST2(R, c, 4); ST2(R, c, 5); ST2(R, c, 6)
#define STOREU(R, o) do { const unsigned _o = (o) + l16; ST14(R, 0); ST14(R, 2); } while (0)
#define FLAGU(u) do { char* _f = fl_base + (size_t)(u) * 64u; \
      asm volatile("s_mov_b64 exec, 0xff\n\tglobal_store_dwordx2 %0, %1, off sc0 sc1\n\ts_mov_b64 exec, -1" :: "v"(_f), "v"(call2) : CLB); } while (0)
#define READ_TK(tk) asm volatile("v_readfirstlane_b32 %0, v223" : "=s"(tk) :: CLB)
#define WAIT28() asm volatile("s_waitcnt vmcnt(28)" ::: CLB)
    unsigned tk;
    DRAW(); asm volatile("s_waitcnt vmcnt(0)" ::: CLB); READ_TK(tk);
    unsigned idx = tk - qbase;
    if (!wait_words((unsigned)WS_LFLAG + lane * 4u, 4, 256u, 0ull)) return;
    if (idx < npush) {
      unsigned cur = decode((int)idx), prev = NUNITS / 2;
      unsigned o = data_off(cur);
      LOADU(224, o); DRAW();
      asm volatile("s_waitcnt vmcnt(0)" ::: CLB);
      for (;;) {
        LOADU(240, o + KTS);
        STOREU(224, o);
        READ_TK(tk); idx = tk - qbase;
        const bool more = idx < npush;
        unsigned nxt = cur, on = o;
        if (more) { nxt = decode((int)idx); on = data_off(nxt); }
        WAIT28();
        FLAGU(prev);
        if (more) { LOADU(224, on); DRAW(); }
        STOREU(240, o + KTS);
        if (!more) break;
        WAIT28();
        prev = cur; cur = nxt; o = on;
      }
      asm volatile("s_waitcnt vmcnt(0)" ::: CLB);
      FLAGU(cur);
    }
    asm volatile("s_waitcnt vmcnt(0)" ::: "memory");
    return;
  }

  const int wr = wid >> 2, wc = wid & 3, fr = lane & 15, fq = lane >> 4;
  int pm, slot;
  { const int b = blockIdx.x - NAG, x = b & 7, r = b >> 3, a = x >> 1, hb = x & 1;
    if (r < 28) { pm = 4 * a + (r & 3); slot = hb * 8 + (r >> 2); } else { pm = 4 * a + 2 * hb + (r - 28); slot = 7; } }
  const bool has_half = (slot & 7) < 4;
  const int hcol = (slot >> 3) * 2 + ((slot >> 1) & 1), hrow = slot & 1;
  const float scale = __builtin_bit_cast(float, __builtin_amdgcn_readfirstlane(__builtin_bit_cast(int, p.sx[0] * p.sw[0])));
  const unsigned voff = (unsigned)((wid >> 2) * 4 + fq) * SRCB + (unsigned)((wid & 3) * 256 + fr * 16);
  const unsigned lds0 = (unsigned)(size_t)lds, ldsw = lds0 + LDS_OP + (unsigned)wid * 2048u;
  int aoff = LDS_OP + wr * 1024 + lane * 16, boff = LDS_OP + 4 * HTB + wc * 2048 + lane * 16;
  asm volatile("" : "+v"(aoff), "+v"(boff));
  const unsigned coff = (unsigned)((wr * 64 + 4 * fq) * (N_ * 4) + (wc * 64 + 4 * fr) * 4);
  const u32x4 srd = {(unsigned)(size_t)myws, (unsigned)((size_t)myws >> 32) & 0xffffu, (unsigned)WS_NEED, 0x00020000u};
  const unsigned cA = (unsigned)par + (unsigned)pm * 4096u;
  const unsigned cW = (unsigned)par + (unsigned)(16 + slot) * 4096u;
  const unsigned hA = cA + hrow * GH, hB = (unsigned)par + (unsigned)(76 + hcol) * 4096u;
  const unsigned long long own = 0x0003000300030003ull << (2 * rk);
#define SA(b, h) (((b) * 2 + (h)) * HTB)
#define SB(b, h) ((4 + (b) * 2 + (h)) * HTB)
#define STAGE(bufoff, goff) asm volatile("s_add_u32 m0, %3, %4\n\ts_nop 0\n\tbuffer_load_dwordx4 %0, %1, %2 offen sc0 sc1 lds\n\t" \
    "buffer_load_dwordx4 %0, %1, %2 offen offset:1024 sc0 sc1 lds" :: "v"(voff), "s"(srd), "s"(goff), "s"(ldsw), "i"(bufoff) : "scc", "memory")
#define FLAG_FETCH(ko, sl) asm volatile("s_mov_b32 m0, %3\n\ts_nop 0\n\tbuffer_load_dword %0, %1, %2 offen sc0 sc1 lds" :: "v"(vf), "s"(srd), "s"((ko) >> 7), "s"(lds0 + (sl)) : "memory")
#define LDA_(dst, base) do { _Pragma("unroll") for (int m = 0; m < 4; ++m) _Pragma("unroll") for (int k = 0; k < 2; ++k) dst[m][k] = *(const LAS i32x4*)(lds + (base) + m * 2048 + k * 8192); } while (0)
#define LDB_(dst, base) do { _Pragma("unroll") for (int n = 0; n < 2; ++n) _Pragma("unroll") for (int k = 0; k < 2; ++k) dst[n][k] = *(const LAS i32x4*)(lds + (base) + n * 1024 + k * 8192); } while (0)
#define LDA(dst, b, h) LDA_(dst, SA(b, h) + aoff)
#define LDB(dst, b, h) LDB_(dst, SA(b, h) + boff)
#define MFMA2(c, a0, b0, a1, b1, Z) do { \
    if (Z) asm volatile("v_mfma_i32_16x16x64_i8 %0, %1, %2, 0.15915494\n\tv_mfma_i32_16x16x64_i8 %0, %3, %4, %0" : "+v"(c) : "v"(a0), "v"(b0), "v"(a1), "v"(b1)); \
    else asm volatile("v_mfma_i32_16x16x64_i8 %0, %1, %2, %0\n\tv_mfma_i32_16x16x64_i8 %0, %3, %4, %0" : "+v"(c) : "v"(a0), "v"(b0), "v"(a1), "v"(b1)); } while (0)
#define MMA2(ai, Z, PARK) do { __builtin_amdgcn_s_setprio(1); \
    _Pragma("unroll") for (int i = 0; i < 16; ++i) { \
      const int k = i & 1, m = i >> 2, c = (m & 1) ? 3 - (i & 3) : (i & 3); \
      if ((PARK) == 1 && m >= 2) asm volatile("v_mfma_i32_16x16x64_i8 %0, %1, %2, 0.15915494\n\tv_mfma_i32_16x16x64_i8 %0, %3, %4, %0" \
          : "=&v"(park[m & 1][c]) : "v"(At[m][k]), "v"(c < 2 ? B0[c & 1][k] : B1[c & 1][k]), "v"(At[m][k ^ 1]), "v"(c < 2 ? B0[c & 1][k ^ 1] : B1[c & 1][k ^ 1])); \
      else if ((PARK) == 2 && m >= 2) asm volatile("v_mfma_i32_16x16x64_i8 %0, %1, %2, %5\n\tv_mfma_i32_16x16x64_i8 %0, %3, %4, %0" \
          : "+v"(acc[ai][c >> 1][m][c & 1]) : "v"(At[m][k]), "v"(c < 2 ? B0[c & 1][k] : B1[c & 1][k]), "v"(At[m][k ^ 1]), "v"(c < 2 ? B0[c & 1][k ^ 1] : B1[c & 1][k ^ 1]), "v"(park[m & 1][c])); \
      else MFMA2(acc[ai][c >> 1][m][c & 1], At[m][k], (c < 2 ? B0[c & 1][k] : B1[c & 1][k]), At[m][k ^ 1], (c < 2 ? B0[c & 1][k ^ 1] : B1[c & 1][k ^ 1]), Z); } \
    __builtin_amdgcn_s_setprio(0); } while (0)
#define WAIT_V(n) asm volatile("s_waitcnt vmcnt(" #n ")" ::: "memory")
#define WAIT_L(n) do { asm volatile("" ::: "memory"); __builtin_amdgcn_s_waitcnt(0xC07F); asm volatile("" ::: "memory"); } while (0)
#define BAR __builtin_amdgcn_s_barrier()
#define SCHED __builtin_amdgcn_sched_barrier(0)
#define PIN do { asm volatile("" ::: "memory"); SCHED; } while (0)
#define DEQ(a) fmaxf((float)((a) - BIAS) * scale, 0.f)
#define EPI_ST(s0, s1) do { \
    _Pragma("unroll") for (int s = (s0); s < (s1); ++s) { const int ai = s >> 4, m = (s >> 2) & 3, j = s & 3; f32x4 v; \
      v[0] = DEQ(acc[ai][0][m][0][j]); v[1] = DEQ(acc[ai][0][m][1][j]); v[2] = DEQ(acc[ai][1][m][0][j]); v[3] = DEQ(acc[ai][1][m][1][j]); \
      asm volatile("buffer_store_dwordx4 %0, %1, %2, %3 offen nt\n\ts_nop 1" :: "v"(v), "v"(coff), "s"(crs), "s"((ai * 128 + m * 16 + j) * (N_ * 4)) : "memory"); } } while (0)
  auto flag_off = [&](int rnd) -> unsigned { const int l = lane_id();
    return (unsigned)WS_FLAG + (unsigned)(((l & 16) ? 16 + 15 * rnd + slot : pm) * 64 + (l & 15) * 4); };
  auto nextk = [&](unsigned k) -> unsigned { k += 2 * KTS; return k < 32 * KTS ? k : 0u; };

  i32x4 acc[2][2][4][2];
  i32x4 At[4][2], B0[2][2], B1[2][2], park[2][4];
  unsigned k0 = (unsigned)start_of(slot) * KTS, k1 = nextk(k0);
  unsigned vf = flag_off(0);
  if (wid == 0) {
    wait_words(vf + (k0 >> 7), 1, 0u, own);
    wait_words((unsigned)WS_LFLAG + lane_id() * 4u, 4, 256u, 0ull);
    FLAG_FETCH(k1, 0u);
  }
  asm volatile("s_waitcnt vmcnt(0)" ::: "memory");
  BAR;
  { const unsigned a = cA + k0, b = cW + k0;
    STAGE(SB(0, 0), b); STAGE(SB(0, 1), b + GH); STAGE(SA(0, 0), a); STAGE(SA(0, 1), a + GH);
    if (wr == 1) BAR;
    WAIT_V(2); BAR;
    STAGE(SB(1, 0), b + KTS); STAGE(SA(1, 0), a + KTS); STAGE(SB(1, 1), b + GH + KTS);
    WAIT_V(0); BAR; PIN; }
#pragma unroll
  for (int a = 0; a < 2; ++a)
#pragma unroll
    for (int b = 0; b < 2; ++b)
#pragma unroll
      for (int m = 0; m < 4; ++m)
#pragma unroll
        for (int n = 0; n < 2; ++n) asm volatile("" : "=v"(acc[a][b][m][n]));
  auto mk_rs = [](const float* b, unsigned n) { return (u32x4){(unsigned)(size_t)b, (unsigned)((size_t)b >> 32) & 0xffffu, n, 0x00020000u}; };
  u32x4 crs = mk_rs(p.out, 0u);
  unsigned fsl = 0;
#define KT(b, ah, a2, b2, Z, PARK, V1, V2, e0, e1, e2, FL) do { \
    unsigned fv = 0; \
    LDB(B0, b, 0); LDB(B1, b, 1); SCHED; LDA(At, b, 0); STAGE(SA((b) ^ 1, 1), ah); \
    if ((FL) && wid == 0) { fv = *(const LAS unsigned*)(lds + fsl + lane_id() * 4); if (newf) vf = flag_off(un); FLAG_FETCH(k2, fsl ^ 256u); } \
    PIN; EPI_ST(e0, e1); PIN; \
    WAIT_V(V1); WAIT_L(0); \
    if ((FL) && wid == 0) { if ((__ballot((int)(fv - call) >= 0) | own) != ~0ull) wait_words(flag_off(c1) + (k1 >> 7), 1, 0u, own); \
      if ((FL) == 2 && tohalf) { const int l = lane_id(); wait_words((unsigned)WS_FLAG + (unsigned)((76 + hcol) * 64 + (l >> 4) * (int)(KTS >> 6) + (l & 15) * 4), 4, KTS >> 4, own); } \
    } \
    BAR; MMA2(0, Z, 0); BAR; PIN; \
    LDA(At, b, 1); STAGE(SB(b, 0), b2); STAGE(SB(b, 1), (b2) + GH); STAGE(SA(b, 0), a2); PIN; EPI_ST(e1, e2); PIN; \
    WAIT_V(V2); WAIT_L(0); BAR; MMA2(1, Z, PARK); BAR; PIN; } while (0)
#define PAIR(C1, NEWF, Z, Va, Vb, Vc, Vd, e0, e1, e2, e3, e4, FL) do { \
    const int c1 = (C1); const bool newf = (NEWF); const unsigned k2 = nextk(k1); \
    unsigned a1 = cA + k1, b1 = cW + (unsigned)c1 * (15u * 4096u) + k1, ah1 = a1 + GH; \
    if ((FL) == 2 && tohalf) { a1 = hA; b1 = hB; ah1 = hB + 2 * KTS; }     \
    KT(0, cA + k0 + KTS + GH, a1, b1, Z, Z, Va, Vb, e0, e1, e2, FL); \
    KT(1, ah1, a1 + KTS, b1 + KTS, 0, 2 * (Z), Vc, Vd, e2, e3, e4, 0); \
    k0 = k1; k1 = k2; fsl ^= 256u; } while (0)
  for (int ui = 0;; ++ui) {
    const bool has_next = ui < 3, tohalf = !has_next && has_half;
    const int un = has_next ? ui + 1 : ui;
    PAIR(ui, false, 1, 24, 32, 28, 24, 8, 16, 24, 28, 32, 1);
    crs = mk_rs(p.out + (size_t)pm * 256 * N_ + (ui * 15 + slot) * 256, 256u * N_ * 4u);
    for (int t = 1; t < 15; ++t) PAIR(ui, t == 14, 0, 8, 8, 8, 8, 0, 0, 0, 0, 0, 1);
    PAIR(un, false, 0, 8, 8, 8, 16, 0, 0, 0, 0, 8, 2);
    if (!has_next) break;
  }
  if (has_half) {
    int aoff2 = 8 * HTB + aoff; asm volatile("" : "+v"(aoff2));
    const int boff2 = boff - 4 * HTB;
#define HST(j, w, g) do { if ((j) == 0) { if ((w) == 0) STAGE(SA(0, 1), g); else if ((w) == 1) STAGE(SA(1, 1), g); else STAGE(8 * HTB, g); } \
      else { if ((w) == 0) STAGE(SB(((j) + 1) & 1, 0), g); else if ((w) == 1) STAGE(SB(((j) + 1) & 1, 1), g); else STAGE(SA(((j) + 1) & 1, 0), g); } } while (0)
#define HKT(j, tn, Z, V, e0, e1) do { \
      const unsigned a = hA + (unsigned)(tn) * KTS, b = hB + (unsigned)(tn) * KTS; \
      if ((j) < 2) LDB(B0, j, 0); else LDB_(B0, SA(0, 1) + boff2); \
      HST(j, 0, b); \
      if ((j) < 2) LDB(B1, j, 1); else LDB_(B1, SA(1, 1) + boff2); \
      HST(j, 1, b + GH); \
      SCHED; if ((j) < 2) LDA(At, j, 0); else LDA_(At, aoff2); \
      HST(j, 2, a); \
      PIN; EPI_ST(e0, e1); PIN; \
      WAIT_V(V); WAIT_L(0); BAR; MMA2(0, Z, 0); BAR; PIN; } while (0)
    HKT(0, 2, 1, 22, 8, 16); HKT(1, 3, 0, 18, 16, 20); HKT(2, 4, 0, 14, 20, 24);
    HKT(0, 5, 0, 14, 24, 28); HKT(1, 6, 0, 14, 28, 32); HKT(2, 7, 0, 10, 0, 0);
    crs = mk_rs(p.out + (size_t)(pm * 256 + hrow * 128) * N_ + (60 + hcol) * 256, 128u * N_ * 4u);
    for (int t = 6; t < 30; t += 3) { HKT(0, t + 2, 0, 6, 0, 0); HKT(1, t + 3, 0, 6, 0, 0); HKT(2, t + 4, 0, 6, 0, 0); }
    HKT(0, 31, 0, 6, 0, 0); HKT(1, 31, 0, 6, 0, 0);
  }
  const unsigned gave_up = __hip_atomic_load((unsigned*)(myws + WS_TMO), __ATOMIC_RELAXED, __HIP_MEMORY_SCOPE_AGENT);
  asm volatile("s_nop 7" ::: "memory"); SCHED;
  if (has_half) EPI_ST(0, 4); else EPI_ST(8, 12);
  if (wr == 0) BAR;
  if (has_half) EPI_ST(4, 16); else EPI_ST(12, 32);
  PIN;
  if (__builtin_amdgcn_readfirstlane((int)gave_up) == (int)call) {
    WAIT_V(0);
    const float q = __builtin_nanf(""); const f32x4 v = {q, q, q, q};
    for (int ui = 0; ui < (has_half ? 5 : 4); ++ui) {
      crs = ui < 4 ? mk_rs(p.out + (size_t)pm * 256 * N_ + (ui * 15 + slot) * 256, 256u * N_ * 4u)
                   : mk_rs(p.out + (size_t)(pm * 256 + hrow * 128) * N_ + (60 + hcol) * 256, 128u * N_ * 4u);
      for (int s = 0; s < 32; ++s)
        asm volatile("buffer_store_dwordx4 %0, %1, %2, %3 offen\n\ts_nop 1" :: "v"(v), "v"(coff), "s"(crs), "s"(((s >> 4) * 128 + ((s >> 2) & 3) * 16 + (s & 3)) * (N_ * 4)) : "memory");
    }
  }
  WAIT_V(0);
}

extern "C" void kernel_launch(int rank, int nranks, ncclComm_t comm, hipStream_t stream,
                              void* const* d_in, const int* in_sizes, int n_in,
                              void* d_out, int out_size, const int* in_dtypes, int out_dtype,
                              void* const* d_out_all, void* const* d_ws_all, size_t ws_bytes) {
  (void)comm; (void)in_dtypes; (void)out_dtype; (void)d_out_all;
  if (nranks != NR || (unsigned)rank >= (unsigned)NR || n_in != 4 || ws_bytes < WS_NEED || out_size != M_ * N_ ||
      in_sizes[0] != M_ * KS || in_sizes[1] != KS * N_) {
    fprintf(stderr, "kernel_launch: unexpected configuration\n");
    return;
  }
  Params p;
  for (int i = 0; i < NR; ++i) p.ws[i] = (char*)d_ws_all[i];
  p.x = (const signed char*)d_in[0]; p.w = (const signed char*)d_in[1];
  p.sx = (const float*)d_in[2]; p.sw = (const float*)d_in[3];
  p.out = (float*)d_out; p.rank = rank;
  hipLaunchKernelGGL(fused_kernel, dim3(GRID), dim3(512), 0, stream, p);
}
```

```cpp
#include <hip/hip_runtime.h>
#include <rccl/rccl.h>
#include <cstdio>

#define LAS __attribute__((address_space(3)))
#define GAS __attribute__((address_space(1)))
typedef int i32x4 __attribute__((ext_vector_type(4)));
typedef unsigned u32x4 __attribute__((ext_vector_type(4)));
typedef float f32x4 __attribute__((ext_vector_type(4)));

namespace {
constexpr int M_ = 4096, N_ = 16384, K_ = 4096, NR = 8;
constexpr int KS = K_ / NR;
constexpr int HTB = 16384;
constexpr int GH = 2048;
constexpr int LDS_OP = 512;
constexpr int LDS_BYTES = LDS_OP + 9 * HTB;
constexpr int BIAS = 0x3e22f983;
constexpr int NAG = 16;
constexpr int GRID = 256;
constexpr int TD = 2;
constexpr int DGAP = 4;
constexpr int DSIDE = 4;
constexpr int TSEL = 52;
constexpr int SSLOT = 8, SROUND = 2;
constexpr size_t WS_CTR = 0;
constexpr size_t WS_TMO = 128 * 1024;
constexpr size_t WS_QUEUE = 128 * 1024 + 256;
constexpr size_t WS_LFLAG = 128 * 1024 + 1024;
constexpr size_t WS_OFLAG = 128 * 1024 + 3072;
constexpr size_t WS_DONE = 128 * 1024 + 4096;
constexpr size_t WS_FLAG = 256 * 1024;
constexpr size_t WS_DATA = 1024 * 1024;
constexpr int KSTR = 80;
constexpr int NUNITS = 32 * KSTR;
constexpr unsigned KTS = KSTR * 4096u;
constexpr unsigned SRCB = 32 * KTS + 256u;
constexpr size_t PARB = 81ull << 20;
static_assert(PARB >= 8ull * SRCB, "parity region");
constexpr size_t WS_NEED = WS_DATA + 2 * PARB;
constexpr unsigned DEADLINE = 50000000u;

struct Params {
  char* ws[NR];
  const signed char* x; const signed char* w; const float* sx; const float* sw;
  float* out; int rank;
  char* outs[NR];
};

__device__ __forceinline__ unsigned perm(unsigned hi, unsigned lo, unsigned sel) { return __builtin_amdgcn_perm(hi, lo, sel); }

__device__ __forceinline__ int start_of(int slot) { return slot == 0 ? 0 : 2 * (slot & 7) + DGAP + (slot < 7 ? DSIDE : 0); }
__device__ __forceinline__ unsigned slot_mask(int tau) {
  unsigned m = 0;
#pragma unroll
  for (int j = 0; j < 15; ++j) m |= ((unsigned)(tau - start_of(j)) < 128u ? 1u : 0u) << j;
  return m;
}
__device__ __forceinline__ int step_units(int tau, unsigned mask) { return (tau < 32 ? 16 : 0) + __builtin_popcount(mask); }
constexpr int TEND = 14 + DSIDE + DGAP + 128;
static_assert(TD <= 4 && TD % 2 == 0 && DGAP % 2 == 0 && DSIDE % 2 == 0 && TSEL % 2 == 0 && TEND < 160 && TSEL < TEND, "schedule");
}

__global__ __launch_bounds__(512, 2) void fused_kernel(const Params p) {
  __shared__ __attribute__((aligned(16))) unsigned char shm[LDS_BYTES];
  LAS unsigned char* lds = (LAS unsigned char*)shm;
  const int tid = threadIdx.x, wid = __builtin_amdgcn_readfirstlane(tid >> 6), lane = tid & 63;
  const int rk = p.rank & 7;
  char* const myws = p.ws[rk];
  unsigned call;
  size_t par;

  {
    const int b = ((int)blockIdx.x + GRID - NAG) & (GRID - 1);
    const bool hasx = wid >= 6;
    const int ex = ((wid & 1) << 8) + b, xkt = ex >> 4, xb = ex & 15;
    u32x4 vx[4] = {};
    if (hasx) {
      const signed char* s = p.x + (size_t)xb * 256 * KS + xkt * 16;
#pragma unroll
      for (int c = 0; c < 4; ++c) vx[c] = *(const GAS u32x4*)(s + (unsigned)((lane + c * 64) * KS));
    }
    const int ew = (wid << 8) + b, wkt = ew >> 6, col = ew & 63;
    unsigned q[16];
    { const signed char* s = p.w + (size_t)wkt * 16 * N_ + col * 256;
#pragma unroll
      for (int j = 0; j < 16; ++j) q[j] = *(const GAS unsigned*)(s + (unsigned)(lane * 4 + j * N_)); }
    asm volatile("" ::: "memory");
    unsigned tkt = 0;
    if (lane == 0) tkt = __hip_atomic_fetch_add((unsigned*)(myws + WS_CTR + (size_t)(blockIdx.x * 8 + wid) * 64), 1u, __ATOMIC_RELAXED, __HIP_MEMORY_SCOPE_AGENT);
    call = (unsigned)__builtin_amdgcn_readfirstlane((int)tkt) + 1u;
    par = WS_DATA + (size_t)(call & 1u) * PARB;
    u32x4 vw[4];
#pragma unroll
    for (int t = 0; t < 4; ++t) {
      const unsigned t0 = perm(q[4 * t + 1], q[4 * t], 0x05010400u), t1 = perm(q[4 * t + 1], q[4 * t], 0x07030602u);
      const unsigned t2 = perm(q[4 * t + 3], q[4 * t + 2], 0x05010400u), t3 = perm(q[4 * t + 3], q[4 * t + 2], 0x07030602u);
      vw[0][t] = perm(t2, t0, 0x05040100u); vw[1][t] = perm(t2, t0, 0x07060302u);
      vw[2][t] = perm(t3, t1, 0x05040100u); vw[3][t] = perm(t3, t1, 0x07060302u);
    }
    const unsigned uw = (unsigned)(wkt * KSTR + 16 + col), ux = (unsigned)(xkt * KSTR + xb);
    const unsigned lo = (unsigned)rk * SRCB + (unsigned)lane * 16u;
#define PUT(d, o, v) asm volatile("global_store_dwordx4 %0, %1, %5 sc0 sc1\n\tglobal_store_dwordx4 %0, %2, %5 offset:1024 sc0 sc1\n\t" \
      "global_store_dwordx4 %0, %3, %5 offset:2048 sc0 sc1\n\tglobal_store_dwordx4 %0, %4, %5 offset:3072 sc0 sc1\n\ts_nop 1" \
      :: "v"(o), "v"(v[0]), "v"(v[1]), "v"(v[2]), "v"(v[3]), "s"(d) : "memory")
    PUT(myws + par, lo + uw * 4096u, vw);
    if (hasx) PUT(myws + par, lo + ux * 4096u, vx);
    const bool dw = wkt < TD && col < 15 && start_of(col) <= wkt, dx = hasx && xkt < TD;
    asm volatile("s_waitcnt vmcnt(0)" ::: "memory");
    __builtin_amdgcn_s_barrier();
    if (tid == 64) __hip_atomic_store((unsigned*)(myws + WS_LFLAG) + blockIdx.x, call, __ATOMIC_RELAXED, __HIP_MEMORY_SCOPE_AGENT);
    if (dw || dx) {
      if (dx) { vw[0] = vx[0]; vw[1] = vx[1]; vw[2] = vx[2]; vw[3] = vx[3]; }
      const unsigned o = lo + (dx ? ux : uw) * 4096u; const int dkt = dx ? xkt : wkt;
      __builtin_amdgcn_s_sleep(1);
#pragma unroll
      for (int i = 1; i < NR; ++i) PUT(p.ws[(rk + 1 + (i + b) % (NR - 1)) & 7] + par, o, vw);
      asm volatile("s_waitcnt vmcnt(0)" ::: "memory");
      if (lane < NR) __hip_atomic_store((unsigned*)(p.ws[lane & 7] + WS_FLAG + rk * 8 + (size_t)(((dkt >> 1) * KSTR + (dx ? xb : 16 + col)) * 64 + (dkt & 1) * 4)), call, __ATOMIC_RELAXED, __HIP_MEMORY_SCOPE_SYSTEM);
    }
  }
  auto lane_id = [&]() -> int { int l; asm volatile("v_mbcnt_lo_u32_b32 %0, -1, 0\n\tv_mbcnt_hi_u32_b32 %0, -1, %0" : "=v"(l)); return l; };
  auto wait_words = [&](unsigned off, int nw, unsigned stride, unsigned long long own) -> bool {
    const unsigned t0 = (unsigned)__builtin_amdgcn_s_memrealtime();
    for (;;) {
      unsigned long long ok = ~0ull;
      unsigned v[4];
#pragma unroll
      for (int i = 0; i < 4; ++i) if (i < nw) v[i] = __hip_atomic_load((const unsigned*)(myws + (off + i * stride)), __ATOMIC_RELAXED, __HIP_MEMORY_SCOPE_SYSTEM);
#pragma unroll
      for (int i = 0; i < 4; ++i) if (i < nw) ok &= __ballot((int)(v[i] - call) >= 0);
      if ((ok | own) == ~0ull) return true;
      unsigned to; asm volatile("v_mov_b32 %0, %1" : "=v"(to) : "i"((unsigned)WS_TMO));
      unsigned* const tp = (unsigned*)(myws + to);
      const unsigned now = (unsigned)__builtin_amdgcn_s_memrealtime();
      if (__builtin_amdgcn_readfirstlane((int)__hip_atomic_load(tp, __ATOMIC_RELAXED, __HIP_MEMORY_SCOPE_AGENT)) == (int)call || now - t0 > DEADLINE) {
        unsigned cv; asm volatile("v_mov_b32 %0, %1" : "=v"(cv) : "s"(call));
        __hip_atomic_store(tp, cv, __ATOMIC_RELAXED, __HIP_MEMORY_SCOPE_AGENT);
        return false;
      }
      __builtin_amdgcn_s_sleep(4);
    }
  };

  if (blockIdx.x < NAG) {
    if (wid > 1) return;
    const int lane = lane_id();
    char* const fl_base = p.ws[lane & 7] + WS_FLAG + rk * 8;
    int ndirect = 0;
    for (int t = 0; t < TD; t += 2) ndirect += step_units(t, slot_mask(t));
    const unsigned npush = (unsigned)(76 * 16 - ndirect);
    const unsigned qbase = (call - 1u) * (npush + (unsigned)NAG);
    const char* const qctr = myws + WS_QUEUE;
    const char* const src = myws + par + (size_t)rk * SRCB;
    if (wid == 1) {
      unsigned thr = 0;
      for (int t = TD; t < TSEL; t += 2) thr += (unsigned)step_units(t, slot_mask(t));
      const unsigned t0 = (unsigned)__builtin_amdgcn_s_memrealtime();
      for (;;) {
        if ((unsigned)__builtin_amdgcn_readfirstlane((int)__hip_atomic_load((const unsigned*)qctr, __ATOMIC_RELAXED, __HIP_MEMORY_SCOPE_AGENT)) - qbase >= thr) break;
        if (__builtin_amdgcn_readfirstlane((int)__hip_atomic_load((unsigned*)(myws + WS_TMO), __ATOMIC_RELAXED, __HIP_MEMORY_SCOPE_AGENT)) == (int)call
            || (unsigned)__builtin_amdgcn_s_memrealtime() - t0 > DEADLINE) {
          __hip_atomic_store((unsigned*)(myws + WS_TMO), call, __ATOMIC_RELAXED, __HIP_MEMORY_SCOPE_AGENT);
          return; }
        __builtin_amdgcn_s_sleep(64);
      }
      for (int i = 0; i < 4; ++i) {
        const int g = (i + (int)blockIdx.x) & 3;
        const unsigned o = (unsigned)(2 * (int)blockIdx.x * KSTR + 76 + g) * 4096u + (unsigned)lane * 16u, o2 = o + KTS;
        u32x4 v[4], w[4];
        asm volatile("global_load_dwordx4 %0, %8, %10 sc0 sc1\n\tglobal_load_dwordx4 %1, %8, %10 offset:1024 sc0 sc1\n\t"
                     "global_load_dwordx4 %2, %8, %10 offset:2048 sc0 sc1\n\tglobal_load_dwordx4 %3, %8, %10 offset:3072 sc0 sc1\n\t"
                     "global_load_dwordx4 %4, %9, %10 sc0 sc1\n\tglobal_load_dwordx4 %5, %9, %10 offset:1024 sc0 sc1\n\t"
                     "global_load_dwordx4 %6, %9, %10 offset:2048 sc0 sc1\n\tglobal_load_dwordx4 %7, %9, %10 offset:3072 sc0 sc1\n\ts_waitcnt vmcnt(0)"
                     : "=&v"(v[0]), "=&v"(v[1]), "=&v"(v[2]), "=&v"(v[3]), "=&v"(w[0]), "=&v"(w[1]), "=&v"(w[2]), "=&v"(w[3]) : "v"(o), "v"(o2), "s"(src) : "memory");
        if (2 * g != rk) { PUT(p.ws[2 * g] + par + (size_t)rk * SRCB, o, v); PUT(p.ws[2 * g] + par + (size_t)rk * SRCB, o2, w); }
        if (2 * g + 1 != rk) { PUT(p.ws[2 * g + 1] + par + (size_t)rk * SRCB, o, v); PUT(p.ws[2 * g + 1] + par + (size_t)rk * SRCB, o2, w); }
      }
      asm volatile("s_waitcnt vmcnt(0)" ::: "memory");
      if (lane < NR) __hip_atomic_store((unsigned long long*)(fl_base + (size_t)((int)blockIdx.x * KSTR + 76 + (lane >> 1)) * 64u),
                                        ((unsigned long long)call << 32) | call, __ATOMIC_RELAXED, __HIP_MEMORY_SCOPE_SYSTEM);
      return;
    }
    char* dst[NR - 1];
#pragma unroll
    for (int q = 0; q < NR - 1; ++q) dst[q] = p.ws[(rk + 1 + (q + (int)blockIdx.x) % (NR - 1)) & 7] + par + (size_t)rk * SRCB;
    const unsigned l16 = (unsigned)lane * 16u, vzero = 0u, vone = 1u;
    const unsigned long long call2 = ((unsigned long long)call << 32) | call;
    int tau = TD, base = 0;
    unsigned mask = slot_mask(TD); int cnt = step_units(TD, mask);
    auto decode = [&](int idx) -> unsigned {
      while (idx >= base + cnt && tau < TEND - 2) { base += cnt; tau += 2; mask = slot_mask(tau); cnt = step_units(tau, mask); }
      int r = idx - base; const int kp = (tau & 31) >> 1;
      if (tau < 32) { if (r < 16) return (unsigned)(kp * KSTR + r); r -= 16; }
      unsigned m = mask; for (int i = 0; i < r; ++i) m &= m - 1u;
      const int slot = __builtin_ctz(m | 0x4000u);
      const unsigned u = (unsigned)(kp * KSTR + 16 + 15 * ((tau - start_of(slot)) >> 5) + slot);
      return u < (unsigned)(NUNITS / 2) ? u : 0u;
    };
    auto data_off = [&](unsigned u) -> unsigned { return (u + (u / KSTR) * KSTR) * 4096u; };
#define CLB_A "v224", "v225", "v226", "v227", "v228", "v229", "v230", "v231", "v232", "v233", "v234", "v235", "v236", "v237", "v238", "v239"
#define CLB_B "v240", "v241", "v242", "v243", "v244", "v245", "v246", "v247", "v248", "v249", "v250", "v251", "v252", "v253", "v254", "v255"
#define CLB "memory", "v223", CLB_A, CLB_B
#define DRAW(ctr) asm volatile("s_mov_b64 exec, 1\n\tglobal_atomic_add v223, %0, %1, %2 sc0\n\ts_mov_b64 exec, -1" :: "v"(vzero), "v"(vone), "s"(ctr) : CLB)
#define LOADU(R, o) asm volatile("global_load_dwordx4 v[%c2:%c2+3], %0, %1 sc0 sc1\n\tglobal_load_dwordx4 v[%c2+4:%c2+7], %0, %1 offset:1024 sc0 sc1\n\t" \
                   "global_load_dwordx4 v[%c2+8:%c2+11], %0, %1 offset:2048 sc0 sc1\n\tglobal_load_dwordx4 v[%c2+12:%c2+15], %0, %1 offset:3072 sc0 sc1" \
                   :: "v"((o) + l16), "s"(src), "i"(R) : CLB)
#define ST2(R, c, q) asm volatile("global_store_dwordx4 %0, v[%c2:%c2+3], %1 offset:%c3 sc0 sc1\n\tglobal_store_dwordx4 %0, v[%c2+4:%c2+7], %1 offset:%c3+1024 sc0 sc1" \
                   :: "v"(_o), "s"(dst[q]), "i"((R) + 4 * (c)), "i"((c) * 1024) : CLB)
#define ST14(R, c) ST2(R, c, 0); ST2(R, c, 1); ST2(R, c, 2); ST2(R, c, 3); ST2(R, c, 4); ST2(R, c, 5); ST2(R, c, 6)
#define STOREU(R, o) do { const unsigned _o = (o) + l16; ST14(R, 0); ST14(R, 2); } while (0)
#define FLAGU(u) do { char* _f = fl_base + (size_t)(u) * 64u; \
      asm volatile("s_mov_b64 exec, 0xff\n\tglobal_store_dwordx2 %0, %1, off sc0 sc1\n\ts_mov_b64 exec, -1" :: "v"(_f), "v"(call2) : CLB); } while (0)
#define READ_TK(tk) asm volatile("v_readfirstlane_b32 %0, v223" : "=s"(tk) :: CLB)
#define WAIT28() asm volatile("s_waitcnt vmcnt(28)" ::: CLB)
    unsigned tk;
    DRAW(qctr); asm volatile("s_waitcnt vmcnt(0)" ::: CLB); READ_TK(tk);
    unsigned idx = tk - qbase;
    if (!wait_words((unsigned)WS_LFLAG + lane * 4u, 4, 256u, 0ull)) return;
    if (idx < npush) {
      unsigned cur = decode((int)idx), prev = NUNITS / 2;
      unsigned o = data_off(cur);
      LOADU(224, o); DRAW(qctr);
      asm volatile("s_waitcnt vmcnt(0)" ::: CLB);
      for (;;) {
        LOADU(240, o + KTS);
        STOREU(224, o);
        READ_TK(tk); idx = tk - qbase;
        const bool more = idx < npush;
        unsigned nxt = cur, on = o;
        if (more) { nxt = decode((int)idx); on = data_off(nxt); }
        WAIT28();
        FLAGU(prev);
        if (more) { LOADU(224, on); DRAW(qctr); }
        STOREU(240, o + KTS);
        if (!more) break;
        WAIT28();
        prev = cur; cur = nxt; o = on;
      }
      asm volatile("s_waitcnt vmcnt(0)" ::: CLB);
      FLAGU(cur);
    }
    {
      constexpr unsigned nfw = 8 * 32, ROW = N_ * 4u;
      const unsigned qb2 = (call - 1u) * (nfw + (unsigned)NAG);
      const char* const osrc = (const char*)p.out;
#pragma unroll
      for (int q = 0; q < NR - 1; ++q) dst[q] = p.outs[(rk + 1 + (q + (int)blockIdx.x) % (NR - 1)) & 7];
      const unsigned obase = (unsigned)(rk & 1) * 2048u * ROW + (unsigned)(60 + (rk >> 1)) * 1024u;
#define OLD(R, o) asm volatile("global_load_dwordx4 v[%c5:%c5+3], %0, %4 sc0 sc1\n\tglobal_load_dwordx4 v[%c5+4:%c5+7], %1, %4 sc0 sc1\n\t" \
        "global_load_dwordx4 v[%c5+8:%c5+11], %2, %4 sc0 sc1\n\tglobal_load_dwordx4 v[%c5+12:%c5+15], %3, %4 sc0 sc1" \
        :: "v"((o) + l16), "v"((o) + l16 + ROW), "v"((o) + l16 + 2 * ROW), "v"((o) + l16 + 3 * ROW), "s"(osrc), "i"(R) : CLB)
#define OS2(R, c, q) asm volatile("global_store_dwordx4 %0, v[%c3:%c3+3], %2 sc0 sc1 nt\n\tglobal_store_dwordx4 %1, v[%c3+4:%c3+7], %2 sc0 sc1 nt" \
        :: "v"(_o + (c) * ROW), "v"(_o + (c) * ROW + ROW), "s"(dst[q]), "i"((R) + 4 * (c)) : CLB)
#define OS14(R, c) OS2(R, c, 0); OS2(R, c, 1); OS2(R, c, 2); OS2(R, c, 3); OS2(R, c, 4); OS2(R, c, 5); OS2(R, c, 6)
#define OST(R, o) do { const unsigned _o = (o) + l16; OS14(R, 0); OS14(R, 2); } while (0)
      DRAW(qctr + 128); asm volatile("s_waitcnt vmcnt(0)" ::: CLB); READ_TK(tk); idx = tk - qb2;
      if (!wait_words((unsigned)WS_OFLAG + (lane & 7) * 4u, 1, 0u, 0ull)) return;
      if (__builtin_amdgcn_readfirstlane((int)__hip_atomic_load((unsigned*)(myws + WS_TMO), __ATOMIC_RELAXED, __HIP_MEMORY_SCOPE_AGENT)) == (int)call) return;
      if (idx < nfw) {
        unsigned o = obase + idx * (8u * ROW);
        OLD(224, o); DRAW(qctr + 128); asm volatile("s_waitcnt vmcnt(0)" ::: CLB);
        for (;;) {
          OLD(240, o + 4u * ROW); OST(224, o);
          READ_TK(tk); idx = tk - qb2;
          const bool more = idx < nfw; const unsigned on = obase + idx * (8u * ROW);
          WAIT28();
          if (more) { OLD(224, on); DRAW(qctr + 128); }
          OST(240, o + 4u * ROW);
          if (!more) break;
          WAIT28();
          o = on;
        }
      }
      asm volatile("s_waitcnt vmcnt(0)" ::: CLB);
      if (lane < NR) __hip_atomic_store((unsigned*)(p.ws[lane & 7] + WS_DONE + blockIdx.x * 32 + rk * 4), call, __ATOMIC_RELAXED, __HIP_MEMORY_SCOPE_SYSTEM);
      if (!wait_words((unsigned)WS_DONE + blockIdx.x * 32u + (lane & 7) * 4u, 1, 0u, 0ull))
        *(float*)((char*)p.out + obase + blockIdx.x * 4u) = __builtin_nanf("");
    }
    asm volatile("s_waitcnt vmcnt(0)" ::: "memory");
    return;
  }

  const int wr = wid >> 2, wc = wid & 3, fr = lane & 15, fq = lane >> 4;
  int pm, slot;
  { const int b = blockIdx.x - NAG, x = b & 7, r = b >> 3, a = x >> 1, hb = x & 1;
    if (r < 28) { pm = 4 * a + (r & 3); slot = hb * 8 + (r >> 2); } else { pm = 4 * a + 2 * hb + (r - 28); slot = 7; } }
  const bool has_half = slot == 0, sharer = slot == SSLOT && (pm >> 3) == (rk & 1);
  constexpr int HCOL = 15 * SROUND + SSLOT;
  const int hblk = 8 * (rk & 1) + (pm >> 1), hrow = pm & 1;
  auto col_of = [&](int rnd) { return (sharer && rnd == SROUND) ? 60 + (rk >> 1) : 15 * rnd + slot; };
  const float scale = __builtin_bit_cast(float, __builtin_amdgcn_readfirstlane(__builtin_bit_cast(int, p.sx[0] * p.sw[0])));
  const unsigned voff = (unsigned)((wid >> 2) * 4 + fq) * SRCB + (unsigned)((wid & 3) * 256 + fr * 16);
  const unsigned lds0 = (unsigned)(size_t)lds, ldsw = lds0 + LDS_OP + (unsigned)wid * 2048u;
  int aoff = LDS_OP + wr * 1024 + lane * 16, boff = LDS_OP + 4 * HTB + wc * 2048 + lane * 16;
  asm volatile("" : "+v"(aoff), "+v"(boff));
  const unsigned coff = (unsigned)((wr * 64 + 4 * fq) * (N_ * 4) + (wc * 64 + 4 * fr) * 4);
  const u32x4 srd = {(unsigned)(size_t)myws, (unsigned)((size_t)myws >> 32) & 0xffffu, (unsigned)WS_NEED, 0x00020000u};
  const unsigned cA = (unsigned)par + (unsigned)pm * 4096u;
  const unsigned cW = (unsigned)par + 16u * 4096u;
  const unsigned hA = (unsigned)par + (unsigned)hblk * 4096u + hrow * GH, hB = cW + HCOL * 4096u;
  const unsigned long long own = 0x0003000300030003ull << (2 * rk);
#define SA(b, h) (((b) * 2 + (h)) * HTB)
#define SB(b, h) ((4 + (b) * 2 + (h)) * HTB)
#define STAGE(bufoff, goff) asm volatile("s_add_u32 m0, %3, %4\n\ts_nop 0\n\tbuffer_load_dwordx4 %0, %1, %2 offen sc0 sc1 lds\n\t" \
    "buffer_load_dwordx4 %0, %1, %2 offen offset:1024 sc0 sc1 lds" :: "v"(voff), "s"(srd), "s"(goff), "s"(ldsw), "i"(bufoff) : "scc", "memory")
#define FLAG_FETCH(ko, sl) asm volatile("s_mov_b32 m0, %3\n\ts_nop 0\n\tbuffer_load_dword %0, %1, %2 offen sc0 sc1 lds" :: "v"(vf), "s"(srd), "s"((ko) >> 7), "s"(lds0 + (sl)) : "memory")
#define LDA_(dst, base) do { _Pragma("unroll") for (int m = 0; m < 4; ++m) _Pragma("unroll") for (int k = 0; k < 2; ++k) dst[m][k] = *(const LAS i32x4*)(lds + (base) + m * 2048 + k * 8192); } while (0)
#define LDB_(dst, base) do { _Pragma("unroll") for (int n = 0; n < 2; ++n) _Pragma("unroll") for (int k = 0; k < 2; ++k) dst[n][k] = *(const LAS i32x4*)(lds + (base) + n * 1024 + k * 8192); } while (0)
#define LDA(dst, b, h) LDA_(dst, SA(b, h) + aoff)
#define LDB(dst, b, h) LDB_(dst, SA(b, h) + boff)
#define MFMA2(c, a0, b0, a1, b1, Z) do { \
    if (Z) asm volatile("v_mfma_i32_16x16x64_i8 %0, %1, %2, 0.15915494\n\tv_mfma_i32_16x16x64_i8 %0, %3, %4, %0" : "+v"(c) : "v"(a0), "v"(b0), "v"(a1), "v"(b1)); \
    else asm volatile("v_mfma_i32_16x16x64_i8 %0, %1, %2, %0\n\tv_mfma_i32_16x16x64_i8 %0, %3, %4, %0" : "+v"(c) : "v"(a0), "v"(b0), "v"(a1), "v"(b1)); } while (0)
#define MMA2(ai, Z, PARK) do { __builtin_amdgcn_s_setprio(1); \
    _Pragma("unroll") for (int i = 0; i < 16; ++i) { \
      const int k = i & 1, m = i >> 2, c = (m & 1) ? 3 - (i & 3) : (i & 3); \
      if ((PARK) == 1 && m >= 2) asm volatile("v_mfma_i32_16x16x64_i8 %0, %1, %2, 0.15915494\n\tv_mfma_i32_16x16x64_i8 %0, %3, %4, %0" \
          : "=&v"(park[m & 1][c]) : "v"(At[m][k]), "v"(c < 2 ? B0[c & 1][k] : B1[c & 1][k]), "v"(At[m][k ^ 1]), "v"(c < 2 ? B0[c & 1][k ^ 1] : B1[c & 1][k ^ 1])); \
      else if ((PARK) == 2 && m >= 2) asm volatile("v_mfma_i32_16x16x64_i8 %0, %1, %2, %5\n\tv_mfma_i32_16x16x64_i8 %0, %3, %4, %0" \
          : "+v"(acc[ai][c >> 1][m][c & 1]) : "v"(At[m][k]), "v"(c < 2 ? B0[c & 1][k] : B1[c & 1][k]), "v"(At[m][k ^ 1]), "v"(c < 2 ? B0[c & 1][k ^ 1] : B1[c & 1][k ^ 1]), "v"(park[m & 1][c])); \
      else MFMA2(acc[ai][c >> 1][m][c & 1], At[m][k], (c < 2 ? B0[c & 1][k] : B1[c & 1][k]), At[m][k ^ 1], (c < 2 ? B0[c & 1][k ^ 1] : B1[c & 1][k ^ 1]), Z); } \
    __builtin_amdgcn_s_setprio(0); } while (0)
#define WAIT_V(n) asm volatile("s_waitcnt vmcnt(" #n ")" ::: "memory")
#define WAIT_L(n) do { asm volatile("" ::: "memory"); __builtin_amdgcn_s_waitcnt(0xC07F); asm volatile("" ::: "memory"); } while (0)
#define BAR __builtin_amdgcn_s_barrier()
#define SCHED __builtin_amdgcn_sched_barrier(0)
#define PIN do { asm volatile("" ::: "memory"); SCHED; } while (0)
#define DEQ(a) fmaxf((float)((a) - BIAS) * scale, 0.f)
#define EPI_ST(s0, s1) do { \
    _Pragma("unroll") for (int s = (s0); s < (s1); ++s) { const int ai = s >> 4, m = (s >> 2) & 3, j = s & 3; f32x4 v; \
      v[0] = DEQ(acc[ai][0][m][0][j]); v[1] = DEQ(acc[ai][0][m][1][j]); v[2] = DEQ(acc[ai][1][m][0][j]); v[3] = DEQ(acc[ai][1][m][1][j]); \
      asm volatile("s_cmp_ge_u32 %4, 60\n\ts_cbranch_scc1 1f\n\tbuffer_store_dwordx4 %0, %1, %2, %3 offen nt\n\ts_branch 2f\n" \
                   "1:\n\tbuffer_store_dwordx4 %0, %1, %2, %3 offen sc0 sc1\n\ts_nop 0\n2:\n\ts_nop 0" \
                   :: "v"(v), "v"(coff), "s"(crs), "s"((ai * 128 + m * 16 + j) * (N_ * 4)), "s"(wt) : "scc", "memory"); } } while (0)
  auto flag_off = [&](int rnd) -> unsigned { const int l = lane_id();
    return (unsigned)WS_FLAG + (unsigned)(((l & 16) ? 16 + col_of(rnd) : pm) * 64 + (l & 15) * 4); };
  auto nextk = [&](unsigned k) -> unsigned { k += 2 * KTS; return k < 32 * KTS ? k : 0u; };

  i32x4 acc[2][2][4][2];
  i32x4 At[4][2], B0[2][2], B1[2][2], park[2][4];
  unsigned k0 = (unsigned)start_of(slot) * KTS, k1 = nextk(k0);
  unsigned vf = flag_off(0);
  if (wid == 0) {
    wait_words(vf + (k0 >> 7), 1, 0u, own);
    wait_words((unsigned)WS_LFLAG + lane_id() * 4u, 4, 256u, 0ull);
    FLAG_FETCH(k1, 0u);
  }
  asm volatile("s_waitcnt vmcnt(0)" ::: "memory");
  BAR;
  { const unsigned a = cA + k0, b = cW + (unsigned)slot * 4096u + k0;
    STAGE(SB(0, 0), b); STAGE(SB(0, 1), b + GH); STAGE(SA(0, 0), a); STAGE(SA(0, 1), a + GH);
    if (wr == 1) BAR;
    WAIT_V(2); BAR;
    STAGE(SB(1, 0), b + KTS); STAGE(SA(1, 0), a + KTS); STAGE(SB(1, 1), b + GH + KTS);
    WAIT_V(0); BAR; PIN; }
#pragma unroll
  for (int a = 0; a < 2; ++a)
#pragma unroll
    for (int b = 0; b < 2; ++b)
#pragma unroll
      for (int m = 0; m < 4; ++m)
#pragma unroll
        for (int n = 0; n < 2; ++n) asm volatile("" : "=v"(acc[a][b][m][n]));
  auto mk_rs = [](const float* b, unsigned n) { return (u32x4){(unsigned)(size_t)b, (unsigned)((size_t)b >> 32) & 0xffffu, n, 0x00020000u}; };
  u32x4 crs = mk_rs(p.out, 0u);
  unsigned wt = 0;
  unsigned fsl = 0;
#define KT(b, ah, a2, b2, Z, PARK, V1, V2, e0, e1, e2, FL) do { \
    unsigned fv = 0; \
    LDB(B0, b, 0); LDB(B1, b, 1); SCHED; LDA(At, b, 0); STAGE(SA((b) ^ 1, 1), ah); \
    if ((FL) && wid == 0) { fv = *(const LAS unsigned*)(lds + fsl + lane_id() * 4); if (newf) vf = flag_off(un); FLAG_FETCH(k2, fsl ^ 256u); } \
    PIN; EPI_ST(e0, e1); PIN; \
    WAIT_V(V1); WAIT_L(0); \
    if ((FL) && wid == 0) { if ((__ballot((int)(fv - call) >= 0) | own) != ~0ull) wait_words(flag_off(c1) + (k1 >> 7), 1, 0u, own); \
      if ((FL) == 2 && tohalf) { const int l = lane_id(); const unsigned fo = (unsigned)WS_FLAG + (unsigned)((l >> 4) * (int)(KTS >> 6) + (l & 15) * 4); \
        wait_words(fo + (16 + HCOL) * 64, 4, KTS >> 4, own); wait_words(fo + (unsigned)hblk * 64u, 4, KTS >> 4, own); } \
      if (pub == 2) __hip_atomic_store((unsigned*)(myws + WS_OFLAG) + (pm & 7), call, __ATOMIC_RELAXED, __HIP_MEMORY_SCOPE_AGENT); \
    } \
    BAR; MMA2(0, Z, 0); BAR; PIN; \
    LDA(At, b, 1); STAGE(SB(b, 0), b2); STAGE(SB(b, 1), (b2) + GH); STAGE(SA(b, 0), a2); PIN; EPI_ST(e1, e2); PIN; \
    WAIT_V(V2); WAIT_L(0); BAR; MMA2(1, Z, PARK); BAR; PIN; } while (0)
#define PAIR(C1, NEWF, PUB, Z, Va, Vb, Vc, Vd, e0, e1, e2, e3, e4, FL) do { \
    const int c1 = (C1); const bool newf = (NEWF); const int pub = (PUB); const unsigned k2 = nextk(k1); \
    unsigned a1 = cA + k1, b1 = cW + (unsigned)col_of(c1) * 4096u + k1, ah1 = a1 + GH; \
    if ((FL) == 2 && tohalf) { a1 = hA; b1 = hB; ah1 = hB + 2 * KTS; }     \
    KT(0, cA + k0 + KTS + GH, a1, b1, Z, Z, Va, Vb, e0, e1, e2, FL); \
    KT(1, ah1, a1 + KTS, b1 + KTS, 0, 2 * (Z), Vc, Vd, e2, e3, e4, 0); \
    k0 = k1; k1 = k2; fsl ^= 256u; } while (0)
  for (int ui = 0;; ++ui) {
    const bool has_next = ui < 3, tohalf = !has_next && has_half;
    const int un = has_next ? ui + 1 : ui;
    PAIR(ui, false, 0, 1, 24, 32, 28, 24, 8, 16, 24, 28, 32, 1);
    crs = mk_rs(p.out + (size_t)pm * 256 * N_ + col_of(ui) * 256, 256u * N_ * 4u);
    wt = (unsigned)col_of(ui);
    for (int t = 1; t < 15; ++t) PAIR(ui, t == 14, (sharer && ui == SROUND + 1) ? t : 0, 0, 8, 8, 8, 8, 0, 0, 0, 0, 0, 1);
    PAIR(un, false, 0, 0, 8, 8, 8, 16, 0, 0, 0, 0, 8, 2);
    if (!has_next) break;
  }
  if (has_half) {
    int aoff2 = 8 * HTB + aoff; asm volatile("" : "+v"(aoff2));
    const int boff2 = boff - 4 * HTB;
#define HST(j, w, g) do { if ((j) == 0) { if ((w) == 0) STAGE(SA(0, 1), g); else if ((w) == 1) STAGE(SA(1, 1), g); else STAGE(8 * HTB, g); } \
      else { if ((w) == 0) STAGE(SB(((j) + 1) & 1, 0), g); else if ((w) == 1) STAGE(SB(((j) + 1) & 1, 1), g); else STAGE(SA(((j) + 1) & 1, 0), g); } } while (0)
#define HKT(j, tn, Z, V, e0, e1) do { \
      const unsigned a = hA + (unsigned)(tn) * KTS, b = hB + (unsigned)(tn) * KTS; \
      if ((j) < 2) LDB(B0, j, 0); else LDB_(B0, SA(0, 1) + boff2); \
      HST(j, 0, b); \
      if ((j) < 2) LDB(B1, j, 1); else LDB_(B1, SA(1, 1) + boff2); \
      HST(j, 1, b + GH); \
      SCHED; if ((j) < 2) LDA(At, j, 0); else LDA_(At, aoff2); \
      HST(j, 2, a); \
      PIN; EPI_ST(e0, e1); PIN; \
      WAIT_V(V); WAIT_L(0); BAR; MMA2(0, Z, 0); BAR; PIN; } while (0)
    HKT(0, 2, 1, 22, 8, 16); HKT(1, 3, 0, 18, 16, 20); HKT(2, 4, 0, 14, 20, 24);
    HKT(0, 5, 0, 14, 24, 28); HKT(1, 6, 0, 14, 28, 32); HKT(2, 7, 0, 10, 0, 0);
    crs = mk_rs(p.out + (size_t)(hblk * 256 + hrow * 128) * N_ + HCOL * 256, 128u * N_ * 4u);
    for (int t = 6; t < 30; t += 3) { HKT(0, t + 2, 0, 6, 0, 0); HKT(1, t + 3, 0, 6, 0, 0); HKT(2, t + 4, 0, 6, 0, 0); }
    HKT(0, 31, 0, 6, 0, 0); HKT(1, 31, 0, 6, 0, 0);
  }
  const unsigned gave_up = __hip_atomic_load((unsigned*)(myws + WS_TMO), __ATOMIC_RELAXED, __HIP_MEMORY_SCOPE_AGENT);
  asm volatile("s_nop 7" ::: "memory"); SCHED;
  if (has_half) EPI_ST(0, 4); else EPI_ST(8, 12);
  if (wr == 0) BAR;
  if (has_half) EPI_ST(4, 16); else EPI_ST(12, 32);
  PIN;
  if (__builtin_amdgcn_readfirstlane((int)gave_up) == (int)call) {
    WAIT_V(0);
    const float q = __builtin_nanf(""); const f32x4 v = {q, q, q, q};
    for (int ui = 0; ui < (has_half ? 5 : 4); ++ui) {
      crs = ui < 4 ? mk_rs(p.out + (size_t)pm * 256 * N_ + col_of(ui) * 256, 256u * N_ * 4u)
                   : mk_rs(p.out + (size_t)(hblk * 256 + hrow * 128) * N_ + HCOL * 256, 128u * N_ * 4u);
      for (int s = 0; s < 32; ++s)
        asm volatile("buffer_store_dwordx4 %0, %1, %2, %3 offen\n\ts_nop 1" :: "v"(v), "v"(coff), "s"(crs), "s"(((s >> 4) * 128 + ((s >> 2) & 3) * 16 + (s & 3)) * (N_ * 4)) : "memory");
    }
  }
  WAIT_V(0);
}

extern "C" void kernel_launch(int rank, int nranks, ncclComm_t comm, hipStream_t stream,
                              void* const* d_in, const int* in_sizes, int n_in,
                              void* d_out, int out_size, const int* in_dtypes, int out_dtype,
                              void* const* d_out_all, void* const* d_ws_all, size_t ws_bytes) {
  (void)comm; (void)in_dtypes; (void)out_dtype;
  if (nranks != NR || (unsigned)rank >= (unsigned)NR || n_in != 4 || ws_bytes < WS_NEED || out_size != M_ * N_ ||
      in_sizes[0] != M_ * KS || in_sizes[1] != KS * N_) {
    fprintf(stderr, "kernel_launch: unexpected configuration\n");
    return;
  }
  Params p;
  for (int i = 0; i < NR; ++i) p.ws[i] = (char*)d_ws_all[i];
  p.x = (const signed char*)d_in[0]; p.w = (const signed char*)d_in[1];
  p.sx = (const float*)d_in[2]; p.sw = (const float*)d_in[3];
  p.out = (float*)d_out; p.rank = rank;
  for (int i = 0; i < NR; ++i) p.outs[i] = (char*)d_out_all[i];
  hipLaunchKernelGGL(fused_kernel, dim3(GRID), dim3(512), 0, stream, p);
}
```

```cpp
#include <hip/hip_runtime.h>
#include <rccl/rccl.h>
#include <cstdio>

#define LAS __attribute__((address_space(3)))
#define GAS __attribute__((address_space(1)))
typedef int i32x4 __attribute__((ext_vector_type(4)));
typedef unsigned u32x4 __attribute__((ext_vector_type(4)));
typedef float f32x4 __attribute__((ext_vector_type(4)));

namespace {
constexpr int M_ = 4096, N_ = 16384, K_ = 4096, NR = 8;
constexpr int KS = K_ / NR;
constexpr int HTB = 16384;
constexpr int GH = 2048;
constexpr int LDS_OP = 512;
constexpr int LDS_BYTES = LDS_OP + 9 * HTB;
constexpr int NAG = 16;
constexpr int GRID = 256;
constexpr int TD = 2;
constexpr int DGAP = 2;
constexpr int DSIDE = 6;
constexpr int TSEL = 52;
constexpr int SSLOT = 8, SROUND = 2;
constexpr size_t WS_CTR = 0;
constexpr size_t WS_TMO = 128 * 1024;
constexpr size_t WS_QUEUE = 128 * 1024 + 256;
constexpr size_t WS_LFLAG = 128 * 1024 + 1024;
constexpr size_t WS_OFLAG = 128 * 1024 + 3072;
constexpr size_t WS_DONE = 128 * 1024 + 4096;
constexpr size_t WS_FLAG = 256 * 1024;
constexpr size_t WS_DATA = 1024 * 1024;
constexpr int KSTR = 80;
constexpr int NUNITS = 32 * KSTR;
constexpr unsigned KTS = KSTR * 4096u;
constexpr unsigned SRCB = 32 * KTS + 256u;
constexpr size_t PARB = 81ull << 20;
static_assert(PARB >= 8ull * SRCB, "parity region");
constexpr size_t WS_NEED = WS_DATA + 2 * PARB;
constexpr unsigned DEADLINE = 50000000u;

struct Params {
  char* ws[NR];
  const signed char* x; const signed char* w; const float* sx; const float* sw;
  float* out; int rank;
  char* outs[NR];
};

__device__ __forceinline__ unsigned perm(unsigned hi, unsigned lo, unsigned sel) { return __builtin_amdgcn_perm(hi, lo, sel); }

__device__ __forceinline__ int start_of(int slot) { return slot == 0 ? 0 : 2 * (slot & 7) + DGAP + (slot < 7 ? DSIDE : 0); }
__device__ __forceinline__ unsigned slot_mask(int tau) {
  unsigned m = 0;
#pragma unroll
  for (int j = 0; j < 15; ++j) m |= ((unsigned)(tau - start_of(j)) < 128u ? 1u : 0u) << j;
  return m;
}
__device__ __forceinline__ int step_units(int tau, unsigned mask) { return (tau < 32 ? 16 : 0) + __builtin_popcount(mask); }
constexpr int TEND = 14 + DSIDE + DGAP + 128;
static_assert(TD <= 4 && TD % 2 == 0 && DGAP % 2 == 0 && DSIDE % 2 == 0 && TSEL % 2 == 0 && TEND < 160 && TSEL < TEND, "schedule");
}

__global__ __launch_bounds__(512, 2) void fused_kernel(const Params p) {
  __shared__ __attribute__((aligned(16))) unsigned char shm[LDS_BYTES];
  LAS unsigned char* lds = (LAS unsigned char*)shm;
  const int tid = threadIdx.x, wid = __builtin_amdgcn_readfirstlane(tid >> 6), lane = tid & 63;
  const int rk = p.rank & 7;
  char* const myws = p.ws[rk];
  unsigned call;
  size_t par;

  {
    const int b = ((int)blockIdx.x + GRID - NAG) & (GRID - 1);
    const bool hasx = wid >= 6;
    const int ex = ((wid & 1) << 8) + b, xkt = ex >> 4, xb = ex & 15;
    u32x4 vx[4] = {};
    if (hasx) {
      const signed char* s = p.x + (size_t)xb * 256 * KS + xkt * 16;
#pragma unroll
      for (int c = 0; c < 4; ++c) vx[c] = *(const GAS u32x4*)(s + (unsigned)((lane + c * 64) * KS));
    }
    const int ew = (wid << 8) + b, wkt = ew >> 6, col = ew & 63;
    unsigned q[16];
    { const signed char* s = p.w + (size_t)wkt * 16 * N_ + col * 256;
#pragma unroll
      for (int j = 0; j < 16; ++j) q[j] = *(const GAS unsigned*)(s + (unsigned)(lane * 4 + j * N_)); }
    asm volatile("" ::: "memory");
    unsigned tkt = 0;
    if (lane == 0) tkt = __hip_atomic_fetch_add((unsigned*)(myws + WS_CTR + (size_t)(blockIdx.x * 8 + wid) * 64), 1u, __ATOMIC_RELAXED, __HIP_MEMORY_SCOPE_AGENT);
    call = (unsigned)__builtin_amdgcn_readfirstlane((int)tkt) + 1u;
    par = WS_DATA + (size_t)(call & 1u) * PARB;
    u32x4 vw[4];
#pragma unroll
    for (int t = 0; t < 4; ++t) {
      const unsigned t0 = perm(q[4 * t + 1], q[4 * t], 0x05010400u), t1 = perm(q[4 * t + 1], q[4 * t], 0x07030602u);
      const unsigned t2 = perm(q[4 * t + 3], q[4 * t + 2], 0x05010400u), t3 = perm(q[4 * t + 3], q[4 * t + 2], 0x07030602u);
      vw[0][t] = perm(t2, t0, 0x05040100u); vw[1][t] = perm(t2, t0, 0x07060302u);
      vw[2][t] = perm(t3, t1, 0x05040100u); vw[3][t] = perm(t3, t1, 0x07060302u);
    }
    const unsigned uw = (unsigned)(wkt * KSTR + 16 + col), ux = (unsigned)(xkt * KSTR + xb);
    const unsigned lo = (unsigned)rk * SRCB + (unsigned)lane * 16u;
#define PUT(d, o, v) asm volatile("global_store_dwordx4 %0, %1, %5 sc0 sc1\n\tglobal_store_dwordx4 %0, %2, %5 offset:1024 sc0 sc1\n\t" \
      "global_store_dwordx4 %0, %3, %5 offset:2048 sc0 sc1\n\tglobal_store_dwordx4 %0, %4, %5 offset:3072 sc0 sc1\n\ts_nop 1" \
      :: "v"(o), "v"(v[0]), "v"(v[1]), "v"(v[2]), "v"(v[3]), "s"(d) : "memory")
    PUT(myws + par, lo + uw * 4096u, vw);
    if (hasx) PUT(myws + par, lo + ux * 4096u, vx);
    const bool dw = wkt < TD && col < 15 && start_of(col) <= wkt, dx = hasx && xkt < TD;
    asm volatile("s_waitcnt vmcnt(0)" ::: "memory");
    __builtin_amdgcn_s_barrier();
    if (tid == 64) __hip_atomic_store((unsigned*)(myws + WS_LFLAG) + blockIdx.x, call, __ATOMIC_RELAXED, __HIP_MEMORY_SCOPE_AGENT);
    if (dw || dx) {
      if (dx) { vw[0] = vx[0]; vw[1] = vx[1]; vw[2] = vx[2]; vw[3] = vx[3]; }
      const unsigned o = lo + (dx ? ux : uw) * 4096u; const int dkt = dx ? xkt : wkt;
      __builtin_amdgcn_s_sleep(1);
#pragma unroll
      for (int i = 1; i < NR; ++i) PUT(p.ws[(rk + 1 + (i + b) % (NR - 1)) & 7] + par, o, vw);
      asm volatile("s_waitcnt vmcnt(0)" ::: "memory");
      if (lane < NR) __hip_atomic_store((unsigned*)(p.ws[lane & 7] + WS_FLAG + rk * 8 + (size_t)(((dkt >> 1) * KSTR + (dx ? xb : 16 + col)) * 64 + (dkt & 1) * 4)), call, __ATOMIC_RELAXED, __HIP_MEMORY_SCOPE_SYSTEM);
    }
  }
  auto lane_id = [&]() -> int { int l; asm volatile("v_mbcnt_lo_u32_b32 %0, -1, 0\n\tv_mbcnt_hi_u32_b32 %0, -1, %0" : "=v"(l)); return l; };
  auto wait_words = [&](unsigned off, int nw, unsigned stride, unsigned long long own) -> bool {
    const unsigned t0 = (unsigned)__builtin_amdgcn_s_memrealtime();
    for (;;) {
      unsigned long long ok = ~0ull;
      unsigned v[4];
#pragma unroll
      for (int i = 0; i < 4; ++i) if (i < nw) v[i] = __hip_atomic_load((const unsigned*)(myws + (off + i * stride)), __ATOMIC_RELAXED, __HIP_MEMORY_SCOPE_SYSTEM);
#pragma unroll
      for (int i = 0; i < 4; ++i) if (i < nw) ok &= __ballot((int)(v[i] - call) >= 0);
      if ((ok | own) == ~0ull) return true;
      unsigned to; asm volatile("v_mov_b32 %0, %1" : "=v"(to) : "i"((unsigned)WS_TMO));
      unsigned* const tp = (unsigned*)(myws + to);
      const unsigned now = (unsigned)__builtin_amdgcn_s_memrealtime();
      if (__builtin_amdgcn_readfirstlane((int)__hip_atomic_load(tp, __ATOMIC_RELAXED, __HIP_MEMORY_SCOPE_AGENT)) == (int)call || now - t0 > DEADLINE) {
        unsigned cv; asm volatile("v_mov_b32 %0, %1" : "=v"(cv) : "s"(call));
        __hip_atomic_store(tp, cv, __ATOMIC_RELAXED, __HIP_MEMORY_SCOPE_AGENT);
        return false;
      }
      __builtin_amdgcn_s_sleep(4);
    }
  };

  if (blockIdx.x < NAG) {
    if (wid > 1) return;
    const int lane = lane_id();
    char* const fl_base = p.ws[lane & 7] + WS_FLAG + rk * 8;
    int ndirect = 0;
    for (int t = 0; t < TD; t += 2) ndirect += step_units(t, slot_mask(t));
    const unsigned npush = (unsigned)(76 * 16 - ndirect);
    const unsigned qbase = (call - 1u) * (npush + (unsigned)NAG);
    const char* const qctr = myws + WS_QUEUE;
    const char* const src = myws + par + (size_t)rk * SRCB;
    if (wid == 1) {
      unsigned thr = 0;
      for (int t = TD; t < TSEL; t += 2) thr += (unsigned)step_units(t, slot_mask(t));
      const unsigned t0 = (unsigned)__builtin_amdgcn_s_memrealtime();
      for (;;) {
        if ((unsigned)__builtin_amdgcn_readfirstlane((int)__hip_atomic_load((const unsigned*)qctr, __ATOMIC_RELAXED, __HIP_MEMORY_SCOPE_AGENT)) - qbase >= thr) break;
        if (__builtin_amdgcn_readfirstlane((int)__hip_atomic_load((unsigned*)(myws + WS_TMO), __ATOMIC_RELAXED, __HIP_MEMORY_SCOPE_AGENT)) == (int)call
            || (unsigned)__builtin_amdgcn_s_memrealtime() - t0 > DEADLINE) {
          __hip_atomic_store((unsigned*)(myws + WS_TMO), call, __ATOMIC_RELAXED, __HIP_MEMORY_SCOPE_AGENT);
          return; }
        __builtin_amdgcn_s_sleep(64);
      }
      for (int i = 0; i < 4; ++i) {
        const int g = (i + (int)blockIdx.x) & 3;
        const unsigned o = (unsigned)(2 * (int)blockIdx.x * KSTR + 76 + g) * 4096u + (unsigned)lane * 16u, o2 = o + KTS;
        u32x4 v[4], w[4];
        asm volatile("global_load_dwordx4 %0, %8, %10 sc0 sc1\n\tglobal_load_dwordx4 %1, %8, %10 offset:1024 sc0 sc1\n\t"
                     "global_load_dwordx4 %2, %8, %10 offset:2048 sc0 sc1\n\tglobal_load_dwordx4 %3, %8, %10 offset:3072 sc0 sc1\n\t"
                     "global_load_dwordx4 %4, %9, %10 sc0 sc1\n\tglobal_load_dwordx4 %5, %9, %10 offset:1024 sc0 sc1\n\t"
                     "global_load_dwordx4 %6, %9, %10 offset:2048 sc0 sc1\n\tglobal_load_dwordx4 %7, %9, %10 offset:3072 sc0 sc1\n\ts_waitcnt vmcnt(0)"
                     : "=&v"(v[0]), "=&v"(v[1]), "=&v"(v[2]), "=&v"(v[3]), "=&v"(w[0]), "=&v"(w[1]), "=&v"(w[2]), "=&v"(w[3]) : "v"(o), "v"(o2), "s"(src) : "memory");
        if (2 * g != rk) { PUT(p.ws[2 * g] + par + (size_t)rk * SRCB, o, v); PUT(p.ws[2 * g] + par + (size_t)rk * SRCB, o2, w); }
        if (2 * g + 1 != rk) { PUT(p.ws[2 * g + 1] + par + (size_t)rk * SRCB, o, v); PUT(p.ws[2 * g + 1] + par + (size_t)rk * SRCB, o2, w); }
      }
      asm volatile("s_waitcnt vmcnt(0)" ::: "memory");
      if (lane < NR) __hip_atomic_store((unsigned long long*)(fl_base + (size_t)((int)blockIdx.x * KSTR + 76 + (lane >> 1)) * 64u),
                                        ((unsigned long long)call << 32) | call, __ATOMIC_RELAXED, __HIP_MEMORY_SCOPE_SYSTEM);
      return;
    }
    char* dst[NR - 1];
#pragma unroll
    for (int q = 0; q < NR - 1; ++q) dst[q] = p.ws[(rk + 1 + (q + (int)blockIdx.x) % (NR - 1)) & 7] + par + (size_t)rk * SRCB;
    const unsigned l16 = (unsigned)lane * 16u, vzero = 0u, vone = 1u;
    const unsigned long long call2 = ((unsigned long long)call << 32) | call;
    int tau = TD, base = 0;
    unsigned mask = slot_mask(TD); int cnt = step_units(TD, mask);
    auto decode = [&](int idx) -> unsigned {
      while (idx >= base + cnt && tau < TEND - 2) { base += cnt; tau += 2; mask = slot_mask(tau); cnt = step_units(tau, mask); }
      int r = idx - base; const int kp = (tau & 31) >> 1;
      if (tau < 32) { if (r < 16) return (unsigned)(kp * KSTR + r); r -= 16; }
      unsigned m = mask; for (int i = 0; i < r; ++i) m &= m - 1u;
      const int slot = __builtin_ctz(m | 0x4000u);
      const unsigned u = (unsigned)(kp * KSTR + 16 + 15 * ((tau - start_of(slot)) >> 5) + slot);
      return u < (unsigned)(NUNITS / 2) ? u : 0u;
    };
    auto data_off = [&](unsigned u) -> unsigned { return (u + (u / KSTR) * KSTR) * 4096u; };
#define CLB_A "v224", "v225", "v226", "v227", "v228", "v229", "v230", "v231", "v232", "v233", "v234", "v235", "v236", "v237", "v238", "v239"
#define CLB_B "v240", "v241", "v242", "v243", "v244", "v245", "v246", "v247", "v248", "v249", "v250", "v251", "v252", "v253", "v254", "v255"
#define CLB "memory", "v223", CLB_A, CLB_B
#define DRAW(ctr) asm volatile("s_mov_b64 exec, 1\n\tglobal_atomic_add v223, %0, %1, %2 sc0\n\ts_mov_b64 exec, -1" :: "v"(vzero), "v"(vone), "s"(ctr) : CLB)
#define LOADU(R, o) asm volatile("global_load_dwordx4 v[%c2:%c2+3], %0, %1 sc0 sc1\n\tglobal_load_dwordx4 v[%c2+4:%c2+7], %0, %1 offset:1024 sc0 sc1\n\t" \
                   "global_load_dwordx4 v[%c2+8:%c2+11], %0, %1 offset:2048 sc0 sc1\n\tglobal_load_dwordx4 v[%c2+12:%c2+15], %0, %1 offset:3072 sc0 sc1" \
                   :: "v"((o) + l16), "s"(src), "i"(R) : CLB)
#define ST2(R, c, q) asm volatile("global_store_dwordx4 %0, v[%c2:%c2+3], %1 offset:%c3 sc0 sc1\n\tglobal_store_dwordx4 %0, v[%c2+4:%c2+7], %1 offset:%c3+1024 sc0 sc1" \
                   :: "v"(_o), "s"(dst[q]), "i"((R) + 4 * (c)), "i"((c) * 1024) : CLB)
#define ST14(R, c) ST2(R, c, 0); ST2(R, c, 1); ST2(R, c, 2); ST2(R, c, 3); ST2(R, c, 4); ST2(R, c, 5); ST2(R, c, 6)
#define STOREU(R, o) do { const unsigned _o = (o) + l16; ST14(R, 0); ST14(R, 2); } while (0)
#define FLAGU(u) do { char* _f = fl_base + (size_t)(u) * 64u; \
      asm volatile("s_mov_b64 exec, 0xff\n\tglobal_store_dwordx2 %0, %1, off sc0 sc1\n\ts_mov_b64 exec, -1" :: "v"(_f), "v"(call2) : CLB); } while (0)
#define READ_TK(tk) asm volatile("v_readfirstlane_b32 %0, v223" : "=s"(tk) :: CLB)
#define WAIT28() asm volatile("s_waitcnt vmcnt(28)" ::: CLB)
    unsigned tk;
    DRAW(qctr); asm volatile("s_waitcnt vmcnt(0)" ::: CLB); READ_TK(tk);
    unsigned idx = tk - qbase;
    if (!wait_words((unsigned)WS_LFLAG + lane * 4u, 4, 256u, 0ull)) return;
    if (idx < npush) {
      unsigned cur = decode((int)idx), prev = NUNITS / 2;
      unsigned o = data_off(cur);
      LOADU(224, o); DRAW(qctr);
      asm volatile("s_waitcnt vmcnt(0)" ::: CLB);
      for (;;) {
        LOADU(240, o + KTS);
        STOREU(224, o);
        READ_TK(tk); idx = tk - qbase;
        const bool more = idx < npush;
        unsigned nxt = cur, on = o;
        if (more) { nxt = decode((int)idx); on = data_off(nxt); }
        WAIT28();
        FLAGU(prev);
        if (more) { LOADU(224, on); DRAW(qctr); }
        STOREU(240, o + KTS);
        if (!more) break;
        WAIT28();
        prev = cur; cur = nxt; o = on;
      }
      asm volatile("s_waitcnt vmcnt(0)" ::: CLB);
      FLAGU(cur);
    }
    {
      constexpr unsigned nfw = 8 * 32, ROW = N_ * 4u;
      const unsigned qb2 = (call - 1u) * (nfw + (unsigned)NAG);
      const char* const osrc = (const char*)p.out;
#pragma unroll
      for (int q = 0; q < NR - 1; ++q) dst[q] = p.outs[(rk + 1 + (q + (int)blockIdx.x) % (NR - 1)) & 7];
      const unsigned obase = (unsigned)(rk & 1) * 2048u * ROW + (unsigned)(60 + (rk >> 1)) * 1024u;
#define OLD(R, o) asm volatile("global_load_dwordx4 v[%c5:%c5+3], %0, %4 sc0 sc1\n\tglobal_load_dwordx4 v[%c5+4:%c5+7], %1, %4 sc0 sc1\n\t" \
        "global_load_dwordx4 v[%c5+8:%c5+11], %2, %4 sc0 sc1\n\tglobal_load_dwordx4 v[%c5+12:%c5+15], %3, %4 sc0 sc1" \
        :: "v"((o) + l16), "v"((o) + l16 + ROW), "v"((o) + l16 + 2 * ROW), "v"((o) + l16 + 3 * ROW), "s"(osrc), "i"(R) : CLB)
#define OS2(R, c, q) asm volatile("global_store_dwordx4 %0, v[%c3:%c3+3], %2 sc0 sc1 nt\n\tglobal_store_dwordx4 %1, v[%c3+4:%c3+7], %2 sc0 sc1 nt" \
        :: "v"(_o + (c) * ROW), "v"(_o + (c) * ROW + ROW), "s"(dst[q]), "i"((R) + 4 * (c)) : CLB)
#define OS14(R, c) OS2(R, c, 0); OS2(R, c, 1); OS2(R, c, 2); OS2(R, c, 3); OS2(R, c, 4); OS2(R, c, 5); OS2(R, c, 6)
#define OST(R, o) do { const unsigned _o = (o) + l16; OS14(R, 0); OS14(R, 2); } while (0)
      DRAW(qctr + 128); asm volatile("s_waitcnt vmcnt(0)" ::: CLB); READ_TK(tk); idx = tk - qb2;
      if (!wait_words((unsigned)WS_OFLAG + (lane & 7) * 4u, 1, 0u, 0ull)) return;
      if (__builtin_amdgcn_readfirstlane((int)__hip_atomic_load((unsigned*)(myws + WS_TMO), __ATOMIC_RELAXED, __HIP_MEMORY_SCOPE_AGENT)) == (int)call) return;
      if (idx < nfw) {
        unsigned o = obase + idx * (8u * ROW);
        OLD(224, o); DRAW(qctr + 128); asm volatile("s_waitcnt vmcnt(0)" ::: CLB);
        for (;;) {
          OLD(240, o + 4u * ROW); OST(224, o);
          READ_TK(tk); idx = tk - qb2;
          const bool more = idx < nfw; const unsigned on = obase + idx * (8u * ROW);
          WAIT28();
          if (more) { OLD(224, on); DRAW(qctr + 128); }
          OST(240, o + 4u * ROW);
          if (!more) break;
          WAIT28();
          o = on;
        }
      }
      asm volatile("s_waitcnt vmcnt(0)" ::: CLB);
      if (lane < NR) __hip_atomic_store((unsigned*)(p.ws[lane & 7] + WS_DONE + blockIdx.x * 32 + rk * 4), call, __ATOMIC_RELAXED, __HIP_MEMORY_SCOPE_SYSTEM);
      if (!wait_words((unsigned)WS_DONE + blockIdx.x * 32u + (lane & 7) * 4u, 1, 0u, 0ull))
        *(float*)((char*)p.out + obase + blockIdx.x * 4u) = __builtin_nanf("");
    }
    asm volatile("s_waitcnt vmcnt(0)" ::: "memory");
    return;
  }

  const int wr = wid >> 2, wc = wid & 3, fr = lane & 15, fq = lane >> 4;
  int pm, slot;
  { const int b = blockIdx.x - NAG, x = b & 7, r = b >> 3, a = x >> 1, hb = x & 1;
    if (r < 28) { pm = 4 * a + (r & 3); slot = hb * 8 + (r >> 2); } else { pm = 4 * a + 2 * hb + (r - 28); slot = 7; } }
  const bool has_half = slot == 0, sharer = slot == SSLOT && (pm >> 3) == (rk & 1);
  constexpr int HCOL = 15 * SROUND + SSLOT;
  const int hblk = 8 * (rk & 1) + (pm >> 1), hrow = pm & 1;
  auto col_of = [&](int rnd) { return (sharer && rnd == SROUND) ? 60 + (rk >> 1) : 15 * rnd + slot; };
  const float scale = __builtin_bit_cast(float, __builtin_amdgcn_readfirstlane(__builtin_bit_cast(int, p.sx[0] * p.sw[0])));
  const unsigned voff = (unsigned)((wid >> 2) * 4 + fq) * SRCB + (unsigned)((wid & 3) * 256 + fr * 16);
  const unsigned lds0 = (unsigned)(size_t)lds, ldsw = lds0 + LDS_OP + (unsigned)wid * 2048u;
  int aoff = LDS_OP + wr * 1024 + lane * 16, boff = LDS_OP + 4 * HTB + wc * 2048 + lane * 16;
  asm volatile("" : "+v"(aoff), "+v"(boff));
  const unsigned coff = (unsigned)((wr * 64 + 4 * fq) * (N_ * 4) + (wc * 64 + 4 * fr) * 4);
  const u32x4 srd = {(unsigned)(size_t)myws, (unsigned)((size_t)myws >> 32) & 0xffffu, (unsigned)WS_NEED, 0x00020000u};
  const unsigned cA = (unsigned)par + (unsigned)pm * 4096u;
  const unsigned cW = (unsigned)par + 16u * 4096u;
  const unsigned hA = (unsigned)par + (unsigned)hblk * 4096u + hrow * GH, hB = cW + HCOL * 4096u;
  const unsigned long long own = 0x0003000300030003ull << (2 * rk);
#define SA(b, h) (((b) * 2 + (h)) * HTB)
#define SB(b, h) ((4 + (b) * 2 + (h)) * HTB)
#define STAGE(bufoff, goff) asm volatile("s_add_u32 m0, %3, %4\n\ts_nop 0\n\tbuffer_load_dwordx4 %0, %1, %2 offen sc0 sc1 lds\n\t" \
    "buffer_load_dwordx4 %0, %1, %2 offen offset:1024 sc0 sc1 lds" :: "v"(voff), "s"(srd), "s"(goff), "s"(ldsw), "i"(bufoff) : "scc", "memory")
#define FLAG_FETCH(ko, sl) asm volatile("s_mov_b32 m0, %3\n\ts_nop 0\n\tbuffer_load_dword %0, %1, %2 offen sc0 sc1 lds" :: "v"(vf), "s"(srd), "s"((ko) >> 7), "s"(lds0 + (sl)) : "memory")
#define LDA_(dst, base) do { _Pragma("unroll") for (int m = 0; m < 4; ++m) _Pragma("unroll") for (int k = 0; k < 2; ++k) dst[m][k] = *(const LAS i32x4*)(lds + (base) + m * 2048 + k * 8192); } while (0)
#define LDB_(dst, base) do { _Pragma("unroll") for (int n = 0; n < 2; ++n) _Pragma("unroll") for (int k = 0; k < 2; ++k) dst[n][k] = *(const LAS i32x4*)(lds + (base) + n * 1024 + k * 8192); } while (0)
#define LDA(dst, b, h) LDA_(dst, SA(b, h) + aoff)
#define LDB(dst, b, h) LDB_(dst, SA(b, h) + boff)
#define MFMA2(c, a0, b0, a1, b1, Z) do { \
    if (Z) asm volatile("v_mfma_i32_16x16x64_i8 %0, %1, %2, 0.15915494\n\tv_mfma_i32_16x16x64_i8 %0, %3, %4, %0" : "+v"(c) : "v"(a0), "v"(b0), "v"(a1), "v"(b1)); \
    else asm volatile("v_mfma_i32_16x16x64_i8 %0, %1, %2, %0\n\tv_mfma_i32_16x16x64_i8 %0, %3, %4, %0" : "+v"(c) : "v"(a0), "v"(b0), "v"(a1), "v"(b1)); } while (0)
#define MMA2(ai, Z, PARK) do { __builtin_amdgcn_s_setprio(1); \
    _Pragma("unroll") for (int i = 0; i < 16; ++i) { \
      const int k = i & 1, m = i >> 2, c = (m & 1) ? 3 - (i & 3) : (i & 3); \
      if ((PARK) == 1 && m >= 2) asm volatile("v_mfma_i32_16x16x64_i8 %0, %1, %2, 0.15915494\n\tv_mfma_i32_16x16x64_i8 %0, %3, %4, %0" \
          : "=&v"(park[m & 1][c]) : "v"(At[m][k]), "v"(c < 2 ? B0[c & 1][k] : B1[c & 1][k]), "v"(At[m][k ^ 1]), "v"(c < 2 ? B0[c & 1][k ^ 1] : B1[c & 1][k ^ 1])); \
      else if ((PARK) == 2 && m >= 2) asm volatile("v_mfma_i32_16x16x64_i8 %0, %1, %2, %5\n\tv_mfma_i32_16x16x64_i8 %0, %3, %4, %0" \
          : "+v"(acc[ai][c >> 1][m][c & 1]) : "v"(At[m][k]), "v"(c < 2 ? B0[c & 1][k] : B1[c & 1][k]), "v"(At[m][k ^ 1]), "v"(c < 2 ? B0[c & 1][k ^ 1] : B1[c & 1][k ^ 1]), "v"(park[m & 1][c])); \
      else MFMA2(acc[ai][c >> 1][m][c & 1], At[m][k], (c < 2 ? B0[c & 1][k] : B1[c & 1][k]), At[m][k ^ 1], (c < 2 ? B0[c & 1][k ^ 1] : B1[c & 1][k ^ 1]), Z); } \
    __builtin_amdgcn_s_setprio(0); } while (0)
#define WAIT_V(n) asm volatile("s_waitcnt vmcnt(" #n ")" ::: "memory")
#define WAIT_L(n) do { asm volatile("" ::: "memory"); __builtin_amdgcn_s_waitcnt(0xC07F); asm volatile("" ::: "memory"); } while (0)
#define BAR __builtin_amdgcn_s_barrier()
#define SCHED __builtin_amdgcn_sched_barrier(0)
#define PIN do { asm volatile("" ::: "memory"); SCHED; } while (0)
#define EPI_(s0, s1, OPS, POL) do { \
    _Pragma("unroll") for (int s = (s0); s < (s1); ++s) { const int ai = s >> 4, m = (s >> 2) & 3, j = s & 3; float f0, f1, f2, f3; \
      asm volatile(OPS : "=&v"(f0), "=&v"(f1), "=&v"(f2), "=&v"(f3) \
                   : "v"(acc[ai][0][m][0][j]), "v"(acc[ai][0][m][1][j]), "v"(acc[ai][1][m][0][j]), "v"(acc[ai][1][m][1][j]), "s"(scale)); \
      const f32x4 v = {f0, f1, f2, f3}; \
      asm volatile("buffer_store_dwordx4 %0, %1, %2, %3 offen " POL "\n\ts_nop 1" :: "v"(v), "v"(coff), "s"(crs), "s"((ai * 128 + m * 16 + j) * (N_ * 4)) : "memory"); } } while (0)
#define OPS_MUL "v_mul_f32 %0, %8, %0\n\tv_mul_f32 %1, %8, %1\n\tv_mul_f32 %2, %8, %2\n\tv_mul_f32 %3, %8, %3"
#define OPS_SAT "v_sub_u32 %0, %4, 0.15915494 clamp\n\tv_sub_u32 %1, %5, 0.15915494 clamp\n\tv_sub_u32 %2, %6, 0.15915494 clamp\n\tv_sub_u32 %3, %7, 0.15915494 clamp\n\t" \
    "v_cvt_f32_u32 %0, %0\n\tv_cvt_f32_u32 %1, %1\n\tv_cvt_f32_u32 %2, %2\n\tv_cvt_f32_u32 %3, %3\n\t" OPS_MUL
#define OPS_ANY "v_subrev_u32 %0, 0.15915494, %4\n\tv_subrev_u32 %1, 0.15915494, %5\n\tv_subrev_u32 %2, 0.15915494, %6\n\tv_subrev_u32 %3, 0.15915494, %7\n\t" \
    "v_cvt_f32_i32 %0, %0\n\tv_cvt_f32_i32 %1, %1\n\tv_cvt_f32_i32 %2, %2\n\tv_cvt_f32_i32 %3, %3\n\t" OPS_MUL "\n\t" \
    "v_max_f32 %0, 0, %0\n\tv_max_f32 %1, 0, %1\n\tv_max_f32 %2, 0, %2\n\tv_max_f32 %3, 0, %3"
#define EPI_ST(s0, s1) do { if ((s1) > (s0)) { if (__builtin_expect(wt >= wt_min, 0)) EPI_(s0, s1, OPS_ANY, "sc0 sc1"); else EPI_(s0, s1, OPS_SAT, "nt"); } } while (0)
  auto flag_off = [&](int rnd) -> unsigned { const int l = lane_id();
    return (unsigned)WS_FLAG + (unsigned)(((l & 16) ? 16 + col_of(rnd) : pm) * 64 + (l & 15) * 4); };
  auto nextk = [&](unsigned k) -> unsigned { k += 2 * KTS; return k < 32 * KTS ? k : 0u; };

  i32x4 acc[2][2][4][2];
  i32x4 At[4][2], B0[2][2], B1[2][2], park[2][4];
  unsigned k0 = (unsigned)start_of(slot) * KTS, k1 = nextk(k0);
  unsigned vf = flag_off(0);
  if (wid == 0) {
    wait_words(vf + (k0 >> 7), 1, 0u, own);
    wait_words((unsigned)WS_LFLAG + lane_id() * 4u, 4, 256u, 0ull);
    FLAG_FETCH(k1, 0u);
  }
  asm volatile("s_waitcnt vmcnt(0)" ::: "memory");
  BAR;
  { const unsigned a = cA + k0, b = cW + (unsigned)slot * 4096u + k0;
    STAGE(SB(0, 0), b); STAGE(SB(0, 1), b + GH); STAGE(SA(0, 0), a); STAGE(SA(0, 1), a + GH);
    if (wr == 1) BAR;
    WAIT_V(2); BAR;
    STAGE(SB(1, 0), b + KTS); STAGE(SA(1, 0), a + KTS); STAGE(SB(1, 1), b + GH + KTS);
    WAIT_V(0); BAR; PIN; }
#pragma unroll
  for (int a = 0; a < 2; ++a)
#pragma unroll
    for (int b = 0; b < 2; ++b)
#pragma unroll
      for (int m = 0; m < 4; ++m)
#pragma unroll
        for (int n = 0; n < 2; ++n) asm volatile("" : "=v"(acc[a][b][m][n]));
  auto mk_rs = [](const float* b, unsigned n) { return (u32x4){(unsigned)(size_t)b, (unsigned)((size_t)b >> 32) & 0xffffu, n, 0x00020000u}; };
  u32x4 crs = mk_rs(p.out, 0u);
  const unsigned wt_min = __builtin_bit_cast(unsigned, scale) < 0x7f800000u ? 60u : 0u;
  unsigned wt = 0;
  unsigned fsl = 0;
#define KT(b, ah, a2, b2, Z, PARK, V1, V2, e0, e1, e2, FL) do { \
    unsigned fv = 0; \
    LDB(B0, b, 0); LDB(B1, b, 1); SCHED; LDA(At, b, 0); STAGE(SA((b) ^ 1, 1), ah); \
    if ((FL) && wid == 0) { fv = *(const LAS unsigned*)(lds + fsl + lane_id() * 4); if (newf) vf = flag_off(un); FLAG_FETCH(k2, fsl ^ 256u); } \
    PIN; EPI_ST(e0, e1); PIN; \
    WAIT_V(V1); WAIT_L(0); \
    if ((FL) && wid == 0) { if ((__ballot((int)(fv - call) >= 0) | own) != ~0ull) wait_words(flag_off(c1) + (k1 >> 7), 1, 0u, own); \
      if ((FL) == 2 && tohalf) { const int l = lane_id(); const unsigned fo = (unsigned)WS_FLAG + (unsigned)((l >> 4) * (int)(KTS >> 6) + (l & 15) * 4); \
        wait_words(fo + (16 + HCOL) * 64, 4, KTS >> 4, own); wait_words(fo + (unsigned)hblk * 64u, 4, KTS >> 4, own); } \
      if (pub == 2) __hip_atomic_store((unsigned*)(myws + WS_OFLAG) + (pm & 7), call, __ATOMIC_RELAXED, __HIP_MEMORY_SCOPE_AGENT); \
    } \
    BAR; MMA2(0, Z, 0); BAR; PIN; \
    LDA(At, b, 1); STAGE(SB(b, 0), b2); STAGE(SB(b, 1), (b2) + GH); STAGE(SA(b, 0), a2); PIN; EPI_ST(e1, e2); PIN; \
    WAIT_V(V2); WAIT_L(0); BAR; MMA2(1, Z, PARK); BAR; PIN; } while (0)
#define PAIR(C1, NEWF, PUB, Z, Va, Vb, Vc, Vd, e0, e1, e2, e3, e4, FL) do { \
    const int c1 = (C1); const bool newf = (NEWF); const int pub = (PUB); const unsigned k2 = nextk(k1); \
    unsigned a1 = cA + k1, b1 = cW + (unsigned)col_of(c1) * 4096u + k1, ah1 = a1 + GH; \
    if ((FL) == 2 && tohalf) { a1 = hA; b1 = hB; ah1 = hB + 2 * KTS; }     \
    KT(0, cA + k0 + KTS + GH, a1, b1, Z, Z, Va, Vb, e0, e1, e2, FL); \
    KT(1, ah1, a1 + KTS, b1 + KTS, 0, 2 * (Z), Vc, Vd, e2, e3, e4, 0); \
    k0 = k1; k1 = k2; fsl ^= 256u; } while (0)
  for (int ui = 0;; ++ui) {
    const bool has_next = ui < 3, tohalf = !has_next && has_half;
    const int un = has_next ? ui + 1 : ui;
    PAIR(ui, false, 0, 1, 24, 32, 28, 24, 8, 16, 24, 28, 32, 1);
    crs = mk_rs(p.out + (size_t)pm * 256 * N_ + col_of(ui) * 256, 256u * N_ * 4u);
    wt = (unsigned)col_of(ui);
    for (int t = 1; t < 15; ++t) PAIR(ui, t == 14, (sharer && ui == SROUND + 1) ? t : 0, 0, 8, 8, 8, 8, 0, 0, 0, 0, 0, 1);
    PAIR(un, false, 0, 0, 8, 8, 8, 16, 0, 0, 0, 0, 8, 2);
    if (!has_next) break;
  }
  if (has_half) {
    int aoff2 = 8 * HTB + aoff; asm volatile("" : "+v"(aoff2));
    const int boff2 = boff - 4 * HTB;
#define HST(j, w, g) do { if ((j) == 0) { if ((w) == 0) STAGE(SA(0, 1), g); else if ((w) == 1) STAGE(SA(1, 1), g); else STAGE(8 * HTB, g); } \
      else { if ((w) == 0) STAGE(SB(((j) + 1) & 1, 0), g); else if ((w) == 1) STAGE(SB(((j) + 1) & 1, 1), g); else STAGE(SA(((j) + 1) & 1, 0), g); } } while (0)
#define HKT(j, tn, Z, V, e0, e1) do { \
      const unsigned a = hA + (unsigned)(tn) * KTS, b = hB + (unsigned)(tn) * KTS; \
      if ((j) < 2) LDB(B0, j, 0); else LDB_(B0, SA(0, 1) + boff2); \
      HST(j, 0, b); \
      if ((j) < 2) LDB(B1, j, 1); else LDB_(B1, SA(1, 1) + boff2); \
      HST(j, 1, b + GH); \
      SCHED; if ((j) < 2) LDA(At, j, 0); else LDA_(At, aoff2); \
      HST(j, 2, a); \
      PIN; EPI_ST(e0, e1); PIN; \
      WAIT_V(V); WAIT_L(0); BAR; MMA2(0, Z, 0); BAR; PIN; } while (0)
    HKT(0, 2, 1, 22, 8, 16); HKT(1, 3, 0, 18, 16, 20); HKT(2, 4, 0, 14, 20, 24);
    HKT(0, 5, 0, 14, 24, 28); HKT(1, 6, 0, 14, 28, 32); HKT(2, 7, 0, 10, 0, 0);
    crs = mk_rs(p.out + (size_t)(hblk * 256 + hrow * 128) * N_ + HCOL * 256, 128u * N_ * 4u);
    for (int t = 6; t < 30; t += 3) { HKT(0, t + 2, 0, 6, 0, 0); HKT(1, t + 3, 0, 6, 0, 0); HKT(2, t + 4, 0, 6, 0, 0); }
    HKT(0, 31, 0, 6, 0, 0); HKT(1, 31, 0, 6, 0, 0);
  }
  const unsigned gave_up = __hip_atomic_load((unsigned*)(myws + WS_TMO), __ATOMIC_RELAXED, __HIP_MEMORY_SCOPE_AGENT);
  asm volatile("s_nop 7" ::: "memory"); SCHED;
  if (has_half) EPI_ST(0, 4); else EPI_ST(8, 12);
  if (wr == 0) BAR;
  if (has_half) EPI_ST(4, 16); else EPI_ST(12, 32);
  PIN;
  if (__builtin_amdgcn_readfirstlane((int)gave_up) == (int)call) {
    WAIT_V(0);
    const float q = __builtin_nanf(""); const f32x4 v = {q, q, q, q};
    for (int ui = 0; ui < (has_half ? 5 : 4); ++ui) {
      crs = ui < 4 ? mk_rs(p.out + (size_t)pm * 256 * N_ + col_of(ui) * 256, 256u * N_ * 4u)
                   : mk_rs(p.out + (size_t)(hblk * 256 + hrow * 128) * N_ + HCOL * 256, 128u * N_ * 4u);
      for (int s = 0; s < 32; ++s)
        asm volatile("buffer_store_dwordx4 %0, %1, %2, %3 offen\n\ts_nop 1" :: "v"(v), "v"(coff), "s"(crs), "s"(((s >> 4) * 128 + ((s >> 2) & 3) * 16 + (s & 3)) * (N_ * 4)) : "memory");
    }
  }
  WAIT_V(0);
}

extern "C" void kernel_launch(int rank, int nranks, ncclComm_t comm, hipStream_t stream,
                              void* const* d_in, const int* in_sizes, int n_in,
                              void* d_out, int out_size, const int* in_dtypes, int out_dtype,
                              void* const* d_out_all, void* const* d_ws_all, size_t ws_bytes) {
  (void)comm; (void)in_dtypes; (void)out_dtype;
  if (nranks != NR || (unsigned)rank >= (unsigned)NR || n_in != 4 || ws_bytes < WS_NEED || out_size != M_ * N_ ||
      in_sizes[0] != M_ * KS || in_sizes[1] != KS * N_) {
    fprintf(stderr, "kernel_launch: unexpected configuration\n");
    return;
  }
  Params p;
  for (int i = 0; i < NR; ++i) p.ws[i] = (char*)d_ws_all[i];
  p.x = (const signed char*)d_in[0]; p.w = (const signed char*)d_in[1];
  p.sx = (const float*)d_in[2]; p.sw = (const float*)d_in[3];
  p.out = (float*)d_out; p.rank = rank;
  for (int i = 0; i < NR; ++i) p.outs[i] = (char*)d_out_all[i];
  hipLaunchKernelGGL(fused_kernel, dim3(GRID), dim3(512), 0, stream, p);
}
```

```cpp
#include <hip/hip_runtime.h>
#include <rccl/rccl.h>
#include <cstdio>

#define LAS __attribute__((address_space(3)))
#define GAS __attribute__((address_space(1)))
typedef int i32x4 __attribute__((ext_vector_type(4)));
typedef unsigned u32x4 __attribute__((ext_vector_type(4)));
typedef float f32x4 __attribute__((ext_vector_type(4)));

namespace {
constexpr int M_ = 4096, N_ = 16384, K_ = 4096, NR = 8;
constexpr int KS = K_ / NR;
constexpr int HTB = 16384;
constexpr int GH = 2048;
constexpr int LDS_OP = 512;
constexpr int LDS_BYTES = LDS_OP + 9 * HTB;
constexpr int NAG = 16;
constexpr int GRID = 256;
constexpr int TD = 2;
constexpr int DGAP = 2;
constexpr int DSIDE = 6;
constexpr int TSEL = 52;
constexpr int SSLOT = 8, SROUND = 2;
constexpr size_t WS_CTR = 0;
constexpr size_t WS_TMO = 128 * 1024;
constexpr size_t WS_QUEUE = 128 * 1024 + 256;
constexpr size_t WS_LFLAG = 128 * 1024 + 1024;
constexpr size_t WS_OFLAG = 128 * 1024 + 3072;
constexpr size_t WS_DONE = 128 * 1024 + 4096;
constexpr size_t WS_FLAG = 256 * 1024;
constexpr size_t WS_DATA = 1024 * 1024;
constexpr int KSTR = 80;
constexpr int NUNITS = 32 * KSTR;
constexpr unsigned KTS = KSTR * 4096u;
constexpr unsigned SRCB = 32 * KTS + 256u;
constexpr size_t PARB = 81ull << 20;
static_assert(PARB >= 8ull * SRCB, "parity region");
constexpr size_t WS_NEED = WS_DATA + 2 * PARB;
constexpr unsigned DEADLINE = 50000000u;

struct Params {
  char* ws[NR];
  const signed char* x; const signed char* w; const float* sx; const float* sw;
  float* out; int rank;
  char* outs[NR];
};

__device__ __forceinline__ unsigned perm(unsigned hi, unsigned lo, unsigned sel) { return __builtin_amdgcn_perm(hi, lo, sel); }

__device__ __forceinline__ int start_of(int slot) { return slot == 0 ? 0 : 2 * (slot & 7) + DGAP + (slot < 7 ? DSIDE : 0); }
__device__ __forceinline__ unsigned slot_mask(int tau) {
  unsigned m = 0;
#pragma unroll
  for (int j = 0; j < 15; ++j) m |= ((unsigned)(tau - start_of(j)) < 128u ? 1u : 0u) << j;
  return m;
}
__device__ __forceinline__ int step_units(int tau, unsigned mask) { return (tau < 32 ? 16 : 0) + __builtin_popcount(mask); }
constexpr int TEND = 14 + DSIDE + DGAP + 128;
static_assert(TD <= 4 && TD % 2 == 0 && DGAP % 2 == 0 && DSIDE % 2 == 0 && TSEL % 2 == 0 && TEND < 160 && TSEL < TEND, "schedule");
}

__global__ __launch_bounds__(512, 2) void fused_kernel(const Params p) {
  __shared__ __attribute__((aligned(16))) unsigned char shm[LDS_BYTES];
  LAS unsigned char* lds = (LAS unsigned char*)shm;
  const int tid = threadIdx.x, wid = __builtin_amdgcn_readfirstlane(tid >> 6), lane = tid & 63;
  const int rk = p.rank & 7;
  char* const myws = p.ws[rk];
  unsigned call;
  size_t par;

  {
    const int b = ((int)blockIdx.x + GRID - NAG) & (GRID - 1);
    const bool hasx = wid >= 6;
    const int ex = ((wid & 1) << 8) + b, xkt = ex >> 4, xb = ex & 15;
    u32x4 vx[4] = {};
    if (hasx) {
      const signed char* s = p.x + (size_t)xb * 256 * KS + xkt * 16;
#pragma unroll
      for (int c = 0; c < 4; ++c) vx[c] = *(const GAS u32x4*)(s + (unsigned)((lane + c * 64) * KS));
    }
    const int ew = (wid << 8) + b, wkt = ew >> 6, col = ew & 63;
    unsigned q[16];
    { const signed char* s = p.w + (size_t)wkt * 16 * N_ + col * 256;
#pragma unroll
      for (int j = 0; j < 16; ++j) q[j] = *(const GAS unsigned*)(s + (unsigned)(lane * 4 + j * N_)); }
    asm volatile("" ::: "memory");
    unsigned tkt = 0;
    if (lane == 0) tkt = __hip_atomic_fetch_add((unsigned*)(myws + WS_CTR + (size_t)(blockIdx.x * 8 + wid) * 64), 1u, __ATOMIC_RELAXED, __HIP_MEMORY_SCOPE_AGENT);
    call = (unsigned)__builtin_amdgcn_readfirstlane((int)tkt) + 1u;
    par = WS_DATA + (size_t)(call & 1u) * PARB;
    u32x4 vw[4];
#pragma unroll
    for (int t = 0; t < 4; ++t) {
      const unsigned t0 = perm(q[4 * t + 1], q[4 * t], 0x05010400u), t1 = perm(q[4 * t + 1], q[4 * t], 0x07030602u);
      const unsigned t2 = perm(q[4 * t + 3], q[4 * t + 2], 0x05010400u), t3 = perm(q[4 * t + 3], q[4 * t + 2], 0x07030602u);
      vw[0][t] = perm(t2, t0, 0x05040100u); vw[1][t] = perm(t2, t0, 0x07060302u);
      vw[2][t] = perm(t3, t1, 0x05040100u); vw[3][t] = perm(t3, t1, 0x07060302u);
    }
    const unsigned uw = (unsigned)(wkt * KSTR + 16 + col), ux = (unsigned)(xkt * KSTR + xb);
    const unsigned lo = (unsigned)rk * SRCB + (unsigned)lane * 16u;
#define PUT(d, o, v) asm volatile("global_store_dwordx4 %0, %1, %5 sc0 sc1\n\tglobal_store_dwordx4 %0, %2, %5 offset:1024 sc0 sc1\n\t" \
      "global_store_dwordx4 %0, %3, %5 offset:2048 sc0 sc1\n\tglobal_store_dwordx4 %0, %4, %5 offset:3072 sc0 sc1\n\ts_nop 1" \
      :: "v"(o), "v"(v[0]), "v"(v[1]), "v"(v[2]), "v"(v[3]), "s"(d) : "memory")
    PUT(myws + par, lo + uw * 4096u, vw);
    if (hasx) PUT(myws + par, lo + ux * 4096u, vx);
    const bool dw = wkt < TD && col < 15 && start_of(col) <= wkt, dx = hasx && xkt < TD;
    if (tid == 0) *(volatile LAS unsigned*)lds = 0u;
    asm volatile("s_waitcnt vmcnt(0) lgkmcnt(0)" ::: "memory");
    __builtin_amdgcn_s_barrier();
    if (tid == 64) __hip_atomic_store((unsigned*)(myws + WS_LFLAG) + blockIdx.x, call, __ATOMIC_RELAXED, __HIP_MEMORY_SCOPE_AGENT);
    if (dw || dx) {
      if (dx) { vw[0] = vx[0]; vw[1] = vx[1]; vw[2] = vx[2]; vw[3] = vx[3]; }
      const unsigned o = lo + (dx ? ux : uw) * 4096u; const int dkt = dx ? xkt : wkt;
      __builtin_amdgcn_s_sleep(1);
#pragma unroll
      for (int i = 1; i < NR; ++i) PUT(p.ws[(rk + 1 + (i + b) % (NR - 1)) & 7] + par, o, vw);
      asm volatile("s_waitcnt vmcnt(0)" ::: "memory");
      if (lane < NR) __hip_atomic_store((unsigned*)(p.ws[lane & 7] + WS_FLAG + rk * 8 + (size_t)(((dkt >> 1) * KSTR + (dx ? xb : 16 + col)) * 64 + (dkt & 1) * 4)), call, __ATOMIC_RELAXED, __HIP_MEMORY_SCOPE_SYSTEM);
    }
  }
  auto lane_id = [&]() -> int { int l; asm volatile("v_mbcnt_lo_u32_b32 %0, -1, 0\n\tv_mbcnt_hi_u32_b32 %0, -1, %0" : "=v"(l)); return l; };
  auto wait_words = [&](unsigned off, int nw, unsigned stride, unsigned long long own) -> bool {
    const unsigned t0 = (unsigned)__builtin_amdgcn_s_memrealtime();
    for (;;) {
      unsigned long long ok = ~0ull;
      unsigned v[4];
#pragma unroll
      for (int i = 0; i < 4; ++i) if (i < nw) v[i] = __hip_atomic_load((const unsigned*)(myws + (off + i * stride)), __ATOMIC_RELAXED, __HIP_MEMORY_SCOPE_SYSTEM);
#pragma unroll
      for (int i = 0; i < 4; ++i) if (i < nw) ok &= __ballot((int)(v[i] - call) >= 0);
      if ((ok | own) == ~0ull) return true;
      unsigned to; asm volatile("v_mov_b32 %0, %1" : "=v"(to) : "i"((unsigned)WS_TMO));
      unsigned* const tp = (unsigned*)(myws + to);
      const unsigned now = (unsigned)__builtin_amdgcn_s_memrealtime();
      if (__builtin_amdgcn_readfirstlane((int)__hip_atomic_load(tp, __ATOMIC_RELAXED, __HIP_MEMORY_SCOPE_AGENT)) == (int)call || now - t0 > DEADLINE) {
        unsigned cv; asm volatile("v_mov_b32 %0, %1" : "=v"(cv) : "s"(call));
        __hip_atomic_store(tp, cv, __ATOMIC_RELAXED, __HIP_MEMORY_SCOPE_AGENT);
        return false;
      }
      __builtin_amdgcn_s_sleep(4);
    }
  };

  if (blockIdx.x < NAG) {
    if (wid > 1) return;
    const int lane = lane_id();
    char* const fl_base = p.ws[lane & 7] + WS_FLAG + rk * 8;
    int ndirect = 0;
    for (int t = 0; t < TD; t += 2) ndirect += step_units(t, slot_mask(t));
    const unsigned npush = (unsigned)(76 * 16 - ndirect);
    const unsigned qbase = (call - 1u) * (npush + (unsigned)NAG);
    const char* const qctr = myws + WS_QUEUE;
    const char* const src = myws + par + (size_t)rk * SRCB;
    if (wid == 1) {
      unsigned thr = 0;
      for (int t = TD; t < TSEL; t += 2) thr += (unsigned)step_units(t, slot_mask(t));
      const unsigned t0 = (unsigned)__builtin_amdgcn_s_memrealtime();
      for (;;) {
        if ((unsigned)__builtin_amdgcn_readfirstlane((int)__hip_atomic_load((const unsigned*)qctr, __ATOMIC_RELAXED, __HIP_MEMORY_SCOPE_AGENT)) - qbase >= thr) break;
        if (__builtin_amdgcn_readfirstlane((int)__hip_atomic_load((unsigned*)(myws + WS_TMO), __ATOMIC_RELAXED, __HIP_MEMORY_SCOPE_AGENT)) == (int)call
            || (unsigned)__builtin_amdgcn_s_memrealtime() - t0 > DEADLINE) {
          __hip_atomic_store((unsigned*)(myws + WS_TMO), call, __ATOMIC_RELAXED, __HIP_MEMORY_SCOPE_AGENT);
          *(volatile LAS unsigned*)lds = 2u;
          return; }
        __builtin_amdgcn_s_sleep(64);
      }
      for (int i = 0; i < 4; ++i) {
        const int g = (i + (int)blockIdx.x) & 3;
        const unsigned o = (unsigned)(2 * (int)blockIdx.x * KSTR + 76 + g) * 4096u + (unsigned)lane * 16u, o2 = o + KTS;
        u32x4 v[4], w[4];
        asm volatile("global_load_dwordx4 %0, %8, %10 sc0 sc1\n\tglobal_load_dwordx4 %1, %8, %10 offset:1024 sc0 sc1\n\t"
                     "global_load_dwordx4 %2, %8, %10 offset:2048 sc0 sc1\n\tglobal_load_dwordx4 %3, %8, %10 offset:3072 sc0 sc1\n\t"
                     "global_load_dwordx4 %4, %9, %10 sc0 sc1\n\tglobal_load_dwordx4 %5, %9, %10 offset:1024 sc0 sc1\n\t"
                     "global_load_dwordx4 %6, %9, %10 offset:2048 sc0 sc1\n\tglobal_load_dwordx4 %7, %9, %10 offset:3072 sc0 sc1\n\ts_waitcnt vmcnt(0)"
                     : "=&v"(v[0]), "=&v"(v[1]), "=&v"(v[2]), "=&v"(v[3]), "=&v"(w[0]), "=&v"(w[1]), "=&v"(w[2]), "=&v"(w[3]) : "v"(o), "v"(o2), "s"(src) : "memory");
        if (2 * g != rk) { PUT(p.ws[2 * g] + par + (size_t)rk * SRCB, o, v); PUT(p.ws[2 * g] + par + (size_t)rk * SRCB, o2, w); }
        if (2 * g + 1 != rk) { PUT(p.ws[2 * g + 1] + par + (size_t)rk * SRCB, o, v); PUT(p.ws[2 * g + 1] + par + (size_t)rk * SRCB, o2, w); }
      }
      asm volatile("s_waitcnt vmcnt(0)" ::: "memory");
      if (lane < NR) __hip_atomic_store((unsigned long long*)(fl_base + (size_t)((int)blockIdx.x * KSTR + 76 + (lane >> 1)) * 64u),
                                        ((unsigned long long)call << 32) | call, __ATOMIC_RELAXED, __HIP_MEMORY_SCOPE_SYSTEM);
      bool go = wait_words((unsigned)WS_OFLAG + (lane & 7) * 4u, 1, 0u, 0ull);
      go = go && __builtin_amdgcn_readfirstlane((int)__hip_atomic_load((unsigned*)(myws + WS_TMO), __ATOMIC_RELAXED, __HIP_MEMORY_SCOPE_AGENT)) != (int)call;
      *(volatile LAS unsigned*)lds = go ? 1u : 2u;
      return;
    }
    char* dst[NR - 1];
#pragma unroll
    for (int q = 0; q < NR - 1; ++q) dst[q] = p.ws[(rk + 1 + (q + (int)blockIdx.x) % (NR - 1)) & 7] + par + (size_t)rk * SRCB;
    const unsigned l16 = (unsigned)lane * 16u, vzero = 0u, vone = 1u;
    const unsigned long long call2 = ((unsigned long long)call << 32) | call;
    int tau = TD, base = 0;
    unsigned mask = slot_mask(TD); int cnt = step_units(TD, mask);
    auto decode = [&](int idx) -> unsigned {
      while (idx >= base + cnt && tau < TEND - 2) { base += cnt; tau += 2; mask = slot_mask(tau); cnt = step_units(tau, mask); }
      int r = idx - base; const int kp = (tau & 31) >> 1;
      if (tau < 32) { if (r < 16) return (unsigned)(kp * KSTR + r); r -= 16; }
      unsigned m = mask; for (int i = 0; i < r; ++i) m &= m - 1u;
      const int slot = __builtin_ctz(m | 0x4000u);
      const unsigned u = (unsigned)(kp * KSTR + 16 + 15 * ((tau - start_of(slot)) >> 5) + slot);
      return u < (unsigned)(NUNITS / 2) ? u : 0u;
    };
    auto data_off = [&](unsigned u) -> unsigned { return (u + (u / KSTR) * KSTR) * 4096u; };
#define CLB_A "v224", "v225", "v226", "v227", "v228", "v229", "v230", "v231", "v232", "v233", "v234", "v235", "v236", "v237", "v238", "v239"
#define CLB_B "v240", "v241", "v242", "v243", "v244", "v245", "v246", "v247", "v248", "v249", "v250", "v251", "v252", "v253", "v254", "v255"
#define CLB "memory", "v223", CLB_A, CLB_B
#define DRAW(ctr) asm volatile("s_mov_b64 exec, 1\n\tglobal_atomic_add v223, %0, %1, %2 sc0\n\ts_mov_b64 exec, -1" :: "v"(vzero), "v"(vone), "s"(ctr) : CLB)
#define LOADU(R, o) asm volatile("global_load_dwordx4 v[%c2:%c2+3], %0, %1 sc0 sc1\n\tglobal_load_dwordx4 v[%c2+4:%c2+7], %0, %1 offset:1024 sc0 sc1\n\t" \
                   "global_load_dwordx4 v[%c2+8:%c2+11], %0, %1 offset:2048 sc0 sc1\n\tglobal_load_dwordx4 v[%c2+12:%c2+15], %0, %1 offset:3072 sc0 sc1" \
                   :: "v"((o) + l16), "s"(src), "i"(R) : CLB)
#define ST2(R, c, q) asm volatile("global_store_dwordx4 %0, v[%c2:%c2+3], %1 offset:%c3 sc0 sc1\n\tglobal_store_dwordx4 %0, v[%c2+4:%c2+7], %1 offset:%c3+1024 sc0 sc1" \
                   :: "v"(_o), "s"(dst[q]), "i"((R) + 4 * (c)), "i"((c) * 1024) : CLB)
#define ST14(R, c) ST2(R, c, 0); ST2(R, c, 1); ST2(R, c, 2); ST2(R, c, 3); ST2(R, c, 4); ST2(R, c, 5); ST2(R, c, 6)
#define STOREU(R, o) do { const unsigned _o = (o) + l16; ST14(R, 0); ST14(R, 2); } while (0)
#define FLAGU(u) do { char* _f = fl_base + (size_t)(u) * 64u; \
      asm volatile("s_mov_b64 exec, 0xff\n\tglobal_store_dwordx2 %0, %1, off sc0 sc1\n\ts_mov_b64 exec, -1" :: "v"(_f), "v"(call2) : CLB); } while (0)
#define READ_TK(tk) asm volatile("v_readfirstlane_b32 %0, v223" : "=s"(tk) :: CLB)
#define WAIT28() asm volatile("s_waitcnt vmcnt(28)" ::: CLB)
    unsigned tk;
    DRAW(qctr); asm volatile("s_waitcnt vmcnt(0)" ::: CLB); READ_TK(tk);
    unsigned idx = tk - qbase;
    if (!wait_words((unsigned)WS_LFLAG + lane * 4u, 4, 256u, 0ull)) return;
    unsigned cur = NUNITS / 2;
    if (idx < npush) {
      unsigned prev = cur; cur = decode((int)idx);
      unsigned o = data_off(cur);
      LOADU(224, o); DRAW(qctr);
      asm volatile("s_waitcnt vmcnt(0)" ::: CLB);
      for (;;) {
        LOADU(240, o + KTS);
        STOREU(224, o);
        READ_TK(tk); idx = tk - qbase;
        const bool more = idx < npush;
        unsigned nxt = cur, on = o;
        if (more) { nxt = decode((int)idx); on = data_off(nxt); }
        WAIT28();
        FLAGU(prev);
        if (more) { LOADU(224, on); DRAW(qctr); } else DRAW(qctr + 128);
        STOREU(240, o + KTS);
        WAIT28();
        if (!more) break;
        prev = cur; cur = nxt; o = on;
      }
    } else { DRAW(qctr + 128); asm volatile("s_waitcnt vmcnt(0)" ::: CLB); }
    {
      constexpr unsigned nfw = 8 * 32, ROW = N_ * 4u;
      const unsigned qb2 = (call - 1u) * (nfw + (unsigned)NAG);
      const char* const osrc = (const char*)p.out;
#pragma unroll
      for (int q = 0; q < NR - 1; ++q) dst[q] = p.outs[(rk + 1 + (q + (int)blockIdx.x) % (NR - 1)) & 7];
      const unsigned obase = (unsigned)(rk & 1) * 2048u * ROW + (unsigned)(60 + (rk >> 1)) * 1024u;
#define OLD(R, o) asm volatile("global_load_dwordx4 v[%c5:%c5+3], %0, %4 sc0 sc1\n\tglobal_load_dwordx4 v[%c5+4:%c5+7], %1, %4 sc0 sc1\n\t" \
        "global_load_dwordx4 v[%c5+8:%c5+11], %2, %4 sc0 sc1\n\tglobal_load_dwordx4 v[%c5+12:%c5+15], %3, %4 sc0 sc1" \
        :: "v"((o) + l16), "v"((o) + l16 + ROW), "v"((o) + l16 + 2 * ROW), "v"((o) + l16 + 3 * ROW), "s"(osrc), "i"(R) : CLB)
#define OS2(R, c, q) asm volatile("global_store_dwordx4 %0, v[%c3:%c3+3], %2 sc0 sc1 nt\n\tglobal_store_dwordx4 %1, v[%c3+4:%c3+7], %2 sc0 sc1 nt" \
        :: "v"(_o + (c) * ROW), "v"(_o + (c) * ROW + ROW), "s"(dst[q]), "i"((R) + 4 * (c)) : CLB)
#define OS14(R, c) OS2(R, c, 0); OS2(R, c, 1); OS2(R, c, 2); OS2(R, c, 3); OS2(R, c, 4); OS2(R, c, 5); OS2(R, c, 6)
#define OST(R, o) do { const unsigned _o = (o) + l16; OS14(R, 0); OS14(R, 2); } while (0)
      READ_TK(tk); idx = tk - qb2;
      unsigned go = *(volatile LAS unsigned*)lds;
      if (__builtin_amdgcn_readfirstlane((int)go) == 0) {
        asm volatile("s_waitcnt vmcnt(0)" ::: CLB); FLAGU(cur);
        do { __builtin_amdgcn_s_sleep(4); go = *(volatile LAS unsigned*)lds; } while (__builtin_amdgcn_readfirstlane((int)go) == 0);
      }
      unsigned o = obase + (idx < nfw ? idx : 0u) * (8u * ROW);
      const bool fwd = __builtin_amdgcn_readfirstlane((int)go) == 1;
      if (fwd && idx < nfw) { OLD(224, o); DRAW(qctr + 128); }
      asm volatile("s_waitcnt vmcnt(0)" ::: CLB);
      FLAGU(cur);
      if (!fwd) { asm volatile("s_waitcnt vmcnt(0)" ::: CLB); return; }
      if (idx < nfw) {
        for (;;) {
          OLD(240, o + 4u * ROW); OST(224, o);
          READ_TK(tk); idx = tk - qb2;
          const bool more = idx < nfw; const unsigned on = obase + idx * (8u * ROW);
          WAIT28();
          if (more) { OLD(224, on); DRAW(qctr + 128); }
          OST(240, o + 4u * ROW);
          if (!more) break;
          WAIT28();
          o = on;
        }
      }
      asm volatile("s_waitcnt vmcnt(0)" ::: CLB);
      if (lane < NR) __hip_atomic_store((unsigned*)(p.ws[lane & 7] + WS_DONE + blockIdx.x * 32 + rk * 4), call, __ATOMIC_RELAXED, __HIP_MEMORY_SCOPE_SYSTEM);
      if (!wait_words((unsigned)WS_DONE + blockIdx.x * 32u + (lane & 7) * 4u, 1, 0u, 0ull))
        *(float*)((char*)p.out + obase + blockIdx.x * 4u) = __builtin_nanf("");
    }
    asm volatile("s_waitcnt vmcnt(0)" ::: "memory");
    return;
  }

  const int wr = wid >> 2, wc = wid & 3, fr = lane & 15, fq = lane >> 4;
  int pm, slot;
  { const int b = blockIdx.x - NAG, x = b & 7, r = b >> 3, a = x >> 1, hb = x & 1;
    if (r < 28) { pm = 4 * a + (r & 3); slot = hb * 8 + (r >> 2); } else { pm = 4 * a + 2 * hb + (r - 28); slot = 7; } }
  const bool has_half = slot == 0, sharer = slot == SSLOT && (pm >> 3) == (rk & 1);
  constexpr int HCOL = 15 * SROUND + SSLOT;
  const int hblk = 8 * (rk & 1) + (pm >> 1), hrow = pm & 1;
  auto col_of = [&](int rnd) { return (sharer && rnd == SROUND) ? 60 + (rk >> 1) : 15 * rnd + slot; };
  const float scale = __builtin_bit_cast(float, __builtin_amdgcn_readfirstlane(__builtin_bit_cast(int, p.sx[0] * p.sw[0])));
  const unsigned voff = (unsigned)((wid >> 2) * 4 + fq) * SRCB + (unsigned)((wid & 3) * 256 + fr * 16);
  const unsigned lds0 = (unsigned)(size_t)lds, ldsw = lds0 + LDS_OP + (unsigned)wid * 2048u;
  int aoff = LDS_OP + wr * 1024 + lane * 16, boff = LDS_OP + 4 * HTB + wc * 2048 + lane * 16;
  asm volatile("" : "+v"(aoff), "+v"(boff));
  const unsigned coff = (unsigned)((wr * 64 + 4 * fq) * (N_ * 4) + (wc * 64 + 4 * fr) * 4);
  const u32x4 srd = {(unsigned)(size_t)myws, (unsigned)((size_t)myws >> 32) & 0xffffu, (unsigned)WS_NEED, 0x00020000u};
  const unsigned cA = (unsigned)par + (unsigned)pm * 4096u;
  const unsigned cW = (unsigned)par + 16u * 4096u;
  const unsigned hA = (unsigned)par + (unsigned)hblk * 4096u + hrow * GH, hB = cW + HCOL * 4096u;
  const unsigned long long own = 0x0003000300030003ull << (2 * rk);
#define SA(b, h) (((b) * 2 + (h)) * HTB)
#define SB(b, h) ((4 + (b) * 2 + (h)) * HTB)
#define STAGE(bufoff, goff) asm volatile("s_add_u32 m0, %3, %4\n\ts_nop 0\n\tbuffer_load_dwordx4 %0, %1, %2 offen sc0 sc1 lds\n\t" \
    "buffer_load_dwordx4 %0, %1, %2 offen offset:1024 sc0 sc1 lds" :: "v"(voff), "s"(srd), "s"(goff), "s"(ldsw), "i"(bufoff) : "scc", "memory")
#define FLAG_FETCH(ko, sl) asm volatile("s_mov_b32 m0, %3\n\ts_nop 0\n\tbuffer_load_dword %0, %1, %2 offen sc0 sc1 lds" :: "v"(vf), "s"(srd), "s"((ko) >> 7), "s"(lds0 + (sl)) : "memory")
#define LDA_(dst, base) do { _Pragma("unroll") for (int m = 0; m < 4; ++m) _Pragma("unroll") for (int k = 0; k < 2; ++k) dst[m][k] = *(const LAS i32x4*)(lds + (base) + m * 2048 + k * 8192); } while (0)
#define LDB_(dst, base) do { _Pragma("unroll") for (int n = 0; n < 2; ++n) _Pragma("unroll") for (int k = 0; k < 2; ++k) dst[n][k] = *(const LAS i32x4*)(lds + (base) + n * 1024 + k * 8192); } while (0)
#define LDAH(dst, base, m0) do { _Pragma("unroll") for (int m = (m0); m < (m0) + 2; ++m) _Pragma("unroll") for (int k = 0; k < 2; ++k) dst[m][k] = *(const LAS i32x4*)(lds + (base) + m * 2048 + k * 8192); } while (0)
#define LDA(dst, b, h) LDA_(dst, SA(b, h) + aoff)
#define LDB(dst, b, h) LDB_(dst, SA(b, h) + boff)
#define MFMA2(c, a0, b0, a1, b1, Z) do { \
    if (Z) asm volatile("v_mfma_i32_16x16x64_i8 %0, %1, %2, 0.15915494\n\tv_mfma_i32_16x16x64_i8 %0, %3, %4, %0" : "+v"(c) : "v"(a0), "v"(b0), "v"(a1), "v"(b1)); \
    else asm volatile("v_mfma_i32_16x16x64_i8 %0, %1, %2, %0\n\tv_mfma_i32_16x16x64_i8 %0, %3, %4, %0" : "+v"(c) : "v"(a0), "v"(b0), "v"(a1), "v"(b1)); } while (0)
#define MMA2(ai, Z, PARK) do { __builtin_amdgcn_s_setprio(1); \
    _Pragma("unroll") for (int i = 0; i < 16; ++i) { \
      const int k = i & 1, m = i >> 2, c = (m & 1) ? 3 - (i & 3) : (i & 3); \
      if ((PARK) == 1 && m >= 2) asm volatile("v_mfma_i32_16x16x64_i8 %0, %1, %2, 0.15915494\n\tv_mfma_i32_16x16x64_i8 %0, %3, %4, %0" \
          : "=&v"(park[m & 1][c]) : "v"(At[m][k]), "v"(c < 2 ? B0[c & 1][k] : B1[c & 1][k]), "v"(At[m][k ^ 1]), "v"(c < 2 ? B0[c & 1][k ^ 1] : B1[c & 1][k ^ 1])); \
      else if ((PARK) == 2 && m >= 2) asm volatile("v_mfma_i32_16x16x64_i8 %0, %1, %2, %5\n\tv_mfma_i32_16x16x64_i8 %0, %3, %4, %0" \
          : "+v"(acc[ai][c >> 1][m][c & 1]) : "v"(At[m][k]), "v"(c < 2 ? B0[c & 1][k] : B1[c & 1][k]), "v"(At[m][k ^ 1]), "v"(c < 2 ? B0[c & 1][k ^ 1] : B1[c & 1][k ^ 1]), "v"(park[m & 1][c])); \
      else MFMA2(acc[ai][c >> 1][m][c & 1], At[m][k], (c < 2 ? B0[c & 1][k] : B1[c & 1][k]), At[m][k ^ 1], (c < 2 ? B0[c & 1][k ^ 1] : B1[c & 1][k ^ 1]), Z); } \
    __builtin_amdgcn_s_setprio(0); } while (0)
#define WAIT_V(n) asm volatile("s_waitcnt vmcnt(" #n ")" ::: "memory")
#define WAIT_L(n) do { asm volatile("" ::: "memory"); __builtin_amdgcn_s_waitcnt(0xC07F); asm volatile("" ::: "memory"); } while (0)
#define BAR __builtin_amdgcn_s_barrier()
#define SCHED __builtin_amdgcn_sched_barrier(0)
#define PIN do { asm volatile("" ::: "memory"); SCHED; } while (0)
#define EPI_(s0, s1, OPS, POL) do { \
    _Pragma("unroll") for (int s = (s0); s < (s1); ++s) { const int ai = s >> 4, m = (s >> 2) & 3, j = s & 3; float f0, f1, f2, f3; \
      asm volatile(OPS : "=&v"(f0), "=&v"(f1), "=&v"(f2), "=&v"(f3) \
                   : "v"(acc[ai][0][m][0][j]), "v"(acc[ai][0][m][1][j]), "v"(acc[ai][1][m][0][j]), "v"(acc[ai][1][m][1][j]), "s"(scale)); \
      const f32x4 v = {f0, f1, f2, f3}; \
      asm volatile("buffer_store_dwordx4 %0, %1, %2, %3 offen " POL "\n\ts_nop 1" :: "v"(v), "v"(coff), "s"(crs), "s"((ai * 128 + m * 16 + j) * (N_ * 4)) : "memory"); } } while (0)
#define OPS_MUL "v_mul_f32 %0, %8, %0\n\tv_mul_f32 %1, %8, %1\n\tv_mul_f32 %2, %8, %2\n\tv_mul_f32 %3, %8, %3"
#define OPS_SAT "v_sub_u32 %0, %4, 0.15915494 clamp\n\tv_sub_u32 %1, %5, 0.15915494 clamp\n\tv_sub_u32 %2, %6, 0.15915494 clamp\n\tv_sub_u32 %3, %7, 0.15915494 clamp\n\t" \
    "v_cvt_f32_u32 %0, %0\n\tv_cvt_f32_u32 %1, %1\n\tv_cvt_f32_u32 %2, %2\n\tv_cvt_f32_u32 %3, %3\n\t" OPS_MUL
#define OPS_ANY "v_subrev_u32 %0, 0.15915494, %4\n\tv_subrev_u32 %1, 0.15915494, %5\n\tv_subrev_u32 %2, 0.15915494, %6\n\tv_subrev_u32 %3, 0.15915494, %7\n\t" \
    "v_cvt_f32_i32 %0, %0\n\tv_cvt_f32_i32 %1, %1\n\tv_cvt_f32_i32 %2, %2\n\tv_cvt_f32_i32 %3, %3\n\t" OPS_MUL "\n\t" \
    "v_max_f32 %0, 0, %0\n\tv_max_f32 %1, 0, %1\n\tv_max_f32 %2, 0, %2\n\tv_max_f32 %3, 0, %3"
#define EPI_ST(s0, s1) do { if ((s1) > (s0)) { if (__builtin_expect(wt >= wt_min, 0)) EPI_(s0, s1, OPS_ANY, "sc0 sc1"); else EPI_(s0, s1, OPS_SAT, "nt"); } } while (0)
  auto flag_off = [&](int rnd) -> unsigned { const int l = lane_id();
    return (unsigned)WS_FLAG + (unsigned)(((l & 16) ? 16 + col_of(rnd) : pm) * 64 + (l & 15) * 4); };
  auto nextk = [&](unsigned k) -> unsigned { k += 2 * KTS; return k < 32 * KTS ? k : 0u; };

  i32x4 acc[2][2][4][2];
  i32x4 At[4][2], B0[2][2], B1[2][2], park[2][4];
  unsigned k0 = (unsigned)start_of(slot) * KTS, k1 = nextk(k0);
  unsigned vf = flag_off(0);
  if (wid == 0) {
    wait_words(vf + (k0 >> 7), 1, 0u, own);
    wait_words((unsigned)WS_LFLAG + lane_id() * 4u, 4, 256u, 0ull);
    FLAG_FETCH(k1, 0u);
  }
  asm volatile("s_waitcnt vmcnt(0)" ::: "memory");
  BAR;
  { const unsigned a = cA + k0, b = cW + (unsigned)slot * 4096u + k0;
    STAGE(SB(0, 0), b); STAGE(SB(0, 1), b + GH); STAGE(SA(0, 0), a); STAGE(SA(0, 1), a + GH);
    if (wr == 1) BAR;
    WAIT_V(2); BAR;
    STAGE(SB(1, 0), b + KTS); STAGE(SA(1, 0), a + KTS); STAGE(SB(1, 1), b + GH + KTS);
    WAIT_V(0); BAR; PIN; }
#pragma unroll
  for (int a = 0; a < 2; ++a)
#pragma unroll
    for (int b = 0; b < 2; ++b)
#pragma unroll
      for (int m = 0; m < 4; ++m)
#pragma unroll
        for (int n = 0; n < 2; ++n) asm volatile("" : "=v"(acc[a][b][m][n]));
  auto mk_rs = [](const float* b, unsigned n) { return (u32x4){(unsigned)(size_t)b, (unsigned)((size_t)b >> 32) & 0xffffu, n, 0x00020000u}; };
  u32x4 crs = mk_rs(p.out, 0u);
  const unsigned wt_min = __builtin_bit_cast(unsigned, scale) < 0x7f800000u ? 60u : 0u;
  unsigned wt = 0;
  unsigned fsl = 0;
#define KT(b, ah, a2, b2, Z, PARK, V1, V2, e0, e1, e2, FL) do { \
    unsigned fv = 0; \
    LDB(B0, b, 0); STAGE(SA((b) ^ 1, 1), ah); LDB(B1, b, 1); SCHED; LDA(At, b, 0); \
    if ((FL) && wid == 0) { fv = *(const LAS unsigned*)(lds + fsl + lane_id() * 4); if (newf) vf = flag_off(un); FLAG_FETCH(k2, fsl ^ 256u); } \
    PIN; EPI_ST(e0, e1); PIN; \
    WAIT_V(V1); WAIT_L(0); \
    if ((FL) && wid == 0) { if ((__ballot((int)(fv - call) >= 0) | own) != ~0ull) wait_words(flag_off(c1) + (k1 >> 7), 1, 0u, own); \
      if ((FL) == 2 && tohalf) { const int l = lane_id(); const unsigned fo = (unsigned)WS_FLAG + (unsigned)((l >> 4) * (int)(KTS >> 6) + (l & 15) * 4); \
        wait_words(fo + (16 + HCOL) * 64, 4, KTS >> 4, own); wait_words(fo + (unsigned)hblk * 64u, 4, KTS >> 4, own); } \
      if (pub == 2) __hip_atomic_store((unsigned*)(myws + WS_OFLAG) + (pm & 7), call, __ATOMIC_RELAXED, __HIP_MEMORY_SCOPE_AGENT); \
    } \
    BAR; MMA2(0, Z, 0); BAR; PIN; \
    STAGE(SB(b, 0), b2); LDAH(At, SA(b, 1) + aoff, 0); STAGE(SB(b, 1), (b2) + GH); LDAH(At, SA(b, 1) + aoff, 2); STAGE(SA(b, 0), a2); \
    PIN; EPI_ST(e1, e2); PIN; \
    WAIT_V(V2); WAIT_L(0); BAR; MMA2(1, Z, PARK); BAR; PIN; } while (0)
#define PAIR(C1, NEWF, PUB, Z, Va, Vb, Vc, Vd, e0, e1, e2, e3, e4, FL) do { \
    const int c1 = (C1); const bool newf = (NEWF); const int pub = (PUB); const unsigned k2 = nextk(k1); \
    unsigned a1 = cA + k1, b1 = cW + (unsigned)col_of(c1) * 4096u + k1, ah1 = a1 + GH; \
    if ((FL) == 2 && tohalf) { a1 = hA; b1 = hB; ah1 = hB + 2 * KTS; }     \
    KT(0, cA + k0 + KTS + GH, a1, b1, Z, Z, Va, Vb, e0, e1, e2, FL); \
    KT(1, ah1, a1 + KTS, b1 + KTS, 0, 2 * (Z), Vc, Vd, e2, e3, e4, 0); \
    k0 = k1; k1 = k2; fsl ^= 256u; } while (0)
  for (int ui = 0;; ++ui) {
    const bool has_next = ui < 3, tohalf = !has_next && has_half;
    const int un = has_next ? ui + 1 : ui;
    PAIR(ui, false, 0, 1, 24, 32, 28, 24, 8, 16, 24, 28, 32, 1);
    crs = mk_rs(p.out + (size_t)pm * 256 * N_ + col_of(ui) * 256, 256u * N_ * 4u);
    wt = (unsigned)col_of(ui);
    for (int t = 1; t < 15; ++t) PAIR(ui, t == 14, (sharer && ui == SROUND + 1) ? t : 0, 0, 8, 8, 8, 8, 0, 0, 0, 0, 0, 1);
    PAIR(un, false, 0, 0, 8, 8, 8, 16, 0, 0, 0, 0, 8, 2);
    if (!has_next) break;
  }
  if (has_half) {
    int aoff2 = 8 * HTB + aoff; asm volatile("" : "+v"(aoff2));
    const int boff2 = boff - 4 * HTB;
#define HST(j, w, g) do { if ((j) == 0) { if ((w) == 0) STAGE(SA(0, 1), g); else if ((w) == 1) STAGE(SA(1, 1), g); else STAGE(8 * HTB, g); } \
      else { if ((w) == 0) STAGE(SB(((j) + 1) & 1, 0), g); else if ((w) == 1) STAGE(SB(((j) + 1) & 1, 1), g); else STAGE(SA(((j) + 1) & 1, 0), g); } } while (0)
#define HKT(j, tn, Z, V, e0, e1) do { \
      const unsigned a = hA + (unsigned)(tn) * KTS, b = hB + (unsigned)(tn) * KTS; \
      if ((j) < 2) LDB(B0, j, 0); else LDB_(B0, SA(0, 1) + boff2); \
      HST(j, 0, b); \
      if ((j) < 2) LDB(B1, j, 1); else LDB_(B1, SA(1, 1) + boff2); \
      HST(j, 1, b + GH); \
      SCHED; if ((j) < 2) LDA(At, j, 0); else LDA_(At, aoff2); \
      HST(j, 2, a); \
      PIN; EPI_ST(e0, e1); PIN; \
      WAIT_V(V); WAIT_L(0); BAR; MMA2(0, Z, 0); BAR; PIN; } while (0)
    HKT(0, 2, 1, 22, 8, 16); HKT(1, 3, 0, 18, 16, 20); HKT(2, 4, 0, 14, 20, 24);
    HKT(0, 5, 0, 14, 24, 28); HKT(1, 6, 0, 14, 28, 32); HKT(2, 7, 0, 10, 0, 0);
    crs = mk_rs(p.out + (size_t)(hblk * 256 + hrow * 128) * N_ + HCOL * 256, 128u * N_ * 4u);
    for (int t = 6; t < 30; t += 3) { HKT(0, t + 2, 0, 6, 0, 0); HKT(1, t + 3, 0, 6, 0, 0); HKT(2, t + 4, 0, 6, 0, 0); }
    HKT(0, 31, 0, 6, 0, 0); HKT(1, 31, 0, 6, 0, 0);
  }
  const unsigned gave_up = __hip_atomic_load((unsigned*)(myws + WS_TMO), __ATOMIC_RELAXED, __HIP_MEMORY_SCOPE_AGENT);
  asm volatile("s_nop 7" ::: "memory"); SCHED;
  if (has_half) EPI_ST(0, 4); else EPI_ST(8, 12);
  if (wr == 0) BAR;
  if (has_half) EPI_ST(4, 16); else EPI_ST(12, 32);
  PIN;
  if (__builtin_amdgcn_readfirstlane((int)gave_up) == (int)call) {
    WAIT_V(0);
    const float q = __builtin_nanf(""); const f32x4 v = {q, q, q, q};
    for (int ui = 0; ui < (has_half ? 5 : 4); ++ui) {
      crs = ui < 4 ? mk_rs(p.out + (size_t)pm * 256 * N_ + col_of(ui) * 256, 256u * N_ * 4u)
                   : mk_rs(p.out + (size_t)(hblk * 256 + hrow * 128) * N_ + HCOL * 256, 128u * N_ * 4u);
      for (int s = 0; s < 32; ++s)
        asm volatile("buffer_store_dwordx4 %0, %1, %2, %3 offen\n\ts_nop 1" :: "v"(v), "v"(coff), "s"(crs), "s"(((s >> 4) * 128 + ((s >> 2) & 3) * 16 + (s & 3)) * (N_ * 4)) : "memory");
    }
  }
  WAIT_V(0);
}

extern "C" void kernel_launch(int rank, int nranks, ncclComm_t comm, hipStream_t stream,
                              void* const* d_in, const int* in_sizes, int n_in,
                              void* d_out, int out_size, const int* in_dtypes, int out_dtype,
                              void* const* d_out_all, void* const* d_ws_all, size_t ws_bytes) {
  (void)comm; (void)in_dtypes; (void)out_dtype;
  if (nranks != NR || (unsigned)rank >= (unsigned)NR || n_in != 4 || ws_bytes < WS_NEED || out_size != M_ * N_ ||
      in_sizes[0] != M_ * KS || in_sizes[1] != KS * N_) {
    fprintf(stderr, "kernel_launch: unexpected configuration\n");
    return;
  }
  Params p;
  for (int i = 0; i < NR; ++i) p.ws[i] = (char*)d_ws_all[i];
  p.x = (const signed char*)d_in[0]; p.w = (const signed char*)d_in[1];
  p.sx = (const float*)d_in[2]; p.sw = (const float*)d_in[3];
  p.out = (float*)d_out; p.rank = rank;
  for (int i = 0; i < NR; ++i) p.outs[i] = (char*)d_out_all[i];
  hipLaunchKernelGGL(fused_kernel, dim3(GRID), dim3(512), 0, stream, p);
}
```
